```python
import numpy as np

kernel_name = "gpu_gemm_rs_m4096_k4096_n16384_e4m3e4m3_dqrelu"


ABSMAX_RTOL = 0.03

COMPUTE_DTYPE = "float32"
EPILOGUE = "dequant_relu"
IN_QUANT = None
OUT_QUANT = None
IN_FP8 = "e4m3e4m3"
IN_DTYPES = {"x": "e4m3", "w_mat": "e4m3",
             "scale_x": "float32", "scale_w": "float32"}
WORLD_SIZE = 8

def _snap_fp8(x, mant_bits, max_abs, min_normal):
    a = np.abs(x.astype(np.float64))
    _m, _e = np.frexp(a)
    q = np.where(a >= min_normal, np.ldexp(1.0, _e - 1 - mant_bits),
                 min_normal * 2.0 ** -mant_bits)
    snapped = np.minimum(np.round(a / q) * q, max_abs)
    return (np.sign(x) * snapped).astype(np.float32)

def setup_inputs():

    x = _snap_fp8(np.random.randn(4096, 4096).astype(np.float32), 3, 448.0, 2.0 ** -6)
    w_mat = _snap_fp8(np.random.randn(4096, 16384).astype(np.float32), 3, 448.0, 2.0 ** -6)
    scale_x = (0.02 * (1.0 + np.random.rand(1))).astype(np.float32)
    scale_w = (0.02 * (1.0 + np.random.rand(1))).astype(np.float32)
    return {"x": x, "w_mat": w_mat, "scale_x": scale_x, "scale_w": scale_w}

def reference(x, w_mat, scale_x, scale_w):
    acc = (x @ w_mat).astype(np.float32)
    y = (acc * (scale_x * scale_w)).astype(np.float32)
    return np.maximum(y, 0.0).astype(np.float32)
```

```cpp
#include <hip/hip_runtime.h>
#include <rccl/rccl.h>
#include <cstdio>
#include <cstdint>

#define HIP_CHECK(expr) do { hipError_t _e = (expr); if (_e != hipSuccess) { \
    fprintf(stderr, "HIP error %s at %s:%d\n", hipGetErrorString(_e), __FILE__, __LINE__); \
    return; } } while (0)

namespace {

typedef float f32x4 __attribute__((ext_vector_type(4)));
typedef float f32x16 __attribute__((ext_vector_type(16)));
typedef int i32x4 __attribute__((ext_vector_type(4)));
typedef int i32x8 __attribute__((ext_vector_type(8)));
typedef unsigned u32x4 __attribute__((ext_vector_type(4)));
#define LAS __attribute__((address_space(3)))

constexpr int NR = 8;
constexpr int GM = 4096, GK = 4096, GN = 16384;
constexpr int KSH = GK / NR;
constexpr int MSH = GM / NR;
constexpr int NSL = GN / NR;
constexpr int NRB = 16, NCB = 16;
constexpr int NKS = GK / 64;
constexpr int NWG = NRB * NCB;
constexpr int NTHR = 512;

constexpr unsigned OFF_FX = 0;
constexpr unsigned OFF_FW = 2048;
constexpr unsigned OFF_TMO = 4096;
constexpr unsigned OFF_TOK = 5120;
constexpr unsigned OFF_FD = 8192;
constexpr unsigned OFF_CNT = OFF_FD + NWG * 128;
constexpr unsigned OFF_DBG = OFF_CNT + NWG * 128;
constexpr unsigned OFF_XS = 1u << 20;
constexpr unsigned XS_BYTES = (unsigned)GM * GK;
constexpr unsigned OFF_WS = OFF_XS + XS_BYTES;
constexpr unsigned WS_BYTES = (unsigned)GK * NSL;
constexpr unsigned WS_TOTAL = OFF_WS + WS_BYTES;
constexpr unsigned OUT_BYTES = (unsigned)MSH * GN * 4;

constexpr int NS = 6;
constexpr int SLOT = 24576;
constexpr int LDS_BYTES = NS * SLOT + 64;
constexpr unsigned long long DEADLINE_TICKS = 50000000ull;
constexpr int AUX_SYS = 17;

struct Params {
    const uint8_t* x;
    const uint8_t* w;
    const float* sx;
    const float* sw;
    uint8_t* out[NR];
    uint8_t* ws[NR];
    int rank;
};

#define WAIT_VM(n) asm volatile("s_waitcnt vmcnt(%0)" :: "n"(n) : "memory")
#define WAIT_LGKM0() asm volatile("s_waitcnt lgkmcnt(0)" ::: "memory")
#define BAR() do { asm volatile("" ::: "memory"); __builtin_amdgcn_s_barrier(); asm volatile("" ::: "memory"); } while (0)

#define STAMP(k) do {} while (0)

__device__ __forceinline__ __amdgpu_buffer_rsrc_t make_rsrc(void* p, unsigned bytes) {
    return __builtin_amdgcn_make_buffer_rsrc(p, 0, (int)bytes, 0x00020000);
}
__device__ __forceinline__ void flag_store(uint8_t* p, unsigned v) {
    __hip_atomic_store((unsigned*)p, v, __ATOMIC_RELAXED, __HIP_MEMORY_SCOPE_SYSTEM);
}
#define POLL_SLEEP 16
__device__ __forceinline__ bool gave_up(unsigned long long t0, unsigned& polls, unsigned call, uint8_t* tmo) {
    if ((++polls & 63u) != 0u) return false;
    const unsigned t = __hip_atomic_load((const unsigned*)tmo, __ATOMIC_RELAXED, __HIP_MEMORY_SCOPE_AGENT);
    if (t == call || __builtin_amdgcn_s_memrealtime() - t0 > DEADLINE_TICKS) {
        __hip_atomic_store((unsigned*)tmo, call, __ATOMIC_RELAXED, __HIP_MEMORY_SCOPE_AGENT);
        return true;
    }
    return false;
}
__device__ __forceinline__ bool wait_flags(const uint8_t* fp, unsigned call, uint8_t* tmo) {
    const unsigned long long t0 = __builtin_amdgcn_s_memrealtime();
    unsigned polls = 0;
    for (;;) {
        unsigned v = call;
        if (fp) v = __hip_atomic_load((const unsigned*)fp, __ATOMIC_RELAXED, __HIP_MEMORY_SCOPE_SYSTEM);
        if (__all(v >= call)) return true;
        if (gave_up(t0, polls, call, tmo)) return false;
        __builtin_amdgcn_s_sleep(POLL_SLEEP);
    }
}

#define MAP 0
#define TOKW 32
#define TOK_SLEEP 12
#define PWIN 1
#define EWIN 2
#define GATE 16
#define TOKREL 3
constexpr unsigned TOK_IN = 192, TOK_CALL = TOK_IN + NWG;
__device__ __forceinline__ bool wait_token(const uint8_t* donep, unsigned need, unsigned call, uint8_t* tmo) {
    const unsigned long long t0 = __builtin_amdgcn_s_memrealtime();
    unsigned polls = 0;
    for (;;) {
        const unsigned v = __builtin_amdgcn_readfirstlane(__hip_atomic_load((const unsigned*)donep, __ATOMIC_RELAXED, __HIP_MEMORY_SCOPE_AGENT));
        const int d = (int)(need - v);
        if (d <= 0) return true;
        if (gave_up(t0, polls, call, tmo)) return false;
        if (d > TOKW + 8) { __builtin_amdgcn_s_sleep(127); __builtin_amdgcn_s_sleep(127); }
        else __builtin_amdgcn_s_sleep(TOK_SLEEP);
    }
}
constexpr int NPD = 8;
constexpr int NROUND = 96 / NPD;
constexpr int WROUND = 32 / NPD;

struct Chunk { unsigned d[16]; };

__device__ __forceinline__ void push_load(Chunk& c, const Params& p, int r, int pi, unsigned dest, int wv, int lane) {
    const int l = lane & 31, kg = lane >> 5;
    if (r < WROUND) {
        const int q = r * NPD + pi, j = q >> 1, half = q & 1, sl = 4 * half + (wv >> 1), h = wv & 1;
        const uint8_t* src = p.w + (size_t)(sl * 64 + h * 32 + kg * 16) * GN + dest * NSL + j * 128 + 4 * l;
#pragma unroll
        for (int kk = 0; kk < 16; ++kk) c.d[kk] = *(const unsigned*)(src + (size_t)kk * GN);
    } else {
        const int q = (r - WROUND) * NPD + pi, i = q >> 2, quarter = q & 3;
        const uint8_t* src = p.x + (size_t)(wv * MSH + i * 32 + l) * KSH + quarter * 128 + kg * 16;
#pragma unroll
        for (int u = 0; u < 4; ++u) {
            const u32x4 v = *(const u32x4*)(src + u * 32);
            c.d[4 * u] = v.x; c.d[4 * u + 1] = v.y; c.d[4 * u + 2] = v.z; c.d[4 * u + 3] = v.w;
        }
    }
}
struct Out4 { u32x4 v[4]; unsigned off[4]; };
__device__ __forceinline__ void push_prep(Out4& o, const Chunk& c, int rank, int r, int pi, int wv, int lane) {
    if (r < WROUND) {
        const int q = r * NPD + pi, j = q >> 1, half = q & 1, sl = 4 * half + (wv >> 1), h = wv & 1;
#pragma unroll
        for (int m = 0; m < 4; ++m) {
            const unsigned d0 = c.d[4 * m], d1 = c.d[4 * m + 1], d2 = c.d[4 * m + 2], d3 = c.d[4 * m + 3];
            const unsigned t0 = __builtin_amdgcn_perm(d1, d0, 0x05010400u), t1 = __builtin_amdgcn_perm(d1, d0, 0x07030602u);
            const unsigned t2 = __builtin_amdgcn_perm(d3, d2, 0x05010400u), t3 = __builtin_amdgcn_perm(d3, d2, 0x07030602u);
            o.v[0][m] = __builtin_amdgcn_perm(t2, t0, 0x05040100u);
            o.v[1][m] = __builtin_amdgcn_perm(t2, t0, 0x07060302u);
            o.v[2][m] = __builtin_amdgcn_perm(t3, t1, 0x05040100u);
            o.v[3][m] = __builtin_amdgcn_perm(t3, t1, 0x07060302u);
        }
        const unsigned off = OFF_WS + (unsigned)(j * NKS + rank * 8 + sl) * 8192u + h * 1024 + lane * 16;
#pragma unroll
        for (int cc = 0; cc < 4; ++cc) o.off[cc] = off + cc * 2048;
    } else {
        const int q = (r - WROUND) * NPD + pi, i = q >> 2, quarter = q & 3;
        const unsigned off = OFF_XS + (unsigned)(i * NKS + rank * 8 + 2 * quarter) * 16384u + wv * 2048 + lane * 16;
#pragma unroll
        for (int u = 0; u < 4; ++u) {
            o.v[u] = (u32x4){c.d[4 * u], c.d[4 * u + 1], c.d[4 * u + 2], c.d[4 * u + 3]};
            o.off[u] = off + (u >> 1) * 16384 + (u & 1) * 1024;
        }
    }
}
__device__ __forceinline__ unsigned chunk_flag_off(int rank, int r, int pi) {
    if (r < WROUND) { const int q = r * NPD + pi; return OFF_FW + (q >> 1) * 128 + (rank * 2 + (q & 1)) * 4; }
    const int q = (r - WROUND) * NPD + pi; return OFF_FX + (q >> 2) * 128 + (rank * 4 + (q & 3)) * 4;
}

struct Frags { i32x4 a[2]; i32x4 b[4][2]; };
__device__ __forceinline__ void read_frags(Frags& f, LAS const uint8_t* pa, LAS const uint8_t* pb) {
    f.a[0] = *(LAS const i32x4*)(pa); f.a[1] = *(LAS const i32x4*)(pa + 1024);
#pragma unroll
    for (int c = 0; c < 4; ++c) { f.b[c][0] = *(LAS const i32x4*)(pb + c * 2048); f.b[c][1] = *(LAS const i32x4*)(pb + c * 2048 + 1024); }
}
__device__ __forceinline__ i32x8 cat(i32x4 a, i32x4 b) { return __builtin_shufflevector(a, b, 0, 1, 2, 3, 4, 5, 6, 7); }
__device__ __forceinline__ void mma(f32x16 (&acc)[4], const Frags& f) {
    const i32x8 a = cat(f.a[0], f.a[1]);
#pragma unroll
    for (int c = 0; c < 4; ++c)
        acc[c] = __builtin_amdgcn_mfma_scale_f32_32x32x64_f8f6f4(a, cat(f.b[c][0], f.b[c][1]), acc[c], 0, 0, 0, 127, 0, 127);
}

__global__ void __launch_bounds__(NTHR, 2) gemm_xchg(const Params p) {
    extern __shared__ __attribute__((aligned(16))) uint8_t smem_raw[];
    LAS uint8_t* lds = (LAS uint8_t*)smem_raw;
    const int tid = threadIdx.x, lane = tid & 63, wv = __builtin_amdgcn_readfirstlane(tid >> 6);
    const int rank = p.rank;
    uint8_t* myws = p.ws[rank];
    const int b = blockIdx.x, xcd = b & 7, slot = b >> 3;
    const int ti = slot >> 1, tj = 2 * xcd + (slot & 1);
    const int tile = ti * NCB + tj;
    STAMP(0);
    unsigned* cntp = (unsigned*)(myws + OFF_CNT + b * 128);
    const unsigned call = __builtin_amdgcn_readfirstlane(__hip_atomic_load(cntp, __ATOMIC_RELAXED, __HIP_MEMORY_SCOPE_AGENT)) + 1u;

    if (slot >= 32 - NPD) {
        const int pi = xcd % NPD + 0, dsel = slot - (32 - NPD);
        const unsigned dest = (unsigned)(rank + 1 + dsel) & (NR - 1);
        uint8_t* dws = p.ws[dest];
        const __amdgpu_buffer_rsrc_t rs = make_rsrc(dws, WS_TOTAL);
        unsigned* tokdone = (unsigned*)(myws + OFF_TOK + dest * 256 + 128);
        Chunk c;
        push_load(c, p, 0, pi, dest, wv, lane);
#pragma unroll
        for (int r = 0; r < NROUND; ++r) {
            Out4 o;
            push_prep(o, c, rank, r, pi, wv, lane);
            const int nl = r + 1 >= NROUND ? 0 : (r + 1 < WROUND ? 16 : 4);
            WAIT_VM(PWIN - 1);
            __builtin_amdgcn_raw_buffer_store_b128(o.v[0], rs, o.off[0], 0, AUX_SYS);
            if (PWIN == 2) {
                WAIT_VM(1);
                __builtin_amdgcn_raw_buffer_store_b128(o.v[1], rs, o.off[1], 0, AUX_SYS);
            }
            if (r > 0) {
                BAR();
                if (tid == 0) { flag_store(dws + chunk_flag_off(rank, r - 1, pi), call);
                                __hip_atomic_fetch_add(tokdone, 2u, __ATOMIC_RELAXED, __HIP_MEMORY_SCOPE_AGENT); }
            }
            asm volatile("" ::: "memory");
            if (r + 1 < NROUND) push_load(c, p, r + 1, pi, dest, wv, lane);
            if (PWIN == 1) {
                WAIT_VM(nl);
                __builtin_amdgcn_raw_buffer_store_b128(o.v[1], rs, o.off[1], 0, AUX_SYS);
                WAIT_VM(0);
                __builtin_amdgcn_raw_buffer_store_b128(o.v[2], rs, o.off[2], 0, AUX_SYS);
                WAIT_VM(0);
                __builtin_amdgcn_raw_buffer_store_b128(o.v[3], rs, o.off[3], 0, AUX_SYS);
            } else {
                WAIT_VM(nl + 1);
                __builtin_amdgcn_raw_buffer_store_b128(o.v[2], rs, o.off[2], 0, AUX_SYS);
                WAIT_VM(nl + 1);
                __builtin_amdgcn_raw_buffer_store_b128(o.v[3], rs, o.off[3], 0, AUX_SYS);
            }
        }
        WAIT_VM(0);
        BAR();
        if (tid == 0) { flag_store(dws + chunk_flag_off(rank, NROUND - 1, pi), call);
                        __hip_atomic_fetch_add(tokdone, 2u, __ATOMIC_RELAXED, __HIP_MEMORY_SCOPE_AGENT); }
    }
    STAMP(1);
    LAS unsigned* ldsflag = (LAS unsigned*)(lds + NS * SLOT);
    if (wv == 0) {
        const uint8_t* fp = lane < 32 ? myws + OFF_FX + ti * 128 + lane * 4 : (lane < 48 ? myws + OFF_FW + tj * 128 + (lane - 32) * 4 : nullptr);
        const bool ok = wait_flags(fp, call, myws + OFF_TMO);
        __builtin_amdgcn_fence(__ATOMIC_ACQUIRE, "");
        if (lane == 0) *ldsflag = ok ? 0u : 1u;
        asm volatile("s_waitcnt vmcnt(0) lgkmcnt(0)" ::: "memory");
    }
    BAR();
    STAMP(2);
    f32x16 acc[4];
#pragma unroll
    for (int c = 0; c < 4; ++c)
#pragma unroll
        for (int r = 0; r < 16; ++r) acc[c][r] = 0.f;
    {
        const uint8_t* ga = myws + OFF_XS + (size_t)ti * NKS * 16384 + wv * 2048 + lane * 16;
        const uint8_t* gb = myws + OFF_WS + (size_t)tj * NKS * 8192 + wv * 1024 + lane * 16;
        auto dma = [&](int s, int sl) {
            LAS uint8_t* d = lds + sl * SLOT;
            __builtin_amdgcn_global_load_lds((const void*)(ga + (size_t)s * 16384), (LAS void*)(d + wv * 2048), 16, 0, 0);
            __builtin_amdgcn_global_load_lds((const void*)(ga + (size_t)s * 16384 + 1024), (LAS void*)(d + wv * 2048 + 1024), 16, 0, 0);
            __builtin_amdgcn_global_load_lds((const void*)(gb + (size_t)s * 8192), (LAS void*)(d + 16384 + wv * 1024), 16, 0, 0);
        };
        LAS const uint8_t* pa = lds + wv * 2048 + lane * 16;
        LAS const uint8_t* pb = lds + 16384 + lane * 16;
#pragma unroll
        for (int s = 0; s < NS; ++s) dma(s, s);
        WAIT_VM(3 * (NS - 1));
        BAR();
        Frags f0, f1;
        read_frags(f0, pa, pb);
        int sl = 0;
#define KSTEP(FC, FN, S, VMN, DO_DMA, DO_READ) do { \
            WAIT_VM(VMN); WAIT_LGKM0(); BAR(); \
            const int nsl = sl + 1 == NS ? 0 : sl + 1; \
            if (DO_READ) read_frags(FN, pa + nsl * SLOT, pb + nsl * SLOT); \
            if (DO_DMA) dma((S) + NS, sl); \
            mma(acc, FC); \
            sl = nsl; } while (0)
        for (int s = 0; s < NKS - NS; s += 2) {
            KSTEP(f0, f1, s, 3 * (NS - 2), true, true);
            KSTEP(f1, f0, s + 1, 3 * (NS - 2), true, true);
        }
        static_assert(NS == 6 && (NKS - NS) % 2 == 0, "tail below is written for NS == 6");
        KSTEP(f0, f1, 58, 12, false, true);
        KSTEP(f1, f0, 59, 9, false, true);
        KSTEP(f0, f1, 60, 6, false, true);
        KSTEP(f1, f0, 61, 3, false, true);
        KSTEP(f0, f1, 62, 0, false, true);
        mma(acc, f1);
#undef KSTEP
    }
    STAMP(3);
    {
        bool bad = *ldsflag != 0u;
        const float scale = p.sx[0] * p.sw[0];
        const unsigned dest = (unsigned)wv & (NR - 1);
        uint8_t* tok = myws + OFF_TOK + dest * 256;
        if (dest != (unsigned)rank) {
            unsigned t = 0;
            if (lane == 0) t = __hip_atomic_fetch_add((unsigned*)tok, 1u, __ATOMIC_RELAXED, __HIP_MEMORY_SCOPE_AGENT);
            t = __builtin_amdgcn_readfirstlane(t) - (call - 1u) * NWG;
            const unsigned need = t < TOKW ? TOK_IN - GATE : TOK_IN + t + 1u - TOKW;
            if (!wait_token(tok + 128, (call - 1u) * TOK_CALL + need, call, myws + OFF_TMO)) bad = true;
        }
        const __amdgpu_buffer_rsrc_t ro = make_rsrc(p.out[dest], OUT_BYTES);
        const unsigned off0 = (unsigned)(ti * 32 + 4 * (lane >> 5)) * (GN * 4) + (unsigned)(rank * NSL + tj * 128 + 4 * (lane & 31)) * 4;
#pragma unroll
        for (int r = 0; r < 16; ++r) {
            f32x4 v;
#pragma unroll
            for (int c = 0; c < 4; ++c) v[c] = fmaxf(acc[c][r] * scale, 0.f);
            if (bad) v = (f32x4){__builtin_nanf(""), __builtin_nanf(""), __builtin_nanf(""), __builtin_nanf("")};
            if (r >= EWIN) { if (r > 16 - TOKREL && r <= 16 - TOKREL + EWIN - 1) WAIT_VM(EWIN); else WAIT_VM(EWIN - 1); }
            if (TOKREL > 0 && r == 16 - TOKREL && lane == 0) __hip_atomic_fetch_add((unsigned*)(tok + 128), 1u, __ATOMIC_RELAXED, __HIP_MEMORY_SCOPE_AGENT);
            __builtin_amdgcn_raw_buffer_store_b128(__builtin_bit_cast(u32x4, v), ro, off0 + (unsigned)((r & 3) + 8 * (r >> 2)) * (GN * 4), 0, AUX_SYS);
        }
        WAIT_VM(0);
        if (TOKREL == 0 && lane == 0) __hip_atomic_fetch_add((unsigned*)(tok + 128), 1u, __ATOMIC_RELAXED, __HIP_MEMORY_SCOPE_AGENT);
        if (lane == 0) flag_store(p.ws[dest] + OFF_FD + tile * 128 + rank * 4, call);
    }
    STAMP(4);
    if (wv == rank) {
        const uint8_t* fp = (lane < NR && lane != rank) ? myws + OFF_FD + tile * 128 + lane * 4 : nullptr;
        if (lane == 0) __hip_atomic_store(cntp, call, __ATOMIC_RELAXED, __HIP_MEMORY_SCOPE_AGENT);
        const bool ok = wait_flags(fp, call, myws + OFF_TMO);
        if (!ok && lane == 0)
            *(float*)(p.out[rank] + (size_t)(ti * 32) * (GN * 4) + (size_t)(rank * NSL + tj * 128) * 4) = __builtin_nanf("");
    }
}

}

extern "C" void kernel_launch(int rank, int nranks, ncclComm_t comm, hipStream_t stream,
                              void* const* d_in, const int* in_sizes, int n_in,
                              void* d_out, int out_size, const int* in_dtypes, int out_dtype,
                              void* const* d_out_all, void* const* d_ws_all, size_t ws_bytes) {
    (void)comm; (void)in_dtypes; (void)out_dtype; (void)d_out;
    if (nranks != NR || n_in != 4 || in_sizes[0] != GM * KSH || in_sizes[1] != KSH * GN || out_size != MSH * GN ||
        ws_bytes < WS_TOTAL || (unsigned)rank >= (unsigned)NR) {
        fprintf(stderr, "kernel_launch: unexpected shapes\n");
        return;
    }
    Params p;
    p.x = (const uint8_t*)d_in[0]; p.w = (const uint8_t*)d_in[1];
    p.sx = (const float*)d_in[2]; p.sw = (const float*)d_in[3];
    for (int i = 0; i < NR; ++i) { p.out[i] = (uint8_t*)d_out_all[i]; p.ws[i] = (uint8_t*)d_ws_all[i]; }
    p.rank = rank;
    HIP_CHECK(hipFuncSetAttribute((const void*)gemm_xchg, hipFuncAttributeMaxDynamicSharedMemorySize, LDS_BYTES));
    hipLaunchKernelGGL(gemm_xchg, dim3(NWG), dim3(NTHR), LDS_BYTES, stream, p);
    HIP_CHECK(hipGetLastError());
}
```

```cpp
#include <hip/hip_runtime.h>
#include <rccl/rccl.h>
#include <cstdio>
#include <cstdint>

#define HIP_CHECK(expr) do { hipError_t _e = (expr); if (_e != hipSuccess) { \
    fprintf(stderr, "HIP error %s at %s:%d\n", hipGetErrorString(_e), __FILE__, __LINE__); \
    return; } } while (0)

namespace {

typedef float f32x4 __attribute__((ext_vector_type(4)));
typedef float f32x16 __attribute__((ext_vector_type(16)));
typedef int i32x4 __attribute__((ext_vector_type(4)));
typedef int i32x8 __attribute__((ext_vector_type(8)));
typedef unsigned u32x4 __attribute__((ext_vector_type(4)));
#define LAS __attribute__((address_space(3)))

constexpr int NR = 8;
constexpr int GM = 4096, GK = 4096, GN = 16384;
constexpr int KSH = GK / NR;
constexpr int MSH = GM / NR;
constexpr int NSL = GN / NR;
constexpr int NRB = 16, NCB = 16;
constexpr int NKS = GK / 64;
constexpr int NWG = NRB * NCB;
constexpr int NTHR = 512;

constexpr unsigned OFF_FX = 0;
constexpr unsigned OFF_FW = 2048;
constexpr unsigned OFF_TMO = 4096;
constexpr unsigned OFF_TOK = 5120;
constexpr unsigned OFF_FD = 8192;
constexpr unsigned OFF_CNT = OFF_FD + NWG * 128;
constexpr unsigned OFF_DBG = OFF_CNT + NWG * 128;
constexpr unsigned OFF_XS = 1u << 20;
constexpr unsigned XS_BYTES = (unsigned)GM * GK;
constexpr unsigned OFF_WS = OFF_XS + XS_BYTES;
constexpr unsigned WS_BYTES = (unsigned)GK * NSL;
constexpr unsigned PK_SLOT = 17408;
constexpr unsigned PK_DATA_END = 512 + 16384;
constexpr unsigned OFF_PK = OFF_WS + WS_BYTES;
constexpr unsigned WS_TOTAL = OFF_PK + NR * NWG * PK_SLOT;
constexpr unsigned OUT_BYTES = (unsigned)MSH * GN * 4;

constexpr int NS = 6;
constexpr int SLOT = 24576;
constexpr int LDS_BYTES = NS * SLOT + 64;
constexpr unsigned long long DEADLINE_TICKS = 50000000ull;
constexpr int AUX_SYS = 17;

struct Params {
    const uint8_t* x;
    const uint8_t* w;
    const float* sx;
    const float* sw;
    uint8_t* out[NR];
    uint8_t* ws[NR];
    int rank;
};

#define WAIT_VM(n) asm volatile("s_waitcnt vmcnt(%0)" :: "n"(n) : "memory")
#define WAIT_LGKM0() asm volatile("s_waitcnt lgkmcnt(0)" ::: "memory")
#define BAR() do { asm volatile("" ::: "memory"); __builtin_amdgcn_s_barrier(); asm volatile("" ::: "memory"); } while (0)

#define STAMP(k) do {} while (0)
#define STAMP_FLUSH() do {} while (0)

__device__ __forceinline__ __amdgpu_buffer_rsrc_t make_rsrc(void* p, unsigned bytes) {
    return __builtin_amdgcn_make_buffer_rsrc(p, 0, (int)bytes, 0x00020000);
}
__device__ __forceinline__ void flag_store(uint8_t* p, unsigned v) {
    __hip_atomic_store((unsigned*)p, v, __ATOMIC_RELAXED, __HIP_MEMORY_SCOPE_SYSTEM);
}
#define POLL_SLEEP 16
__device__ __forceinline__ bool gave_up(unsigned long long t0, unsigned& polls, unsigned call, uint8_t* tmo) {
    if ((++polls & 63u) != 0u) return false;
    const unsigned t = __hip_atomic_load((const unsigned*)tmo, __ATOMIC_RELAXED, __HIP_MEMORY_SCOPE_AGENT);
    if (t == call || __builtin_amdgcn_s_memrealtime() - t0 > DEADLINE_TICKS) {
        __hip_atomic_store((unsigned*)tmo, call, __ATOMIC_RELAXED, __HIP_MEMORY_SCOPE_AGENT);
        return true;
    }
    return false;
}
__device__ __forceinline__ bool wait_flags(const uint8_t* fp, unsigned call, uint8_t* tmo) {
    const unsigned long long t0 = __builtin_amdgcn_s_memrealtime();
    unsigned polls = 0;
    for (;;) {
        unsigned v = call;
        if (fp) v = __hip_atomic_load((const unsigned*)fp, __ATOMIC_RELAXED, __HIP_MEMORY_SCOPE_SYSTEM);
        if (__all(v >= call)) return true;
        if (gave_up(t0, polls, call, tmo)) return false;
        __builtin_amdgcn_s_sleep(POLL_SLEEP);
    }
}

#define MAP 0
#define TOKW 96
#define TOK_SLEEP 12
#define ACT1 40
#define ACT2 20
#define ACT4 8
#define RX_SLEEP 8
#define GATE 24
#define TOKREL 2
constexpr unsigned TOK_IN = 192, TOK_CALL = TOK_IN + NWG;
__device__ __forceinline__ unsigned long long wait_token(const uint8_t* tokp, unsigned need, unsigned call, uint8_t* tmo) {
    const unsigned long long t0 = __builtin_amdgcn_s_memrealtime();
    unsigned polls = 0;
    for (;;) {
        const unsigned long long w = __hip_atomic_load((const unsigned long long*)tokp, __ATOMIC_RELAXED, __HIP_MEMORY_SCOPE_AGENT);
        const unsigned head = __builtin_amdgcn_readfirstlane((unsigned)w), v = __builtin_amdgcn_readfirstlane((unsigned)(w >> 32));
        const int d = (int)(need - v);
        if (d <= 0 || gave_up(t0, polls, call, tmo)) return ((unsigned long long)v << 32) | head;
        if (d > TOKW + 8) __builtin_amdgcn_s_sleep(127);
        else __builtin_amdgcn_s_sleep(TOK_SLEEP);
    }
}
constexpr int NPD = 8;
constexpr int NCHUNK = 96, WCHUNK = 32;
constexpr unsigned INTK_CALL = NCHUNK - 2 * NPD + NPD;
#define NBLAST 0

struct Chunk { unsigned d[16]; };
__device__ __forceinline__ void load_w(Chunk& c, const Params& p, int q, unsigned dest, int wv, int lane) {
    const int l = lane & 31, kg = lane >> 5;
    const int j = q >> 1, half = q & 1, sl = 4 * half + (wv >> 1), h = wv & 1;
    const uint8_t* src = p.w + (size_t)(sl * 64 + h * 32 + kg * 16) * GN + dest * NSL + j * 128 + 4 * l;
#pragma unroll
    for (int kk = 0; kk < 16; ++kk) c.d[kk] = *(const unsigned*)(src + (size_t)kk * GN);
}
__device__ __forceinline__ void load_x(Chunk& c, const Params& p, int q, int wv, int lane) {
    const int l = lane & 31, kg = lane >> 5;
    const int i = (q - WCHUNK) >> 2, quarter = q & 3;
    const uint8_t* src = p.x + (size_t)(wv * MSH + i * 32 + l) * KSH + quarter * 128 + kg * 16;
#pragma unroll
    for (int u = 0; u < 4; ++u) {
        const u32x4 v = *(const u32x4*)(src + u * 32);
        c.d[4 * u] = v.x; c.d[4 * u + 1] = v.y; c.d[4 * u + 2] = v.z; c.d[4 * u + 3] = v.w;
    }
}
struct Out4 { u32x4 v[4]; unsigned off, step1, step2; };
__device__ __forceinline__ void push_prep(Out4& o, const Chunk& c, int rank, int q, int wv, int lane) {
    if (q < WCHUNK) {
        const int j = q >> 1, half = q & 1, sl = 4 * half + (wv >> 1), h = wv & 1;
#pragma unroll
        for (int m = 0; m < 4; ++m) {
            const unsigned d0 = c.d[4 * m], d1 = c.d[4 * m + 1], d2 = c.d[4 * m + 2], d3 = c.d[4 * m + 3];
            const unsigned t0 = __builtin_amdgcn_perm(d1, d0, 0x05010400u), t1 = __builtin_amdgcn_perm(d1, d0, 0x07030602u);
            const unsigned t2 = __builtin_amdgcn_perm(d3, d2, 0x05010400u), t3 = __builtin_amdgcn_perm(d3, d2, 0x07030602u);
            o.v[0][m] = __builtin_amdgcn_perm(t2, t0, 0x05040100u);
            o.v[1][m] = __builtin_amdgcn_perm(t2, t0, 0x07060302u);
            o.v[2][m] = __builtin_amdgcn_perm(t3, t1, 0x05040100u);
            o.v[3][m] = __builtin_amdgcn_perm(t3, t1, 0x07060302u);
        }
        o.off = OFF_WS + (unsigned)(j * NKS + rank * 8 + sl) * 8192u + h * 1024 + lane * 16;
        o.step1 = 2048; o.step2 = 4096;
    } else {
        const int i = (q - WCHUNK) >> 2, quarter = q & 3;
#pragma unroll
        for (int u = 0; u < 4; ++u) o.v[u] = (u32x4){c.d[4 * u], c.d[4 * u + 1], c.d[4 * u + 2], c.d[4 * u + 3]};
        o.off = OFF_XS + (unsigned)(i * NKS + rank * 8 + 2 * quarter) * 16384u + wv * 2048 + lane * 16;
        o.step1 = 1024; o.step2 = 16384;
    }
}
__device__ __forceinline__ unsigned chunk_flag_off(int rank, int q) {
    if (q < WCHUNK) return OFF_FW + (q >> 1) * 128 + (rank * 2 + (q & 1)) * 4;
    return OFF_FX + ((q - WCHUNK) >> 2) * 128 + (rank * 4 + (q & 3)) * 4;
}

struct Frags { i32x4 a[2]; i32x4 b[4][2]; };
__device__ __forceinline__ void read_frags(Frags& f, LAS const uint8_t* pa, LAS const uint8_t* pb) {
    f.a[0] = *(LAS const i32x4*)(pa); f.a[1] = *(LAS const i32x4*)(pa + 1024);
#pragma unroll
    for (int c = 0; c < 4; ++c) { f.b[c][0] = *(LAS const i32x4*)(pb + c * 2048); f.b[c][1] = *(LAS const i32x4*)(pb + c * 2048 + 1024); }
}
__device__ __forceinline__ i32x8 cat(i32x4 a, i32x4 b) { return __builtin_shufflevector(a, b, 0, 1, 2, 3, 4, 5, 6, 7); }
__device__ __forceinline__ void mma(f32x16 (&acc)[4], const Frags& f) {
    const i32x8 a = cat(f.a[0], f.a[1]);
#pragma unroll
    for (int c = 0; c < 4; ++c)
        acc[c] = __builtin_amdgcn_mfma_scale_f32_32x32x64_f8f6f4(a, cat(f.b[c][0], f.b[c][1]), acc[c], 0, 0, 0, 127, 0, 127);
}

__global__ void __launch_bounds__(NTHR, 2) gemm_xchg(const Params p) {
    extern __shared__ __attribute__((aligned(16))) uint8_t smem_raw[];
    LAS uint8_t* lds = (LAS uint8_t*)smem_raw;
    const int tid = threadIdx.x, lane = tid & 63, wv = __builtin_amdgcn_readfirstlane(tid >> 6);
    const int rank = p.rank;
    uint8_t* myws = p.ws[rank];
    const int b = blockIdx.x, xcd = b & 7, slot = b >> 3;
    const int ti = slot >> 1, tj = 2 * xcd + (slot & 1);
    const int tile = ti * NCB + tj;
    STAMP(0);
    unsigned* cntp = (unsigned*)(myws + OFF_CNT + b * 128);
    const unsigned cnt_v = __hip_atomic_load(cntp, __ATOMIC_RELAXED, __HIP_MEMORY_SCOPE_AGENT);
    unsigned call;

    if (slot >= 32 - NPD) {
        const int pi = xcd, dsel = slot - (32 - NPD);
        const unsigned dest = (unsigned)(rank + 1 + dsel) & (NR - 1);
        uint8_t* dws = p.ws[dest];
        const __amdgpu_buffer_rsrc_t rs = make_rsrc(dws, WS_TOTAL);
        unsigned* tokdone = (unsigned*)(myws + OFF_TOK + dest * 256 + 4);
        unsigned* intk = (unsigned*)(myws + OFF_TOK + dest * 256 + 128);
        LAS unsigned* tk = (LAS unsigned*)(lds + NS * SLOT + 16);
        Chunk c;
        load_w(c, p, pi, dest, wv, lane);
        call = __builtin_amdgcn_readfirstlane(cnt_v) + 1u;
        const unsigned tkbase = (call - 1u) * INTK_CALL - 2 * NPD;
        int prev = -1, cur = pi, nxt = NPD + pi, par = 0;
        unsigned drawn = 0;
        for (;;) {
            Out4 o;
            push_prep(o, c, rank, cur, wv, lane);
            const bool blast = cur >= NCHUNK - NBLAST;
            WAIT_VM(0);
            __builtin_amdgcn_raw_buffer_store_b128(o.v[0], rs, o.off, 0, AUX_SYS);
            if (blast) {
                __builtin_amdgcn_raw_buffer_store_b128(o.v[1], rs, o.off + o.step1, 0, AUX_SYS);
                __builtin_amdgcn_raw_buffer_store_b128(o.v[2], rs, o.off + o.step2, 0, AUX_SYS);
                __builtin_amdgcn_raw_buffer_store_b128(o.v[3], rs, o.off + o.step1 + o.step2, 0, AUX_SYS);
            }
            if (prev >= 0) {
                if (tid == 0) tk[par] = drawn;
                WAIT_LGKM0();
                BAR();
                if (tid == 0) { flag_store(dws + chunk_flag_off(rank, prev), call);
                                __hip_atomic_fetch_add(tokdone, 2u, __ATOMIC_RELAXED, __HIP_MEMORY_SCOPE_AGENT); }
                nxt = (int)(__builtin_amdgcn_readfirstlane(tk[par]) - tkbase);
                par ^= 1;
            }
            asm volatile("" ::: "memory");
            const bool more = (unsigned)nxt < (unsigned)NCHUNK;
            if (more && nxt < WCHUNK) { load_w(c, p, nxt, dest, wv, lane); WAIT_VM(16); }
            else if (more) { load_x(c, p, nxt, wv, lane); if (!blast) WAIT_VM(4); }
            else if (!blast) WAIT_VM(0);
            if (!blast) {
                __builtin_amdgcn_raw_buffer_store_b128(o.v[1], rs, o.off + o.step1, 0, AUX_SYS);
                WAIT_VM(0);
                __builtin_amdgcn_raw_buffer_store_b128(o.v[2], rs, o.off + o.step2, 0, AUX_SYS);
                WAIT_VM(0);
                __builtin_amdgcn_raw_buffer_store_b128(o.v[3], rs, o.off + o.step1 + o.step2, 0, AUX_SYS);
            }
            prev = cur; cur = nxt;
            if (!more) break;
            if (tid == 0) drawn = __hip_atomic_fetch_add(intk, 1u, __ATOMIC_RELAXED, __HIP_MEMORY_SCOPE_AGENT);
        }
        WAIT_VM(0);
        BAR();
        if (tid == 0) { flag_store(dws + chunk_flag_off(rank, prev), call);
                        __hip_atomic_fetch_add(tokdone, 2u, __ATOMIC_RELAXED, __HIP_MEMORY_SCOPE_AGENT); }
    } else {
        call = __builtin_amdgcn_readfirstlane(cnt_v) + 1u;
    }
    STAMP(1);
    LAS unsigned* ldsflag = (LAS unsigned*)(lds + NS * SLOT);
    if (wv == 0) {
        const uint8_t* fp = lane < 32 ? myws + OFF_FX + ti * 128 + lane * 4 : (lane < 48 ? myws + OFF_FW + tj * 128 + (lane - 32) * 4 : nullptr);
        const bool ok = wait_flags(fp, call, myws + OFF_TMO);
        __builtin_amdgcn_fence(__ATOMIC_ACQUIRE, "");
        if (lane == 0) *ldsflag = ok ? 0u : 1u;
        asm volatile("s_waitcnt vmcnt(0) lgkmcnt(0)" ::: "memory");
    }
    BAR();
    STAMP(2);
    f32x16 acc[4];
#pragma unroll
    for (int c = 0; c < 4; ++c)
#pragma unroll
        for (int r = 0; r < 16; ++r) acc[c][r] = 0.f;
    {
        const uint8_t* ga = myws + OFF_XS + (size_t)ti * NKS * 16384 + wv * 2048 + lane * 16;
        const uint8_t* gb = myws + OFF_WS + (size_t)tj * NKS * 8192 + wv * 1024 + lane * 16;
        auto dma = [&](int s, int sl) {
            LAS uint8_t* d = lds + sl * SLOT;
            __builtin_amdgcn_global_load_lds((const void*)(ga + (size_t)s * 16384), (LAS void*)(d + wv * 2048), 16, 0, 0);
            __builtin_amdgcn_global_load_lds((const void*)(ga + (size_t)s * 16384 + 1024), (LAS void*)(d + wv * 2048 + 1024), 16, 0, 0);
            __builtin_amdgcn_global_load_lds((const void*)(gb + (size_t)s * 8192), (LAS void*)(d + 16384 + wv * 1024), 16, 0, 0);
        };
        LAS const uint8_t* pa = lds + wv * 2048 + lane * 16;
        LAS const uint8_t* pb = lds + 16384 + lane * 16;
#pragma unroll
        for (int s = 0; s < NS; ++s) dma(s, s);
        WAIT_VM(3 * (NS - 1));
        BAR();
        Frags f0, f1;
        read_frags(f0, pa, pb);
        int sl = 0;
#define KSTEP(FC, FN, S, VMN, DO_DMA, DO_READ) do { \
            WAIT_VM(VMN); WAIT_LGKM0(); BAR(); \
            const int nsl = sl + 1 == NS ? 0 : sl + 1; \
            if (DO_READ) read_frags(FN, pa + nsl * SLOT, pb + nsl * SLOT); \
            if (DO_DMA) dma((S) + NS, sl); \
            mma(acc, FC); \
            sl = nsl; } while (0)
        for (int s = 0; s < NKS - NS; s += 2) {
            KSTEP(f0, f1, s, 3 * (NS - 2), true, true);
            KSTEP(f1, f0, s + 1, 3 * (NS - 2), true, true);
        }
        static_assert(NS == 6 && (NKS - NS) % 2 == 0, "tail below is written for NS == 6");
        KSTEP(f0, f1, 58, 12, false, true);
        KSTEP(f1, f0, 59, 9, false, true);
        KSTEP(f0, f1, 60, 6, false, true);
        KSTEP(f1, f0, 61, 3, false, true);
        KSTEP(f0, f1, 62, 0, false, true);
        mma(acc, f1);
#undef KSTEP
    }
    STAMP(3);
    BAR();
    const float qnan = __builtin_nanf("");
    const float scale = *ldsflag != 0u ? qnan : p.sx[0] * p.sw[0];
    const unsigned peer = (unsigned)wv & (NR - 1);
    const __amdgpu_buffer_rsrc_t ro = make_rsrc(p.out[rank], OUT_BYTES);
    const unsigned off0 = (unsigned)(ti * 32 + 4 * (lane >> 5)) * (GN * 4) + (unsigned)(peer * NSL + tj * 128 + 4 * (lane & 31)) * 4;
    if (peer == (unsigned)rank) {
#pragma unroll
        for (int r = 0; r < 16; ++r) {
            f32x4 v;
#pragma unroll
            for (int c = 0; c < 4; ++c) { const float y = acc[c][r] * scale; v[c] = y <= 0.f ? 0.f : y; }
            __builtin_amdgcn_raw_buffer_store_b128(__builtin_bit_cast(u32x4, v), ro, off0 + (unsigned)((r & 3) + 8 * (r >> 2)) * (GN * 4), 0, AUX_SYS);
        }
        if (lane == 0) __hip_atomic_store(cntp, call, __ATOMIC_RELAXED, __HIP_MEMORY_SCOPE_AGENT);
        STAMP(4); STAMP(5); STAMP(6);
        STAMP_FLUSH();
        return;
    }
    typedef unsigned u32x2 __attribute__((ext_vector_type(2)));
    LAS uint8_t* pk = lds + wv * PK_SLOT;
    uint8_t* tok = myws + OFF_TOK + peer * 256;
    {
        unsigned t = 0;
        if (lane == 0) t = __hip_atomic_fetch_add((unsigned*)tok, 1u, __ATOMIC_RELAXED, __HIP_MEMORY_SCOPE_AGENT);
        unsigned mk[2] = {0u, 0u}, base = 512;
        const unsigned dump = PK_DATA_END + lane * 4;
#pragma unroll
        for (int r = 0; r < 16; ++r)
#pragma unroll
            for (int c = 0; c < 4; ++c) {
                const float y = acc[c][r] * scale;
                const bool nz = !(y <= 0.f);
                const unsigned long long m = __ballot(nz);
                const unsigned pos = __builtin_amdgcn_mbcnt_hi((unsigned)(m >> 32), __builtin_amdgcn_mbcnt_lo((unsigned)m, 0u)) * 4 + base;
                *(LAS float*)(pk + (nz ? pos : dump)) = y;
                base += (unsigned)__builtin_popcountll(m) * 4;
                mk[r >> 3] = mk[r >> 3] * 2 + (nz ? 1u : 0u);
            }
        *(LAS u32x2*)(pk + lane * 8) = (u32x2){mk[0], mk[1]};
        const unsigned bytes = base;
        const int nst = (int)((bytes + 1023u) >> 10);
        t = __builtin_amdgcn_readfirstlane(t) - (call - 1u) * NWG;
        const unsigned need = t < TOKW ? TOK_IN - GATE : TOK_IN + t + 1u - TOKW;
        const unsigned long long seen = wait_token(tok, (call - 1u) * TOK_CALL + need, call, myws + OFF_TMO);
        const int fin = (int)((unsigned)(seen >> 32) - (call - 1u) * TOK_CALL - TOK_IN);
        const int act = fin < 0 ? 1000 : (int)((unsigned)seen - (call - 1u) * NWG) - fin;
        STAMP(4);
        uint8_t* pws = p.ws[peer];
        const __amdgpu_buffer_rsrc_t rp = make_rsrc(pws + OFF_PK + (unsigned)(rank * NWG + tile) * PK_SLOT, (bytes + 63u) & ~63u);
        const int rel = nst > TOKREL ? nst - TOKREL : 0;
        u32x4 v = *(LAS const u32x4*)(pk + lane * 16);
        for (int i = 0; i < nst; ++i) {
            const u32x4 cur = v;
            v = *(LAS const u32x4*)(pk + (i + 1 < nst ? i + 1 : i) * 1024 + lane * 16);
            if (act >= ACT1) WAIT_VM(0); else if (act >= ACT2) WAIT_VM(1); else if (act >= ACT4) WAIT_VM(3);
            if (i == rel && lane == 0) __hip_atomic_fetch_add((unsigned*)(tok + 4), 1u, __ATOMIC_RELAXED, __HIP_MEMORY_SCOPE_AGENT);
            __builtin_amdgcn_raw_buffer_store_b128(cur, rp, i * 1024 + lane * 16, 0, AUX_SYS);
        }
        WAIT_VM(0);
        if (lane == 0) __hip_atomic_store((unsigned long long*)(pws + OFF_FD + tile * 128 + rank * 8), ((unsigned long long)call << 32) | bytes,
                                          __ATOMIC_RELAXED, __HIP_MEMORY_SCOPE_SYSTEM);
        STAMP(5);
    }
    {
        const unsigned long long* fp = (const unsigned long long*)(myws + OFF_FD + tile * 128 + peer * 8);
        const unsigned long long t0 = __builtin_amdgcn_s_memrealtime();
        unsigned polls = 0, bytes = 0; bool ok = true;
        for (;;) {
            const unsigned long long f = __hip_atomic_load(fp, __ATOMIC_RELAXED, __HIP_MEMORY_SCOPE_SYSTEM);
            const unsigned hi = __builtin_amdgcn_readfirstlane((unsigned)(f >> 32));
            bytes = __builtin_amdgcn_readfirstlane((unsigned)f);
            if (hi >= call) break;
            if (gave_up(t0, polls, call, myws + OFF_TMO)) { ok = false; break; }
            __builtin_amdgcn_s_sleep(RX_SLEEP);
        }
        __builtin_amdgcn_fence(__ATOMIC_ACQUIRE, "");
        STAMP(6);
        if (!ok) {
            for (int r = 0; r < 16; ++r)
                __builtin_amdgcn_raw_buffer_store_b128(__builtin_bit_cast(u32x4, (f32x4){qnan, qnan, qnan, qnan}), ro, off0 + (unsigned)((r & 3) + 8 * (r >> 2)) * (GN * 4), 0, AUX_SYS);
            return;
        }
        if (bytes > PK_SLOT) bytes = PK_SLOT;
        const int nld = (int)((bytes + 1023u) >> 10);
        const uint8_t* src = myws + OFF_PK + (unsigned)(peer * NWG + tile) * PK_SLOT + lane * 16;
        for (int i = 0; i < nld; ++i)
            __builtin_amdgcn_global_load_lds((const void*)(src + i * 1024), (LAS void*)(pk + i * 1024), 16, 0, 0);
        WAIT_VM(0);
        const u32x2 mm = *(LAS const u32x2*)(pk + lane * 8);
        unsigned mk[2] = {mm.x, mm.y}, base = 512;
#pragma unroll
        for (int r = 0; r < 16; ++r) {
            f32x4 v;
#pragma unroll
            for (int c = 0; c < 4; ++c) {
                const bool nz = (int)(mk[r >> 3] << ((r & 7) * 4 + c)) < 0;
                const unsigned long long m = __ballot(nz);
                const unsigned pos = __builtin_amdgcn_mbcnt_hi((unsigned)(m >> 32), __builtin_amdgcn_mbcnt_lo((unsigned)m, 0u)) * 4 + base;
                const float y = *(LAS const float*)(pk + pos);
                base += (unsigned)__builtin_popcountll(m) * 4;
                v[c] = nz ? y : 0.f;
            }
            __builtin_amdgcn_raw_buffer_store_b128(__builtin_bit_cast(u32x4, v), ro, off0 + (unsigned)((r & 3) + 8 * (r >> 2)) * (GN * 4), 0, AUX_SYS);
        }
        STAMP_FLUSH();
    }
}

}

extern "C" void kernel_launch(int rank, int nranks, ncclComm_t comm, hipStream_t stream,
                              void* const* d_in, const int* in_sizes, int n_in,
                              void* d_out, int out_size, const int* in_dtypes, int out_dtype,
                              void* const* d_out_all, void* const* d_ws_all, size_t ws_bytes) {
    (void)comm; (void)in_dtypes; (void)out_dtype; (void)d_out;
    if (nranks != NR || n_in != 4 || in_sizes[0] != GM * KSH || in_sizes[1] != KSH * GN || out_size != MSH * GN ||
        ws_bytes < WS_TOTAL || (unsigned)rank >= (unsigned)NR) {
        fprintf(stderr, "kernel_launch: unexpected shapes\n");
        return;
    }
    Params p;
    p.x = (const uint8_t*)d_in[0]; p.w = (const uint8_t*)d_in[1];
    p.sx = (const float*)d_in[2]; p.sw = (const float*)d_in[3];
    for (int i = 0; i < NR; ++i) { p.out[i] = (uint8_t*)d_out_all[i]; p.ws[i] = (uint8_t*)d_ws_all[i]; }
    p.rank = rank;
    HIP_CHECK(hipFuncSetAttribute((const void*)gemm_xchg, hipFuncAttributeMaxDynamicSharedMemorySize, LDS_BYTES));
    hipLaunchKernelGGL(gemm_xchg, dim3(NWG), dim3(NTHR), LDS_BYTES, stream, p);
    HIP_CHECK(hipGetLastError());
}
```

```cpp
#include <hip/hip_runtime.h>
#include <rccl/rccl.h>
#include <cstdio>
#include <cstdint>

#define HIP_CHECK(expr) do { hipError_t _e = (expr); if (_e != hipSuccess) { \
    fprintf(stderr, "HIP error %s at %s:%d\n", hipGetErrorString(_e), __FILE__, __LINE__); \
    return; } } while (0)

namespace {

typedef float f32x4 __attribute__((ext_vector_type(4)));
typedef float f32x16 __attribute__((ext_vector_type(16)));
typedef int i32x4 __attribute__((ext_vector_type(4)));
typedef int i32x8 __attribute__((ext_vector_type(8)));
typedef unsigned u32x2 __attribute__((ext_vector_type(2)));
typedef unsigned u32x4 __attribute__((ext_vector_type(4)));
#define LAS __attribute__((address_space(3)))

constexpr int NR = 8;
constexpr int GM = 4096, GK = 4096, GN = 16384;
constexpr int KSH = GK / NR;
constexpr int MSH = GM / NR;
constexpr int NSL = GN / NR;
constexpr int NRB = 16, NCB = 16;
constexpr int NKS = GK / 64;
constexpr int NWG = NRB * NCB;
constexpr int NTHR = 512;

#define NCOMM 32
#define PWIN 2
#define LEAD 4
#define IN_AUX 17
constexpr int PB = 4096;
constexpr unsigned FXB = 128, FWB = 64;

constexpr unsigned OFF_FX = 0;
constexpr unsigned OFF_FW = 32768;
constexpr unsigned OFF_FS = 49152;
constexpr unsigned OFF_TMO = 50176;
constexpr unsigned OFF_TK = 51200;
constexpr unsigned FLAG_BYTES = 65536;
constexpr unsigned OFF_FD = 65536;
constexpr unsigned OFF_CNT = OFF_FD + NWG * 128;
constexpr unsigned OFF_DBG = OFF_CNT + NWG * 128;
constexpr unsigned OFF_XS = 1u << 20;
constexpr unsigned XS_BYTES = (unsigned)GM * GK;
constexpr unsigned OFF_WS = OFF_XS + XS_BYTES;
constexpr unsigned WS_BYTES = (unsigned)GK * NSL;
constexpr unsigned OFF_SW = OFF_WS + WS_BYTES;
constexpr unsigned SW_BYTES = (unsigned)KSH * GN;
constexpr unsigned PK_SLOT = 17408;
constexpr unsigned PK_DATA_END = 512 + 16384;
constexpr unsigned OFF_PK = OFF_SW + SW_BYTES;
constexpr unsigned WS_TOTAL = OFF_PK + NR * NWG * PK_SLOT;
constexpr unsigned OUT_BYTES = (unsigned)MSH * GN * 4;

constexpr int NS = 6;
constexpr int SLOT = 24576;
constexpr int LDS_BYTES = NS * SLOT + 64;
constexpr unsigned long long DEADLINE_TICKS = 50000000ull;
constexpr int AUX_SYS = 17;

#define NPUSH0 6
#define POLL_SLEEP 8
#define POLL_SLOW 32
#define RX_SLEEP 8
#define KS_R1 6
#define XMON 200
#define ADSL 2
#define ROT 3
#define ACQ 2
#define CHAIN 72
#define CHAIN_EARLY 2
#define GRANT_EARLY 16
#define PUSH_WIN 2
#define GATE_SLEEP 16
#define LAT_LO 85
#define LAT_HI 110
#define LAT_BACK 170
#define BACKOFF 16
#define WMAX 8

struct Params {
    const uint8_t* x;
    const uint8_t* w;
    const float* sx;
    const float* sw;
    uint8_t* out;
    uint8_t* ws[NR];
    int rank;
};

#define WAIT_VM(n) asm volatile("s_waitcnt vmcnt(%0)" :: "n"(n) : "memory")
#define WAIT_LGKM0() asm volatile("s_waitcnt lgkmcnt(0)" ::: "memory")
#define BAR() do { asm volatile("" ::: "memory"); __builtin_amdgcn_s_barrier(); asm volatile("" ::: "memory"); } while (0)

#define STAMP(k) do {} while (0)
#define STAMP_FLUSH() do {} while (0)

__device__ __forceinline__ __amdgpu_buffer_rsrc_t make_rsrc(const void* p, unsigned bytes) {
    return __builtin_amdgcn_make_buffer_rsrc((void*)p, 0, (int)bytes, 0x00020000);
}
__device__ __forceinline__ void flag_store(uint8_t* p, unsigned v) {
    __hip_atomic_store((unsigned*)p, v, __ATOMIC_RELAXED, __HIP_MEMORY_SCOPE_SYSTEM);
}
__device__ __forceinline__ bool gave_up(unsigned long long t0, unsigned& polls, unsigned call, uint8_t* tmo) {
    if ((++polls & 63u) != 0u) return false;
    const unsigned t = __builtin_amdgcn_readfirstlane(__hip_atomic_load((const unsigned*)tmo, __ATOMIC_RELAXED, __HIP_MEMORY_SCOPE_AGENT));
    if (t == call || __builtin_amdgcn_s_memrealtime() - t0 > DEADLINE_TICKS) {
        __hip_atomic_store((unsigned*)tmo, call, __ATOMIC_RELAXED, __HIP_MEMORY_SCOPE_AGENT);
        return true;
    }
    return false;
}
__device__ __forceinline__ bool wait_operands(uint8_t* ws, int ti, int tj, int lane, unsigned call, int ph, int rank) {
    const unsigned long long t0 = __builtin_amdgcn_s_memrealtime();
    const __amdgpu_buffer_rsrc_t rf = make_rsrc(ws, FLAG_BYTES);
    unsigned polls = 0;
    if (ph != 0) {
        for (;;) {
            unsigned v = call;
            if (lane < 8) v = __builtin_amdgcn_raw_buffer_load_b32(rf, OFF_FX + (ti * 8 + lane) * FXB + (ph == 1 ? 0 : 64) + tj * 4, 0, AUX_SYS);
            if (__builtin_popcountll(__ballot(v >= call)) >= 56 + 3) break;
            if (gave_up(t0, polls, call, ws + OFF_TMO)) return false;
            __builtin_amdgcn_s_sleep(POLL_SLOW);
            asm volatile("" ::: "memory");
        }
    }
    const bool xneed = ph == 0 ? (lane >> 3) == rank : (ph == 3 || ((lane >> 2) & 1) == ph - 1);
    const bool wneed = lane < 32 && (ph == 0 ? (lane >> 2) == rank : (ph == 3 || ((lane >> 1) & 1) == ph - 1));
    bool wok = false;
    for (;;) {
        u32x4 v = {call, call, call, call}, w = v;
        if (xneed) v = __builtin_amdgcn_raw_buffer_load_b128(rf, OFF_FX + ti * 8 * FXB + lane * 16, 0, AUX_SYS);
        if (wneed && !wok) w = __builtin_amdgcn_raw_buffer_load_b128(rf, OFF_FW + tj * 8 * FWB + lane * 16, 0, AUX_SYS);
        wok = __all(w.x >= call && w.y >= call && w.z >= call && w.w >= call);
        if (wok && __all(v.x >= call && v.y >= call && v.z >= call && v.w >= call)) return true;
        if (gave_up(t0, polls, call, ws + OFF_TMO)) return false;
        __builtin_amdgcn_s_sleep(POLL_SLEEP);
        asm volatile("" ::: "memory");
    }
}

static_assert(NCOMM == 32 && CHAIN >= NCOMM, "");
__device__ __forceinline__ void grant(uint8_t* ws, unsigned peer, unsigned call, unsigned first, int n, int lane) {
    const unsigned t = first - (call - 1u) * (CHAIN - NCOMM + NWG) + (unsigned)lane + NCOMM;
    if (lane < n && t < (unsigned)NWG) flag_store(ws + OFF_FD + t * 128 + 64 + peer * 8 + 4, call);
}

struct Frags { i32x4 a[2]; i32x4 b[4][2]; };
__device__ __forceinline__ void read_frags(Frags& f, LAS const uint8_t* pa, LAS const uint8_t* pb) {
    f.a[0] = *(LAS const i32x4*)(pa); f.a[1] = *(LAS const i32x4*)(pa + 1024);
#pragma unroll
    for (int c = 0; c < 4; ++c) { f.b[c][0] = *(LAS const i32x4*)(pb + c * 2048); f.b[c][1] = *(LAS const i32x4*)(pb + c * 2048 + 1024); }
}
__device__ __forceinline__ i32x8 cat(i32x4 a, i32x4 b) { return __builtin_shufflevector(a, b, 0, 1, 2, 3, 4, 5, 6, 7); }
template <int C0, int C1> __device__ __forceinline__ void mma(f32x16 (&acc)[4], const Frags& f) {
    const i32x8 a = cat(f.a[0], f.a[1]);
#pragma unroll
    for (int c = C0; c < C1; ++c)
        acc[c] = __builtin_amdgcn_mfma_scale_f32_32x32x64_f8f6f4(a, cat(f.b[c][0], f.b[c][1]), acc[c], 0, 0, 0, 127, 0, 127);
}

constexpr int NW1 = 8 - NPUSH0 / 2;
constexpr int NPC = (NW1 + 8 + 2 * NRB) * 16;
static_assert(NPUSH0 >= 2 && NPUSH0 % 2 == 0 && NPUSH0 <= 16 && NCOMM % 8 == 0 && 4 % PWIN == 0, "FS index 0 is the x flag");
struct Piece { unsigned so, dso, fo; int staged; };
__device__ __forceinline__ Piece piece_of(int pc, int rank, unsigned peer) {
    int V = pc >> 4;
    const int k = pc & 15;
    int isw, half, idx;
    if (V < NW1) { isw = 1; half = 0; idx = NPUSH0 / 2 + V; }
    else if ((V -= NW1) < LEAD) { isw = 0; half = 0; idx = V; }
    else if ((V -= LEAD) < 8) { isw = 1; half = 1; idx = V; }
    else if ((V -= 8) < 2 * (NRB - LEAD)) { isw = 0; half = (V & 1) ^ 1; idx = (V >> 1) + ((V & 1) ? LEAD : 0); }
    else { isw = 0; half = 1; idx = V - 2 * (NRB - LEAD) + NRB - LEAD; }
    Piece r;
    if (isw) {
        const unsigned j = 2 * idx + (k >> 3), o = half * 32768 + (k & 7) * PB;
        r.so = OFF_SW + (unsigned)rank * 1048576u + j * 65536u + o;
        r.dso = OFF_WS + (j * NKS + peer * 8) * 8192u + o;
        r.fo = OFF_FW + (j * 8 + peer) * FWB + (half * 8 + (k & 7)) * 4;
        r.staged = (int)(2 * j) + half;
    } else {
        r.so = r.dso = OFF_XS + ((unsigned)idx * NKS + peer * 8) * 16384u + (half * 16 + k) * PB;
        r.fo = OFF_FX + ((unsigned)idx * 8 + peer) * FXB + (half * 16 + k) * 4;
        r.staged = 0;
    }
    return r;
}

__global__ void __launch_bounds__(NTHR, 2) gemm_xchg(const Params p) {
    extern __shared__ __attribute__((aligned(16))) uint8_t smem_raw[];
    LAS uint8_t* lds = (LAS uint8_t*)smem_raw;
    const int tid = threadIdx.x, lane = tid & 63, wv = __builtin_amdgcn_readfirstlane(tid >> 6);
    const int rank = p.rank;
    uint8_t* myws = p.ws[rank];
    const int b = blockIdx.x, xcd = b & 7, slot = b >> 3;
    constexpr int CS = NCOMM / 8;
    const bool comm = slot >= 32 - CS;
    const unsigned dstr = (unsigned)wv & (NR - 1);
    const unsigned peer = (unsigned)(wv + ROT * slot) & (NR - 1);
    STAMP(0);
    unsigned* cntp = (unsigned*)(myws + OFF_CNT + b * 128);
    const unsigned cnt_v = __hip_atomic_load(cntp, __ATOMIC_RELAXED, __HIP_MEMORY_SCOPE_AGENT);
    unsigned call;

    {
        const int l = lane & 31, kg = lane >> 5;
        const int q = slot;
        const unsigned dest = (unsigned)(rank + 1 + xcd + slot) & (NR - 1);
        const int j = q >> 1, sl = 4 * (q & 1) + (wv >> 1), h = wv & 1;
        const uint8_t* wsrc = p.w + (size_t)(sl * 64 + h * 32 + kg * 16) * GN + dest * NSL + j * 128 + 4 * l;
        unsigned d[16];
#pragma unroll
        for (int kk = 0; kk < 16; ++kk) d[kk] = *(const unsigned*)(wsrc + (size_t)kk * GN);
        const int xi = b >> 4, xk = b & 15;
        const int g = 4 * (xk & 1) + (wv >> 1);
        const u32x4 xv = *(const u32x4*)(p.x + (size_t)(g * MSH + xi * 32 + l) * KSH + (xk >> 1) * 64 + h * 32 + kg * 16);
        const bool direct = dest == (unsigned)rank || ((q & 1) == 0 && j < NPUSH0);
        uint8_t* dws = p.ws[dest];
        const __amdgpu_buffer_rsrc_t rw = make_rsrc(direct ? dws : myws, WS_TOTAL);
        const unsigned woff = (direct ? OFF_WS + (unsigned)(j * NKS + rank * 8 + sl) * 8192u : OFF_SW + (dest * 128 + j * 8 + sl) * 8192u) + h * 1024 + lane * 16;
        u32x4 o[4];
#pragma unroll
        for (int m = 0; m < 4; ++m) {
            const unsigned d0 = d[4 * m], d1 = d[4 * m + 1], d2 = d[4 * m + 2], d3 = d[4 * m + 3];
            const unsigned t0 = __builtin_amdgcn_perm(d1, d0, 0x05010400u), t1 = __builtin_amdgcn_perm(d1, d0, 0x07030602u);
            const unsigned t2 = __builtin_amdgcn_perm(d3, d2, 0x05010400u), t3 = __builtin_amdgcn_perm(d3, d2, 0x07030602u);
            o[0][m] = __builtin_amdgcn_perm(t2, t0, 0x05040100u);
            o[1][m] = __builtin_amdgcn_perm(t2, t0, 0x07060302u);
            o[2][m] = __builtin_amdgcn_perm(t3, t1, 0x05040100u);
            o[3][m] = __builtin_amdgcn_perm(t3, t1, 0x07060302u);
        }
        const bool paced = direct && dest != (unsigned)rank;
        const __amdgpu_buffer_rsrc_t rm = make_rsrc(myws, WS_TOTAL);
        call = __builtin_amdgcn_readfirstlane(cnt_v) + 1u;
        uint8_t* xflag = myws + OFF_FX + (xi * 8 + rank) * FXB + (xk * 2 + (lane & 1)) * 4;
        __builtin_amdgcn_raw_buffer_store_b128(o[0], rw, woff, 0, AUX_SYS);
        __builtin_amdgcn_raw_buffer_store_b128(xv, rm, OFF_XS + (unsigned)(xi * NKS + rank * 8) * 16384u + xk * 8192 + wv * 1024 + lane * 16, 0, AUX_SYS);
        __builtin_amdgcn_raw_buffer_store_b128(o[1], rw, woff + 2048, 0, AUX_SYS);
        if (paced) WAIT_VM(1);
        __builtin_amdgcn_raw_buffer_store_b128(o[2], rw, woff + 4096, 0, AUX_SYS);
        if (paced) {
            BAR();
            if (wv == 1 && lane < 2) flag_store(xflag, call);
            WAIT_VM(1);
        }
        __builtin_amdgcn_raw_buffer_store_b128(o[3], rw, woff + 6144, 0, AUX_SYS);
        WAIT_VM(0);
        BAR();
        if (wv == 0) {
            if (direct) { if (lane < 8) flag_store(dws + OFF_FW + (j * 8 + rank) * FWB + (8 * (q & 1) + lane) * 4, call); }
            else if (lane == 0) flag_store(dws + OFF_FS + rank * 128 + q * 4, call);
        } else if (wv == 1) {
            if (lane < 2 && !paced) flag_store(xflag, call);
            if (b == XMON) {
                const __amdgpu_buffer_rsrc_t rf = make_rsrc(myws, FLAG_BYTES);
                const unsigned long long t0 = __builtin_amdgcn_s_memrealtime();
                unsigned polls = 0;
                const unsigned o = OFF_FX + ((lane >> 3) * 8 + rank) * FXB + (lane & 7) * 16;
                for (;;) {
                    const u32x4 v = __builtin_amdgcn_raw_buffer_load_b128(rf, o, 0, AUX_SYS), w = __builtin_amdgcn_raw_buffer_load_b128(rf, o + 64 * FXB, 0, AUX_SYS);
                    if (__all(v.x >= call && v.y >= call && v.z >= call && v.w >= call && w.x >= call && w.y >= call && w.z >= call && w.w >= call)) {
                        if (lane < NR) flag_store(p.ws[lane] + OFF_FS + rank * 128, call);
                        break;
                    }
                    if (gave_up(t0, polls, call, myws + OFF_TMO)) break;
                    __builtin_amdgcn_s_sleep(8);
                    asm volatile("" ::: "memory");
                }
            }
        }
    }
    LAS unsigned* ldsflag = (LAS unsigned*)(lds + NS * SLOT);

    if (comm) {
        const int pk_ = xcd * (NCOMM / 8) + (slot - (32 - NCOMM / 8));
        volatile LAS unsigned* fs = (volatile LAS unsigned*)(lds + NR * PK_SLOT);
        if (tid < 256) fs[tid] = 0u;
        if (tid == 0) ldsflag[1] = 0u;
        WAIT_LGKM0();
        BAR();
        const unsigned long long t0 = __builtin_amdgcn_s_memrealtime();
        unsigned polls = 0;
        if (peer == (unsigned)rank) {
            const __amdgpu_buffer_rsrc_t rf = make_rsrc(myws, FLAG_BYTES);
            const int idx = (lane & 7) * 4;
            for (;;) {
                const u32x4 v = __builtin_amdgcn_raw_buffer_load_b128(rf, OFF_FS + lane * 16, 0, AUX_SYS);
                *(volatile LAS u32x4*)(lds + NR * PK_SLOT + lane * 16) = v;
                const bool done = (lane >> 3) == rank ||
                                  ((v.x >= call || (idx > 0 && idx < 2 * NPUSH0)) && v.y >= call &&
                                   (v.z >= call || idx + 2 < 2 * NPUSH0) && v.w >= call);
                if (__all(done)) break;
                if (gave_up(t0, polls, call, myws + OFF_TMO)) { if (lane == 0) *(volatile LAS unsigned*)(ldsflag + 1) = 1u; break; }
                __builtin_amdgcn_s_sleep(2);
            }
        } else {
            const __amdgpu_buffer_rsrc_t rs = make_rsrc(p.ws[peer], WS_TOTAL), rd = make_rsrc(myws, WS_TOTAL);
            unsigned dead = 0;
            auto wait_staged = [&](int idx) {
                while ((unsigned)__builtin_amdgcn_readfirstlane(fs[peer * 32 + idx]) < call) {
                    if (__builtin_amdgcn_readfirstlane(*(volatile LAS unsigned*)(ldsflag + 1)) != 0u) { dead = 0x40000000u; break; }
                    __builtin_amdgcn_s_sleep(1);
                }
                asm volatile("" ::: "memory");
            };
            constexpr unsigned OOB = 0x40000000u;
            static_assert(WS_TOTAL < OOB, "");
            const unsigned lo = lane * 16, one = lane == 0 ? 0u : OOB;
            const unsigned tkbase = (call - 1u) * NPC - NCOMM;
            Piece cur = piece_of(pk_, rank, peer);
            unsigned pfo = OOB;
            u32x4 v[PWIN];
            wait_staged(cur.staged);
#pragma unroll
            for (int u = 0; u < PWIN; ++u) {
                __builtin_amdgcn_raw_buffer_store_b128((u32x4){0u, 0u, 0u, 0u}, rd, OOB + lo + u * 1024, 0, AUX_SYS);
                v[u] = __builtin_amdgcn_raw_buffer_load_b128(rs, lo + u * 1024, cur.so, IN_AUX);
                asm volatile("" ::: "memory");
            }
            int tk = __builtin_amdgcn_raw_ptr_buffer_atomic_add_i32(1, rd, one + OFF_TK + peer * 256, 0, 0);
            static_assert(PWIN == 1 || PWIN == 2, "");
            bool granted = false;
            constexpr int UF = 0;
            for (;;) {
                Piece nxt = cur;
                bool more = false;
                unsigned lo2 = OOB + lo, tko = OOB;
#pragma unroll
                for (int u = 0; u < 4; ++u) {
                    __builtin_amdgcn_raw_buffer_store_b128(v[u % PWIN], rd, lo + u * 1024, cur.dso, AUX_SYS);
                    if (u < 4 - PWIN) v[u % PWIN] = __builtin_amdgcn_raw_buffer_load_b128(rs, lo + (u + PWIN) * 1024, cur.so, IN_AUX);
                    else v[u % PWIN] = __builtin_amdgcn_raw_buffer_load_b128(rs, lo2 + (u + PWIN - 4) * 1024, nxt.so, IN_AUX);
                    if (u == UF) {
                        WAIT_VM(2 * UF + 3 + 1);
                        __builtin_amdgcn_raw_buffer_store_b32(call, rd, one | pfo | dead, 0, AUX_SYS);
                    }
                    if (u == 3 - PWIN) {
                        const unsigned pc = (unsigned)__builtin_amdgcn_readfirstlane(tk) - tkbase;
                        more = pc < (unsigned)NPC;
                        if (more) { nxt = piece_of((int)pc, rank, peer); wait_staged(nxt.staged); lo2 = lo; tko = one; }
                        if (pc + GRANT_EARLY >= (unsigned)NPC && !granted) {
                            granted = true;
                            const int ng = CHAIN / NCOMM - 1 + (pk_ < CHAIN % NCOMM ? 1 : 0);
                            const int g = __builtin_amdgcn_raw_ptr_buffer_atomic_add_i32(ng, rd, one + OFF_TK + peer * 256 + 128, 0, 0);
                            grant(myws, peer, call, (unsigned)__builtin_amdgcn_readfirstlane(g), ng, lane);
                        }
                    }
                    if (u == 3) tk = __builtin_amdgcn_raw_ptr_buffer_atomic_add_i32(1, rd, tko + OFF_TK + peer * 256, 0, 0);
                    asm volatile("" ::: "memory");
                }
                pfo = cur.fo;
                cur = nxt;
                if (!more) break;
            }
            WAIT_VM(0);
            __builtin_amdgcn_raw_buffer_store_b32(call, rd, one | pfo | dead, 0, AUX_SYS);
        }
    }
    STAMP(1);
    const float qnan = __builtin_nanf("");
    const __amdgpu_buffer_rsrc_t ro = make_rsrc(p.out, OUT_BYTES);
    LAS uint8_t* pk = lds + wv * PK_SLOT;
    if (tid == 0) *ldsflag = 0u;
    int tslot = comm ? slot - (32 - CS) : slot;
    int ti = (xcd & 1) + 2 * (tslot >> 2), tj = 4 * (xcd >> 1) + (tslot & 3);
    for (int job = comm ? 2 : (slot < CS ? 0 : 1); job < 2; ++job) {
    f32x16 acc[4];
#pragma unroll
    for (int c = 0; c < 4; ++c)
#pragma unroll
        for (int r = 0; r < 16; ++r) acc[c][r] = 0.f;
    for (int ph = 0; ph < 3; ++ph) {
        if (wv == 0) {
            const bool ok = wait_operands(myws, ti, tj, lane, call, ph, rank);
            __builtin_amdgcn_fence(__ATOMIC_ACQUIRE, "");
            if (lane == 0 && !ok) *ldsflag = 1u;
            asm volatile("s_waitcnt vmcnt(0)" ::: "memory");
        }
        WAIT_LGKM0();
        BAR();
        STAMP(2);
        const int nstep = ph == 0 ? 8 : 28;
        const int sh = ph == 0 ? 3 : 2, sbase = ph == 0 ? rank : rank + 1, sadd = ph == 2 ? 4 : 0;
        const uint8_t* ga = myws + OFF_XS + (size_t)ti * NKS * 16384 + wv * 2048 + lane * 16;
        const uint8_t* gb = myws + OFF_WS + (size_t)tj * NKS * 8192 + wv * 1024 + lane * 16;
        auto dma = [&](int n, int sl) {
            const unsigned s = (unsigned)((((sbase + (n >> sh)) & 7) << 3) + (n & ((1 << sh) - 1)) + sadd);
            LAS uint8_t* d = lds + sl * SLOT;
            __builtin_amdgcn_global_load_lds((const void*)(ga + s * 16384u), (LAS void*)(d + wv * 2048), 16, 0, 0);
            __builtin_amdgcn_global_load_lds((const void*)(ga + s * 16384u + 1024), (LAS void*)(d + wv * 2048 + 1024), 16, 0, 0);
            __builtin_amdgcn_global_load_lds((const void*)(gb + s * 8192u), (LAS void*)(d + 16384 + wv * 1024), 16, 0, 0);
        };
        LAS const uint8_t* pa = lds + wv * 2048 + lane * 16;
        LAS const uint8_t* pb = lds + 16384 + lane * 16;
#pragma unroll
        for (int s = 0; s < NS; ++s) dma(s, s);
        WAIT_VM(3 * (NS - 1));
        BAR();
        Frags f0, f1;
        read_frags(f0, pa, pb);
        int sl = 0;
#define SGB(mask, n) __builtin_amdgcn_sched_group_barrier(mask, n, 0)
#define KSTEP(FC, FN, S, VMN, DO_DMA) do { \
            __builtin_amdgcn_sched_barrier(0); \
            WAIT_VM(VMN); WAIT_LGKM0(); BAR(); \
            __builtin_amdgcn_sched_barrier(0); \
            mma<0, 1>(acc, FC); \
            __builtin_amdgcn_sched_barrier(0); \
            const int nsl = sl + 1 == NS ? 0 : sl + 1; \
            read_frags(FN, pa + nsl * SLOT, pb + nsl * SLOT); \
            if (DO_DMA) dma((S) + NS, sl); \
            mma<1, 4>(acc, FC); \
            SGB(0x100, KS_R1); SGB(0x8, 1); SGB(0x100, 10 - KS_R1); SGB(0x10, 1); \
            SGB(0x8, 1); SGB(0x10, 2); \
            SGB(0x8, 1); \
            sl = nsl; } while (0)
        for (int s = 0; s < nstep - NS; s += 2) {
            KSTEP(f0, f1, s, 3 * (NS - 2), true);
            KSTEP(f1, f0, s + 1, 3 * (NS - 2), true);
        }
        static_assert(NS == 6, "tail below is written for NS == 6");
        KSTEP(f0, f1, 0, 12, false);
        KSTEP(f1, f0, 0, 9, false);
        KSTEP(f0, f1, 0, 6, false);
        KSTEP(f1, f0, 0, 3, false);
        KSTEP(f0, f1, 0, 0, false);
        mma<0, 4>(acc, f1);
#undef KSTEP
    }
    STAMP(3);
    WAIT_LGKM0();
    BAR();
    const float scale = *ldsflag != 0u ? qnan : p.sx[0] * p.sw[0];
    if (dstr == (unsigned)rank) {
        const unsigned off0 = (unsigned)(ti * 32 + 4 * (lane >> 5)) * (GN * 4) + (unsigned)(rank * NSL + tj * 128 + 4 * (lane & 31)) * 4;
#pragma unroll
        for (int r = 0; r < 16; ++r) {
            f32x4 v;
#pragma unroll
            for (int c = 0; c < 4; ++c) { const float y = acc[c][r] * scale; v[c] = y <= 0.f ? 0.f : y; }
            __builtin_amdgcn_raw_buffer_store_b128(__builtin_bit_cast(u32x4, v), ro, off0 + (unsigned)((r & 3) + 8 * (r >> 2)) * (GN * 4), 0, AUX_SYS);
        }
    } else {
        const int tile = ti * NCB + tj;
        unsigned mk[2] = {0u, 0u}, base = 512;
        const unsigned dump = PK_DATA_END + lane * 4;
#pragma unroll
        for (int r = 0; r < 16; ++r)
#pragma unroll
            for (int c = 0; c < 4; ++c) {
                const float y = acc[c][r] * scale;
                const bool nz = !(y <= 0.f);
                const unsigned long long m = __ballot(nz);
                const unsigned pos = __builtin_amdgcn_mbcnt_hi((unsigned)(m >> 32), __builtin_amdgcn_mbcnt_lo((unsigned)m, 0u)) * 4 + base;
                *(LAS float*)(pk + (nz ? pos : dump)) = y;
                base += (unsigned)__builtin_popcountll(m) * 4;
                mk[r >> 3] = mk[r >> 3] * 2 + (nz ? 1u : 0u);
            }
        *(LAS u32x2*)(pk + lane * 8) = (u32x2){mk[0], mk[1]};
        const unsigned bytes = base;
        const int nst = (int)((bytes + 1023u) >> 10);
        const __amdgpu_buffer_rsrc_t rp = make_rsrc(myws + OFF_PK + (unsigned)(dstr * NWG + tile) * PK_SLOT, PK_SLOT);
        for (int i = 0; i < nst; ++i) {
            const u32x4 v = *(LAS const u32x4*)(pk + i * 1024 + lane * 16);
            __builtin_amdgcn_raw_buffer_store_b128(v, rp, i * 1024 + lane * 16, 0, AUX_SYS);
        }
        WAIT_VM(0);
        if (lane == 0) __hip_atomic_store((unsigned long long*)(p.ws[dstr] + OFF_FD + tile * 128 + rank * 8), ((unsigned long long)call << 32) | bytes,
                                          __ATOMIC_RELAXED, __HIP_MEMORY_SCOPE_SYSTEM);
    }
    STAMP(4);
    if (job == 0) { tslot += 32 - CS; ti = (xcd & 1) + 2 * (tslot >> 2); tj = 4 * (xcd >> 1) + (tslot & 3); }
    }
    if (peer == (unsigned)rank) {
        if (lane == 0) __hip_atomic_store(cntp, call, __ATOMIC_RELAXED, __HIP_MEMORY_SCOPE_AGENT);
        STAMP(5); STAMP(6);
        STAMP_FLUSH();
        return;
    }
    {
        const int tile = ti * NCB + tj;
        const unsigned off0 = (unsigned)(ti * 32 + 4 * (lane >> 5)) * (GN * 4) + (unsigned)(peer * NSL + tj * 128 + 4 * (lane & 31)) * 4;
        bool ok = true;
        unsigned bytes = 0;
        {
            const unsigned long long t0 = __builtin_amdgcn_s_memrealtime();
            unsigned polls = 0;
            const unsigned long long* fp = (const unsigned long long*)(myws + OFF_FD + tile * 128 + (lane & 1) * 64 + peer * 8);
            for (;;) {
                unsigned long long f = (unsigned long long)call << 32;
                if (lane < (comm ? 1 : 2)) f = __hip_atomic_load(fp, __ATOMIC_RELAXED, __HIP_MEMORY_SCOPE_SYSTEM);
                bytes = __builtin_amdgcn_readfirstlane((unsigned)f);
                const unsigned long long miss = __ballot((unsigned)(f >> 32) < call);
                if (miss == 0) break;
                if (gave_up(t0, polls, call, myws + OFF_TMO)) { ok = false; break; }
                if (miss & 2) __builtin_amdgcn_s_sleep(GATE_SLEEP); else __builtin_amdgcn_s_sleep(RX_SLEEP);
            }
        }
        auto pass_on = [&]() {
            unsigned g = 0;
            if (lane == 0) g = __hip_atomic_fetch_add((unsigned*)(myws + OFF_TK + peer * 256 + 128), 1u, __ATOMIC_RELAXED, __HIP_MEMORY_SCOPE_AGENT);
            grant(myws, peer, call, (unsigned)__builtin_amdgcn_readfirstlane(g), 1, lane);
        };
        asm volatile("" ::: "memory");
        STAMP(5);
        if (!ok) {
            pass_on();
            for (int r = 0; r < 16; ++r)
                __builtin_amdgcn_raw_buffer_store_b128(__builtin_bit_cast(u32x4, (f32x4){qnan, qnan, qnan, qnan}), ro, off0 + (unsigned)((r & 3) + 8 * (r >> 2)) * (GN * 4), 0, AUX_SYS);
            return;
        }
        if (bytes > PK_DATA_END) bytes = PK_DATA_END;
        const int nld = (int)((bytes + 1023u) >> 10);
        const uint8_t* src = p.ws[peer] + OFF_PK + (unsigned)(rank * NWG + tile) * PK_SLOT + lane * 16;
        int win = 1;
        bool passed = false;
        const __amdgpu_buffer_rsrc_t rfl = make_rsrc(myws, FLAG_BYTES);
        unsigned ta = 0;
        int g = 0, i = 0;
        bool pass = false;
        auto issue = [&]() {
            const int n = win < nld - i ? win : nld - i;
            pass = i + n + CHAIN_EARLY >= nld && !passed;
            ta = (unsigned)__builtin_amdgcn_s_memrealtime();
            if (pass) g = __builtin_amdgcn_raw_ptr_buffer_atomic_add_i32(1, rfl, (lane == 0 ? 0u : 0x40000000u) + OFF_TK + peer * 256 + 128, 0, 0);
            for (int u = 0; u < n; ++u, ++i)
                if ((unsigned)(i * 1024 + (lane & ~7) * 16) < bytes)
                    __builtin_amdgcn_global_load_lds((const void*)(src + i * 1024), (LAS void*)(pk + i * 1024), 16, 0, AUX_SYS);
        };
        auto land = [&]() {
            WAIT_VM(0);
            asm volatile("" : "+v"(g));
            const unsigned lat = (unsigned)__builtin_amdgcn_s_memrealtime() - ta;
            if (pass) { passed = true; grant(myws, peer, call, (unsigned)__builtin_amdgcn_readfirstlane(g), 1, lane); }
            if (lat < LAT_LO) win = win < WMAX ? win * 2 : win;
            else if (lat > LAT_HI) { if (win > 1) win >>= 1; else if (lat > LAT_BACK) __builtin_amdgcn_s_sleep(BACKOFF); }
        };
        *(LAS unsigned*)(pk + PK_DATA_END + 256 + lane * 4) = 0u;
        issue(); land();
        const u32x2 mm = *(LAS const u32x2*)(pk + lane * 8);
        WAIT_LGKM0();
        unsigned ad[64];
        unsigned base = 512;
        const unsigned zero = PK_DATA_END + 256 + lane * 4;
#pragma unroll
        for (int q = 0; q < ADSL; ++q) {
            const bool fly = i < nld;
            if (fly) issue();
#pragma unroll
            for (int k = q * (64 / ADSL); k < (q + 1) * (64 / ADSL); ++k) {
                const bool nz = (int)((k < 32 ? mm.x : mm.y) << (k & 31)) < 0;
                const unsigned long long m = __ballot(nz);
                const unsigned pos = __builtin_amdgcn_mbcnt_hi((unsigned)(m >> 32), __builtin_amdgcn_mbcnt_lo((unsigned)m, 0u)) * 4 + base;
                base += (unsigned)__builtin_popcountll(m) * 4;
                ad[k] = (unsigned)(size_t)pk + (nz ? pos : zero);
                asm volatile("" : "+v"(ad[k]));
            }
            if (fly) land();
        }
        while (i < nld) { issue(); land(); }
        __builtin_amdgcn_s_waitcnt(0x0F70);
        STAMP(6);
#pragma unroll
        for (int r = 0; r < 16; ++r) {
            f32x4 v;
#pragma unroll
            for (int c = 0; c < 4; ++c) v[c] = *(LAS const float*)(size_t)ad[r * 4 + c];
            __builtin_amdgcn_raw_buffer_store_b128(__builtin_bit_cast(u32x4, v), ro, off0 + (unsigned)((r & 3) + 8 * (r >> 2)) * (GN * 4), 0, AUX_SYS);
        }
        STAMP_FLUSH();
    }
}

}

extern "C" void kernel_launch(int rank, int nranks, ncclComm_t comm, hipStream_t stream,
                              void* const* d_in, const int* in_sizes, int n_in,
                              void* d_out, int out_size, const int* in_dtypes, int out_dtype,
                              void* const* d_out_all, void* const* d_ws_all, size_t ws_bytes) {
    (void)comm; (void)in_dtypes; (void)out_dtype; (void)d_out_all;
    if (nranks != NR || n_in != 4 || in_sizes[0] != GM * KSH || in_sizes[1] != KSH * GN || out_size != MSH * GN ||
        ws_bytes < WS_TOTAL || (unsigned)rank >= (unsigned)NR) {
        fprintf(stderr, "kernel_launch: unexpected shapes\n");
        return;
    }
    Params p;
    p.x = (const uint8_t*)d_in[0]; p.w = (const uint8_t*)d_in[1];
    p.sx = (const float*)d_in[2]; p.sw = (const float*)d_in[3];
    p.out = (uint8_t*)d_out;
    for (int i = 0; i < NR; ++i) p.ws[i] = (uint8_t*)d_ws_all[i];
    p.rank = rank;
    HIP_CHECK(hipFuncSetAttribute((const void*)gemm_xchg, hipFuncAttributeMaxDynamicSharedMemorySize, LDS_BYTES));
    hipLaunchKernelGGL(gemm_xchg, dim3(NWG), dim3(NTHR), LDS_BYTES, stream, p);
    HIP_CHECK(hipGetLastError());
}
```

```cpp
#include <hip/hip_runtime.h>
#include <rccl/rccl.h>
#include <cstdio>
#include <cstdint>

#define HIP_CHECK(expr) do { hipError_t _e = (expr); if (_e != hipSuccess) { \
    fprintf(stderr, "HIP error %s at %s:%d\n", hipGetErrorString(_e), __FILE__, __LINE__); \
    return; } } while (0)

namespace {

typedef float f32x4 __attribute__((ext_vector_type(4)));
typedef float f32x16 __attribute__((ext_vector_type(16)));
typedef int i32x4 __attribute__((ext_vector_type(4)));
typedef int i32x8 __attribute__((ext_vector_type(8)));
typedef unsigned u32x2 __attribute__((ext_vector_type(2)));
typedef unsigned u32x4 __attribute__((ext_vector_type(4)));
#define LAS __attribute__((address_space(3)))

constexpr int NR = 8;
constexpr int GM = 4096, GK = 4096, GN = 16384;
constexpr int KSH = GK / NR;
constexpr int MSH = GM / NR;
constexpr int NSL = GN / NR;
constexpr int NRB = 16, NCB = 16;
constexpr int NKS = GK / 64;
constexpr int NWG = NRB * NCB;
constexpr int NTHR = 512;

#define NCOMM 32
#define PWIN 2
#define LEAD 4
#define IN_AUX 17
constexpr int PB = 4096;
constexpr unsigned FXB = 128, FWB = 64;

constexpr unsigned OFF_FX = 0;
constexpr unsigned OFF_FW = 32768;
constexpr unsigned OFF_FS = 16384;
constexpr unsigned OFF_WC = 20480;
constexpr unsigned OFF_TMO = 50176;
constexpr unsigned OFF_TK = 51200;
constexpr unsigned FLAG_BYTES = 65536;
constexpr unsigned OFF_FD = 65536;
constexpr unsigned OFF_CNT = OFF_FD + NWG * 128;
constexpr unsigned OFF_DBG = OFF_CNT + NWG * 128;
constexpr unsigned OFF_XS = 1u << 20;
constexpr unsigned XS_BYTES = (unsigned)GM * GK;
constexpr unsigned OFF_WS = OFF_XS + XS_BYTES;
constexpr unsigned WS_BYTES = (unsigned)GK * NSL;
constexpr unsigned OFF_SW = OFF_WS + WS_BYTES;
constexpr unsigned SW_BYTES = (unsigned)KSH * GN;
constexpr unsigned PK_SLOT = 17408;
constexpr unsigned PK_DATA_END = 512 + 16384;
constexpr unsigned OFF_PK = OFF_SW + SW_BYTES;
constexpr unsigned WS_TOTAL = OFF_PK + NR * NWG * PK_SLOT;
constexpr unsigned OUT_BYTES = (unsigned)MSH * GN * 4;

constexpr int NS = 6;
constexpr int SLOT = 24576;
constexpr int LDS_BYTES = NS * SLOT + 64;
constexpr unsigned long long DEADLINE_TICKS = 50000000ull;
constexpr int AUX_SYS = 17;

#define NPUSH0 8
#define POLL_SLEEP 8
#define POLL_SLOW 32
#define RX_SLEEP 8
#define KS_R1 6
#define XMON 100
#define ADSL 2
#define ROT 3
#define RGRAN 8
#define STAGE_DELAY 16
#define STAGE_DELAY2 16
#define W2A 4
#define PH0_DELAY 100
#define MON_SLEEP 4
#define ALLW 1
#define ENDGAME 1
#define UNIT_TICKS 13
#define EG2 44
#define EG3 26
#define EG4 18
#define W1ONLY 0
#define ACQ 2
#define CHAIN 64
#define CHAIN_EARLY 2
#define GRANT_EARLY 16
#define PUSH_WIN 1
#define GATE_SLEEP 16
#define LAT_LO 85
#define LAT_HI 110
#define LAT_BACK 170
#define BACKOFF 16
#define WMAX 8

struct Params {
    const uint8_t* x;
    const uint8_t* w;
    uint8_t* myws;
    int rank, pad;
    const float* sx;
    const float* sw;
    uint8_t* out;
    uint8_t* ws[NR];
};

#define WAIT_VM(n) asm volatile("s_waitcnt vmcnt(%0)" :: "n"(n) : "memory")
#define WAIT_LGKM0() asm volatile("s_waitcnt lgkmcnt(0)" ::: "memory")
#define BAR() do { asm volatile("" ::: "memory"); __builtin_amdgcn_s_barrier(); asm volatile("" ::: "memory"); } while (0)

#define STAMP(k) do {} while (0)
#define STAMP_FLUSH() do {} while (0)

__device__ __forceinline__ __amdgpu_buffer_rsrc_t make_rsrc(const void* p, unsigned bytes) {
    return __builtin_amdgcn_make_buffer_rsrc((void*)p, 0, (int)bytes, 0x00020000);
}
__device__ __forceinline__ void flag_store(uint8_t* p, unsigned v) {
    __hip_atomic_store((unsigned*)p, v, __ATOMIC_RELAXED, __HIP_MEMORY_SCOPE_SYSTEM);
}
__device__ __forceinline__ bool gave_up(unsigned long long t0, unsigned& polls, unsigned call, uint8_t* tmo) {
    if ((++polls & 63u) != 0u) return false;
    const unsigned t = __builtin_amdgcn_readfirstlane(__hip_atomic_load((const unsigned*)tmo, __ATOMIC_RELAXED, __HIP_MEMORY_SCOPE_AGENT));
    if (t == call || __builtin_amdgcn_s_memrealtime() - t0 > DEADLINE_TICKS) {
        __hip_atomic_store((unsigned*)tmo, call, __ATOMIC_RELAXED, __HIP_MEMORY_SCOPE_AGENT);
        return true;
    }
    return false;
}
__device__ __forceinline__ bool wait_operands(uint8_t* ws, int ti, int tj, int lane, unsigned call, int ph, int rank) {
    const unsigned long long t0 = __builtin_amdgcn_s_memrealtime();
    const __amdgpu_buffer_rsrc_t rf = make_rsrc(ws, FLAG_BYTES);
    unsigned polls = 0;
    if (ph == 0) __builtin_amdgcn_s_sleep(PH0_DELAY);
    else {
        for (;;) {
            unsigned v = call;
            if (lane < 8) v = __builtin_amdgcn_raw_buffer_load_b32(rf, OFF_FX + (ti * 8 + lane) * FXB + (ph == 1 ? 0 : 64) + tj * 4, 0, AUX_SYS);
            if (__builtin_popcountll(__ballot(v >= call)) >= 56 + 3) break;
            if (gave_up(t0, polls, call, ws + OFF_TMO)) return false;
            __builtin_amdgcn_s_sleep(POLL_SLOW);
            asm volatile("" ::: "memory");
        }
    }
    const bool xneed = ph == 0 ? (lane >> 3) == rank : (ph == 3 || ((lane >> 2) & 1) == ph - 1);
    const bool wneed = lane < 32 && (ph == 0 ? (lane >> 2) == rank : (ph == 3 || ((lane >> 1) & 1) == ph - 1));
    bool wok = false;
    for (;;) {
        u32x4 v = {call, call, call, call}, w = v;
        if (xneed) v = __builtin_amdgcn_raw_buffer_load_b128(rf, OFF_FX + ti * 8 * FXB + lane * 16, 0, AUX_SYS);
        if (wneed && !wok) w = __builtin_amdgcn_raw_buffer_load_b128(rf, OFF_FW + tj * 8 * FWB + lane * 16, 0, AUX_SYS);
        wok = __all(w.x >= call && w.y >= call && w.z >= call && w.w >= call);
        if (wok && __all(v.x >= call && v.y >= call && v.z >= call && v.w >= call)) return true;
        if (gave_up(t0, polls, call, ws + OFF_TMO)) return false;
        __builtin_amdgcn_s_sleep(POLL_SLEEP);
        asm volatile("" ::: "memory");
    }
}

static_assert(NCOMM == 32 && CHAIN >= NCOMM, "");
__device__ __forceinline__ void grant(uint8_t* ws, unsigned peer, unsigned call, unsigned first, int n, int lane) {
    const unsigned t = first - (call - 1u) * (CHAIN - NCOMM + NWG) + (unsigned)lane + NCOMM;
    if (lane < n && t < (unsigned)NWG) flag_store(ws + OFF_FD + t * 128 + 64 + peer * 8 + 4, call);
}

struct Frags { i32x4 a[2]; i32x4 b[4][2]; };
__device__ __forceinline__ void read_frags(Frags& f, LAS const uint8_t* pa, LAS const uint8_t* pb) {
    f.a[0] = *(LAS const i32x4*)(pa); f.a[1] = *(LAS const i32x4*)(pa + 1024);
#pragma unroll
    for (int c = 0; c < 4; ++c) { f.b[c][0] = *(LAS const i32x4*)(pb + c * 2048); f.b[c][1] = *(LAS const i32x4*)(pb + c * 2048 + 1024); }
}
__device__ __forceinline__ i32x8 cat(i32x4 a, i32x4 b) { return __builtin_shufflevector(a, b, 0, 1, 2, 3, 4, 5, 6, 7); }
template <int C0, int C1> __device__ __forceinline__ void mma(f32x16 (&acc)[4], const Frags& f) {
    const i32x8 a = cat(f.a[0], f.a[1]);
#pragma unroll
    for (int c = C0; c < C1; ++c)
        acc[c] = __builtin_amdgcn_mfma_scale_f32_32x32x64_f8f6f4(a, cat(f.b[c][0], f.b[c][1]), acc[c], 0, 0, 0, 127, 0, 127);
}

constexpr int NW1 = 8 - NPUSH0 / 2;
constexpr int NPC = (NW1 + 8 + 2 * NRB) * 16;
static_assert(NPUSH0 >= 2 && NPUSH0 % 2 == 0 && NPUSH0 <= 16 && NCOMM % 8 == 0 && 4 % PWIN == 0, "FS index 0 is the x flag");
struct Piece { unsigned so, dso, fo; int staged; };
__device__ __forceinline__ Piece piece_of(int pc, int rank, unsigned peer) {
    int V = pc >> 4;
    const int k = pc & 15;
    int isw, half, idx;
    if (V < NW1) { isw = 1; half = 0; idx = NPUSH0 / 2 + V; }
    else if ((V -= NW1) < W2A) { isw = 1; half = 1; idx = V; }
    else if ((V -= W2A) < LEAD) { isw = 0; half = 0; idx = V; }
    else if ((V -= LEAD) < 8 - W2A) { isw = 1; half = 1; idx = W2A + V; }
    else if ((V -= 8 - W2A) < 2 * (NRB - LEAD)) { isw = 0; half = (V & 1) ^ 1; idx = (V >> 1) + ((V & 1) ? LEAD : 0); }
    else { isw = 0; half = 1; idx = V - 2 * (NRB - LEAD) + NRB - LEAD; }
    Piece r;
    if (isw) {
        const unsigned j = 2 * idx + (k >> 3), o = half * 32768 + (k & 7) * PB;
        r.so = OFF_SW + (unsigned)rank * 1048576u + j * 65536u + o;
        r.dso = OFF_WS + (j * NKS + peer * 8) * 8192u + o;
        r.fo = OFF_FW + (j * 8 + peer) * FWB + (half * 8 + (k & 7)) * 4;
        r.staged = W1ONLY && half ? 0 : 1;
    } else {
        r.so = r.dso = OFF_XS + ((unsigned)idx * NKS + peer * 8) * 16384u + (half * 16 + k) * PB;
        r.fo = OFF_FX + ((unsigned)idx * 8 + peer) * FXB + (half * 16 + k) * 4;
        r.staged = 0;
    }
    return r;
}

__global__ void __launch_bounds__(NTHR, 2) gemm_xchg(const Params p) {
    extern __shared__ __attribute__((aligned(16))) uint8_t smem_raw[];
    LAS uint8_t* lds = (LAS uint8_t*)smem_raw;
    const int tid = threadIdx.x, lane = tid & 63, wv = __builtin_amdgcn_readfirstlane(tid >> 6);
    const int rank = p.rank;
    uint8_t* myws = p.myws;
    const int b = blockIdx.x, xcd = b & 7, slot = b >> 3;
    constexpr int CS = NCOMM / 8;
    const bool comm = slot >= 32 - CS;
    const unsigned dstr = (unsigned)wv & (NR - 1);
    const unsigned peer = (unsigned)(wv + ROT * slot) & (NR - 1);
    STAMP(0);
    unsigned* cntp = (unsigned*)(myws + OFF_CNT + b * 128);
    const unsigned cnt_v = __hip_atomic_load(cntp, __ATOMIC_RELAXED, __HIP_MEMORY_SCOPE_AGENT);
    unsigned call;
    LAS unsigned* ldsflag = (LAS unsigned*)(lds + NS * SLOT);
    if (tid < 2 * NR) *(LAS unsigned*)(lds + NR * PK_SLOT + tid * 4) = 0u;
    if (tid < 2) ldsflag[tid] = 0u;

    {
        const int l = lane & 31, kg = lane >> 5;
        const int s16 = slot & 15;
        const int j = slot < 16 || s16 < NPUSH0 ? s16 : (s16 >= 16 - CS ? s16 - (16 - CS) + NPUSH0 : s16 + CS);
        const int q = (j << 1) | (slot < 16 ? 1 : 0);
        const unsigned dest = (unsigned)(rank + 1 + xcd + slot) & (NR - 1);
        const int sl = 4 * (q & 1) + (wv >> 1), h = wv & 1;
        uint8_t* dws = p.ws[dest];
        if (slot < 16) __builtin_amdgcn_s_sleep(STAGE_DELAY);
        else if (j >= NPUSH0) __builtin_amdgcn_s_sleep(STAGE_DELAY2);
        const uint8_t* wsrc = p.w + (size_t)(sl * 64 + h * 32 + (lane >> 3)) * GN + dest * NSL + j * 128 + (lane & 7) * 16;
#pragma unroll
        for (int u = 0; u < 4; ++u)
            __builtin_amdgcn_global_load_lds((const void*)(wsrc + (size_t)u * 8 * GN), (LAS void*)(lds + wv * 4096 + u * 1024), 16, 0, 0);
        const int xi = b >> 4, xk = b & 15;
        const int g = 4 * (xk & 1) + (wv >> 1);
        const u32x4 xv = *(const u32x4*)(p.x + (size_t)(g * MSH + xi * 32 + l) * KSH + (xk >> 1) * 64 + h * 32 + kg * 16);
        WAIT_VM(1);
        unsigned d[16];
#pragma unroll
        for (int kk = 0; kk < 16; ++kk) d[kk] = *(LAS const unsigned*)(lds + wv * 4096 + (kg * 16 + kk) * 128 + l * 4);
        const bool direct = dest == (unsigned)rank || ((q & 1) == 0 && j < NPUSH0);
        const __amdgpu_buffer_rsrc_t rw = make_rsrc(direct ? dws : myws, WS_TOTAL);
        const unsigned woff = (direct ? OFF_WS + (unsigned)(j * NKS + rank * 8 + sl) * 8192u : OFF_SW + (dest * 128 + j * 8 + sl) * 8192u) + h * 1024 + lane * 16;
        u32x4 o[4];
#pragma unroll
        for (int m = 0; m < 4; ++m) {
            const unsigned d0 = d[4 * m], d1 = d[4 * m + 1], d2 = d[4 * m + 2], d3 = d[4 * m + 3];
            const unsigned t0 = __builtin_amdgcn_perm(d1, d0, 0x05010400u), t1 = __builtin_amdgcn_perm(d1, d0, 0x07030602u);
            const unsigned t2 = __builtin_amdgcn_perm(d3, d2, 0x05010400u), t3 = __builtin_amdgcn_perm(d3, d2, 0x07030602u);
            o[0][m] = __builtin_amdgcn_perm(t2, t0, 0x05040100u);
            o[1][m] = __builtin_amdgcn_perm(t2, t0, 0x07060302u);
            o[2][m] = __builtin_amdgcn_perm(t3, t1, 0x05040100u);
            o[3][m] = __builtin_amdgcn_perm(t3, t1, 0x07060302u);
        }
        const bool paced = direct && dest != (unsigned)rank;
        const __amdgpu_buffer_rsrc_t rm = make_rsrc(myws, WS_TOTAL);
        call = __builtin_amdgcn_readfirstlane(cnt_v) + 1u;
        uint8_t* xflag = myws + OFF_FX + (xi * 8 + rank) * FXB + (xk * 2 + (lane & 1)) * 4;
        __builtin_amdgcn_raw_buffer_store_b128(o[0], rw, woff, 0, AUX_SYS);
        __builtin_amdgcn_raw_buffer_store_b128(xv, rm, OFF_XS + (unsigned)(xi * NKS + rank * 8) * 16384u + xk * 8192 + wv * 1024 + lane * 16, 0, AUX_SYS);
        if (paced && PUSH_WIN == 1) WAIT_VM(0);
        __builtin_amdgcn_raw_buffer_store_b128(o[1], rw, woff + 2048, 0, AUX_SYS);
        if (paced) WAIT_VM(PUSH_WIN - 1);
        __builtin_amdgcn_raw_buffer_store_b128(o[2], rw, woff + 4096, 0, AUX_SYS);
        if (paced) {
            BAR();
            if (wv == 1 && lane < 2) flag_store(xflag, call);
            WAIT_VM(PUSH_WIN - 1);
        }
        __builtin_amdgcn_raw_buffer_store_b128(o[3], rw, woff + 6144, 0, AUX_SYS);
        WAIT_VM(0);
        WAIT_LGKM0();
        BAR();
        if (peer == (unsigned)rank) {
            if (lane < 2 && !paced) flag_store(xflag, call);
            if (direct) { if (lane < 8) flag_store(dws + OFF_FW + (j * 8 + rank) * FWB + (8 * (q & 1) + lane) * 4, call); }
            else if (!W1ONLY || (q & 1) == 0) {
                unsigned t = 0;
                if (lane == 0) t = __hip_atomic_fetch_add((unsigned*)(myws + OFF_WC + dest * 128), 1u, __ATOMIC_RELAXED, __HIP_MEMORY_SCOPE_AGENT);
                if (lane == 0 && t == call * ((W1ONLY ? 16 : 32) - NPUSH0) - 1u) flag_store(dws + OFF_FS + rank * 256 + 128, call);
            }
            if (b == XMON) {
                const __amdgpu_buffer_rsrc_t rf = make_rsrc(myws, FLAG_BYTES);
                const unsigned long long t0 = __builtin_amdgcn_s_memrealtime();
                unsigned polls = 0;
                const unsigned o = OFF_FX + ((lane >> 3) * 8 + rank) * FXB + (lane & 7) * 16;
                for (;;) {
                    const u32x4 v = __builtin_amdgcn_raw_buffer_load_b128(rf, o, 0, AUX_SYS), w = __builtin_amdgcn_raw_buffer_load_b128(rf, o + 64 * FXB, 0, AUX_SYS);
                    if (__all(v.x >= call && v.y >= call && v.z >= call && v.w >= call && w.x >= call && w.y >= call && w.z >= call && w.w >= call)) {
                        if (lane < NR) flag_store(p.ws[lane] + OFF_FS + rank * 256, call);
                        break;
                    }
                    if (gave_up(t0, polls, call, myws + OFF_TMO)) break;
                    __builtin_amdgcn_s_sleep(16);
                    asm volatile("" ::: "memory");
                }
            }
        }
    }

    if (comm) {
        const int pk_ = xcd * (NCOMM / 8) + (slot - (32 - NCOMM / 8));
        volatile LAS unsigned* fs = (volatile LAS unsigned*)(lds + NR * PK_SLOT);
        const unsigned long long t0 = __builtin_amdgcn_s_memrealtime();
        unsigned polls = 0;
        if (peer == (unsigned)rank) {
            for (;;) {
                unsigned v = call;
                if (lane < 2 * NR && (lane >> 1) != rank) {
                    v = __hip_atomic_load((const unsigned*)(myws + OFF_FS + lane * 128), __ATOMIC_RELAXED, __HIP_MEMORY_SCOPE_SYSTEM);
                }
                if ((__ballot(v < call) & 0xAAAAull) == 0 && lane < 2 * NR) fs[lane] = v;
                if (__all(v >= call)) break;
                if (gave_up(t0, polls, call, myws + OFF_TMO)) { if (lane == 0) *(volatile LAS unsigned*)(ldsflag + 1) = 1u; break; }
                __builtin_amdgcn_s_sleep(MON_SLEEP);
            }
        } else {
            const __amdgpu_buffer_rsrc_t rs = make_rsrc(p.ws[peer], WS_TOTAL), rd = make_rsrc(myws, WS_TOTAL);
            unsigned dead = 0;
            auto wait_staged = [&](int idx) {
                while ((unsigned)__builtin_amdgcn_readfirstlane(fs[peer * 2 + idx]) < call) {
                    if (__builtin_amdgcn_readfirstlane(*(volatile LAS unsigned*)(ldsflag + 1)) != 0u) { dead = 0x40000000u; break; }
                    __builtin_amdgcn_s_sleep(1);
                }
                asm volatile("" ::: "memory");
            };
            constexpr unsigned OOB = 0x40000000u;
            static_assert(WS_TOTAL < OOB, "");
            const unsigned lo = lane * 16, one = lane == 0 ? 0u : OOB;
            const unsigned tkbase = (call - 1u) * NPC - NCOMM;
            Piece cur = piece_of(pk_, rank, peer);
            unsigned pfo = OOB;
            u32x4 v[PWIN];
            wait_staged(cur.staged);
#pragma unroll
            for (int u = 0; u < PWIN; ++u) {
                __builtin_amdgcn_raw_buffer_store_b128((u32x4){0u, 0u, 0u, 0u}, rd, OOB + lo + u * 1024, 0, AUX_SYS);
                v[u] = __builtin_amdgcn_raw_buffer_load_b128(rs, lo + u * 1024, cur.so, IN_AUX);
                asm volatile("" ::: "memory");
            }
            int tk = __builtin_amdgcn_raw_ptr_buffer_atomic_add_i32(1, rd, one + OFF_TK + peer * 256, 0, 0);
            static_assert(PWIN == 1 || PWIN == 2, "");
            bool granted = false;
            constexpr int UF = 0;
            for (;;) {
                Piece nxt = cur;
                bool more = false;
                unsigned lo2 = OOB + lo, tko = OOB;
#pragma unroll
                for (int u = 0; u < 4; ++u) {
                    __builtin_amdgcn_raw_buffer_store_b128(v[u % PWIN], rd, lo + u * 1024, cur.dso, AUX_SYS);
                    if (u < 4 - PWIN) v[u % PWIN] = __builtin_amdgcn_raw_buffer_load_b128(rs, lo + (u + PWIN) * 1024, cur.so, IN_AUX);
                    else v[u % PWIN] = __builtin_amdgcn_raw_buffer_load_b128(rs, lo2 + (u + PWIN - 4) * 1024, nxt.so, IN_AUX);
                    if (u == UF) {
                        WAIT_VM(2 * UF + 3 + 1);
                        __builtin_amdgcn_raw_buffer_store_b32(call, rd, one | pfo | dead, 0, AUX_SYS);
                    }
                    if (u == 3 - PWIN) {
                        const unsigned pc = (unsigned)__builtin_amdgcn_readfirstlane(tk) - tkbase;
                        more = pc < (unsigned)NPC;
                        if (more) { nxt = piece_of((int)pc, rank, peer); wait_staged(nxt.staged); lo2 = lo; tko = one; }
                        if (pc + GRANT_EARLY >= (unsigned)NPC && !granted) {
                            granted = true;
                            const int ng = CHAIN / NCOMM - 1 + (pk_ < CHAIN % NCOMM ? 1 : 0);
                            const int g = __builtin_amdgcn_raw_ptr_buffer_atomic_add_i32(ng, rd, one + OFF_TK + peer * 256 + 128, 0, 0);
                            grant(myws, peer, call, (unsigned)__builtin_amdgcn_readfirstlane(g), ng, lane);
                        }
                    }
                    if (u == 3) tk = __builtin_amdgcn_raw_ptr_buffer_atomic_add_i32(1, rd, tko + OFF_TK + peer * 256, 0, 0);
                    asm volatile("" ::: "memory");
                }
                pfo = cur.fo;
                cur = nxt;
                if (!more) break;
            }
            WAIT_VM(0);
            __builtin_amdgcn_raw_buffer_store_b32(call, rd, one | pfo | dead, 0, AUX_SYS);
        }
    }
    STAMP(1);
    const float qnan = __builtin_nanf("");
    const __amdgpu_buffer_rsrc_t ro = make_rsrc(p.out, OUT_BYTES);
    LAS uint8_t* pk = lds + wv * PK_SLOT;
    int tslot = comm ? slot - (32 - CS) : slot;
    int ti = (xcd & 1) + 2 * (tslot >> 2), tj = 4 * (xcd >> 1) + (tslot & 3);
    for (int job = comm ? 2 : (slot < CS ? 0 : 1); job < 2; ++job) {
    f32x16 acc[4];
#pragma unroll
    for (int c = 0; c < 4; ++c)
#pragma unroll
        for (int r = 0; r < 16; ++r) acc[c][r] = 0.f;
    for (int ph = 0; ph < 3; ++ph) {
        if (wv == 0) {
            const bool ok = wait_operands(myws, ti, tj, lane, call, ph, rank);
            __builtin_amdgcn_fence(__ATOMIC_ACQUIRE, "");
            if (lane == 0 && !ok) *ldsflag = 1u;
            asm volatile("s_waitcnt vmcnt(0)" ::: "memory");
        }
        WAIT_LGKM0();
        BAR();
        STAMP(2);
        const int nstep = ph == 0 ? 8 : 28;
        const int sh = ph == 0 ? 3 : 2, sbase = ph == 0 ? rank : rank + 1, sadd = ph == 2 ? 4 : 0;
        const uint8_t* ga = myws + OFF_XS + (size_t)ti * NKS * 16384 + wv * 2048 + lane * 16;
        const uint8_t* gb = myws + OFF_WS + (size_t)tj * NKS * 8192 + wv * 1024 + lane * 16;
        auto dma = [&](int n, int sl) {
            const unsigned s = (unsigned)((((sbase + (n >> sh)) & 7) << 3) + (n & ((1 << sh) - 1)) + sadd);
            LAS uint8_t* d = lds + sl * SLOT;
            __builtin_amdgcn_global_load_lds((const void*)(ga + s * 16384u), (LAS void*)(d + wv * 2048), 16, 0, 0);
            __builtin_amdgcn_global_load_lds((const void*)(ga + s * 16384u + 1024), (LAS void*)(d + wv * 2048 + 1024), 16, 0, 0);
            __builtin_amdgcn_global_load_lds((const void*)(gb + s * 8192u), (LAS void*)(d + 16384 + wv * 1024), 16, 0, 0);
        };
        LAS const uint8_t* pa = lds + wv * 2048 + lane * 16;
        LAS const uint8_t* pb = lds + 16384 + lane * 16;
#pragma unroll
        for (int s = 0; s < NS; ++s) dma(s, s);
        WAIT_VM(3 * (NS - 1));
        BAR();
        Frags f0, f1;
        read_frags(f0, pa, pb);
        int sl = 0;
#define SGB(mask, n) __builtin_amdgcn_sched_group_barrier(mask, n, 0)
#define KSTEP(FC, FN, S, VMN, DO_DMA) do { \
            __builtin_amdgcn_sched_barrier(0); \
            WAIT_VM(VMN); WAIT_LGKM0(); BAR(); \
            __builtin_amdgcn_sched_barrier(0); \
            mma<0, 1>(acc, FC); \
            __builtin_amdgcn_sched_barrier(0); \
            const int nsl = sl + 1 == NS ? 0 : sl + 1; \
            read_frags(FN, pa + nsl * SLOT, pb + nsl * SLOT); \
            if (DO_DMA) dma((S) + NS, sl); \
            mma<1, 4>(acc, FC); \
            SGB(0x100, KS_R1); SGB(0x8, 1); SGB(0x100, 10 - KS_R1); SGB(0x10, 1); \
            SGB(0x8, 1); SGB(0x10, 2); \
            SGB(0x8, 1); \
            sl = nsl; } while (0)
        for (int s = 0; s < nstep - NS; s += 2) {
            KSTEP(f0, f1, s, 3 * (NS - 2), true);
            KSTEP(f1, f0, s + 1, 3 * (NS - 2), true);
        }
        static_assert(NS == 6, "tail below is written for NS == 6");
        KSTEP(f0, f1, 0, 12, false);
        KSTEP(f1, f0, 0, 9, false);
        KSTEP(f0, f1, 0, 6, false);
        KSTEP(f1, f0, 0, 3, false);
        KSTEP(f0, f1, 0, 0, false);
        mma<0, 4>(acc, f1);
#undef KSTEP
    }
    STAMP(3);
    WAIT_LGKM0();
    BAR();
    const float scale = *ldsflag != 0u ? qnan : p.sx[0] * p.sw[0];
    if (dstr == (unsigned)rank) {
        const unsigned off0 = (unsigned)(ti * 32 + 4 * (lane >> 5)) * (GN * 4) + (unsigned)(rank * NSL + tj * 128 + 4 * (lane & 31)) * 4;
#pragma unroll
        for (int r = 0; r < 16; ++r) {
            f32x4 v;
#pragma unroll
            for (int c = 0; c < 4; ++c) { const float y = acc[c][r] * scale; v[c] = y <= 0.f ? 0.f : y; }
            __builtin_amdgcn_raw_buffer_store_b128(__builtin_bit_cast(u32x4, v), ro, off0 + (unsigned)((r & 3) + 8 * (r >> 2)) * (GN * 4), 0, AUX_SYS);
        }
    } else {
        const int tile = ti * NCB + tj;
        unsigned mk[2] = {0u, 0u}, base = 512;
        const unsigned dump = PK_DATA_END + lane * 4;
#pragma unroll
        for (int r = 0; r < 16; ++r)
#pragma unroll
            for (int c = 0; c < 4; ++c) {
                const float y = acc[c][r] * scale;
                const bool nz = !(y <= 0.f);
                const unsigned long long m = __ballot(nz);
                const unsigned pos = __builtin_amdgcn_mbcnt_hi((unsigned)(m >> 32), __builtin_amdgcn_mbcnt_lo((unsigned)m, 0u)) * 4 + base;
                *(LAS float*)(pk + (nz ? pos : dump)) = y;
                base += (unsigned)__builtin_popcountll(m) * 4;
                mk[r >> 3] = mk[r >> 3] * 2 + (nz ? 1u : 0u);
            }
        *(LAS u32x2*)(pk + lane * 8) = (u32x2){mk[0], mk[1]};
        const unsigned bytes = base;
        const int nst = (int)((bytes + 1023u) >> 10);
        const __amdgpu_buffer_rsrc_t rp = make_rsrc(myws + OFF_PK + (unsigned)(dstr * NWG + tile) * PK_SLOT, PK_SLOT);
        for (int i = 0; i < nst; ++i) {
            const u32x4 v = *(LAS const u32x4*)(pk + i * 1024 + lane * 16);
            __builtin_amdgcn_raw_buffer_store_b128(v, rp, i * 1024 + lane * 16, 0, AUX_SYS);
        }
        WAIT_VM(0);
        if (lane == 0) __hip_atomic_store((unsigned long long*)(p.ws[dstr] + OFF_FD + tile * 128 + rank * 8), ((unsigned long long)call << 32) | bytes,
                                          __ATOMIC_RELAXED, __HIP_MEMORY_SCOPE_SYSTEM);
    }
    STAMP(4);
    if (job == 0) { tslot += 32 - CS; ti = (xcd & 1) + 2 * (tslot >> 2); tj = 4 * (xcd >> 1) + (tslot & 3); }
    }
    if (peer == (unsigned)rank) {
        if (lane == 0) __hip_atomic_store(cntp, call, __ATOMIC_RELAXED, __HIP_MEMORY_SCOPE_AGENT);
        STAMP(5); STAMP(6);
        STAMP_FLUSH();
        return;
    }
    {
        const int tile = ti * NCB + tj;
        const unsigned off0 = (unsigned)(ti * 32 + 4 * (lane >> 5)) * (GN * 4) + (unsigned)(peer * NSL + tj * 128 + 4 * (lane & 31)) * 4;
        bool ok = true, permit = comm;
        unsigned bytes = 0;
        const unsigned long long t0 = __builtin_amdgcn_s_memrealtime();
        unsigned polls = 0;
        const unsigned long long* fp = (const unsigned long long*)(myws + OFF_FD + tile * 128 + (lane & 1) * 64 + peer * 8);
        for (;;) {
            unsigned long long f = (unsigned long long)call << 32;
            if (lane < 2) f = __hip_atomic_load(fp, __ATOMIC_RELAXED, __HIP_MEMORY_SCOPE_SYSTEM);
            bytes = __builtin_amdgcn_readfirstlane((unsigned)f);
            const unsigned long long miss = __ballot((unsigned)(f >> 32) < call);
            permit = comm || (miss & 2) == 0;
            if ((miss & 1) == 0) break;
            if (gave_up(t0, polls, call, myws + OFF_TMO)) { ok = false; break; }
            __builtin_amdgcn_s_sleep(RX_SLEEP);
        }
        auto pass_on = [&]() {
            unsigned g = 0;
            if (lane == 0) g = __hip_atomic_fetch_add((unsigned*)(myws + OFF_TK + peer * 256 + 128), 1u, __ATOMIC_RELAXED, __HIP_MEMORY_SCOPE_AGENT);
            grant(myws, peer, call, (unsigned)__builtin_amdgcn_readfirstlane(g), 1, lane);
        };
        asm volatile("" ::: "memory");
        if (!ok) {
            pass_on();
            for (int r = 0; r < 16; ++r)
                __builtin_amdgcn_raw_buffer_store_b128(__builtin_bit_cast(u32x4, (f32x4){qnan, qnan, qnan, qnan}), ro, off0 + (unsigned)((r & 3) + 8 * (r >> 2)) * (GN * 4), 0, AUX_SYS);
            return;
        }
        if (bytes > PK_DATA_END) bytes = PK_DATA_END;
        const int nld = (int)((bytes + 1023u) >> 10);
        const uint8_t* src = p.ws[peer] + OFF_PK + (unsigned)(rank * NWG + tile) * PK_SLOT + lane * 16;
        int win = 1;
        bool passed = false;
        const __amdgpu_buffer_rsrc_t rfl = make_rsrc(myws, FLAG_BYTES);
        unsigned ta = 0;
        int g = 0, i = 0;
        bool pass = false;
        auto issue = [&]() {
            const int n = win < nld - i ? win : nld - i;
            pass = i + n + CHAIN_EARLY >= nld && !passed;
            ta = (unsigned)__builtin_amdgcn_s_memrealtime();
            if (pass) g = __builtin_amdgcn_raw_ptr_buffer_atomic_add_i32(1, rfl, (lane == 0 ? 0u : 0x40000000u) + OFF_TK + peer * 256 + 128, 0, 0);
            for (int u = 0; u < n; ++u, ++i)
                if ((unsigned)(i * 1024 + (lane & ~(RGRAN - 1)) * 16) < bytes)
                    __builtin_amdgcn_global_load_lds((const void*)(src + i * 1024), (LAS void*)(pk + i * 1024), 16, 0, AUX_SYS);
        };
        unsigned tstart = 0;
        auto land = [&](bool steer) {
            WAIT_VM(0);
            asm volatile("" : "+v"(g));
            const unsigned now = (unsigned)__builtin_amdgcn_s_memrealtime(), lat = now - ta;
            if (pass) { passed = true; grant(myws, peer, call, (unsigned)__builtin_amdgcn_readfirstlane(g), 1, lane); }
            if (!steer) return;
            if (lat < LAT_LO) win = win * 2 < WMAX ? win * 2 : WMAX;
            else if (lat > LAT_HI) { if (win > 1) win >>= 1; else if (lat > LAT_BACK) __builtin_amdgcn_s_sleep(BACKOFF); }
            const int left = CHAIN + (NWG - 1 - tile) - (int)((now - tstart) / UNIT_TICKS);
            const int wmin = left > EG2 ? 1 : (left > EG3 ? 2 : (left > EG4 ? 3 : (left > EG4 / 2 ? 4 : 8)));
            win = win < wmin ? wmin : win;
        };
        *(LAS unsigned*)(pk + PK_DATA_END + 256 + lane * 4) = 0u;
        issue(); land(false);
        const u32x2 mm = *(LAS const u32x2*)(pk + lane * 8);
        WAIT_LGKM0();
        tstart = (unsigned)__builtin_amdgcn_s_memrealtime();
        STAMP(5);
        unsigned ad[64];
        unsigned base = 512;
        const unsigned zero = PK_DATA_END + 256 + lane * 4;
#pragma unroll
        for (int q = 0; q < ADSL; ++q) {
            const bool fly = permit && i < nld;
            if (fly) issue();
#pragma unroll
            for (int k = q * (64 / ADSL); k < (q + 1) * (64 / ADSL); ++k) {
                const bool nz = (int)((k < 32 ? mm.x : mm.y) << (k & 31)) < 0;
                const unsigned long long m = __ballot(nz);
                const unsigned pos = __builtin_amdgcn_mbcnt_hi((unsigned)(m >> 32), __builtin_amdgcn_mbcnt_lo((unsigned)m, 0u)) * 4 + base;
                base += (unsigned)__builtin_popcountll(m) * 4;
                ad[k] = (unsigned)(size_t)pk + (nz ? pos : zero);
                asm volatile("" : "+v"(ad[k]));
            }
            if (fly) land(true);
        }
        if (!permit && i < nld) {
            for (;;) {
                unsigned f = 0;
                if (lane == 0) f = __hip_atomic_load((const unsigned*)(myws + OFF_FD + tile * 128 + 64 + peer * 8 + 4), __ATOMIC_RELAXED, __HIP_MEMORY_SCOPE_SYSTEM);
                if ((unsigned)__builtin_amdgcn_readfirstlane(f) >= call || gave_up(t0, polls, call, myws + OFF_TMO)) break;
                __builtin_amdgcn_s_sleep(GATE_SLEEP);
            }
            tstart = (unsigned)__builtin_amdgcn_s_memrealtime();
            STAMP(5);
        }
        while (i < nld) { issue(); land(true); }
        __builtin_amdgcn_s_waitcnt(0x0F70);
        STAMP(6);
#pragma unroll
        for (int r = 0; r < 16; ++r) {
            f32x4 v;
#pragma unroll
            for (int c = 0; c < 4; ++c) v[c] = *(LAS const float*)(size_t)ad[r * 4 + c];
            __builtin_amdgcn_raw_buffer_store_b128(__builtin_bit_cast(u32x4, v), ro, off0 + (unsigned)((r & 3) + 8 * (r >> 2)) * (GN * 4), 0, AUX_SYS);
        }
        STAMP_FLUSH();
    }
}

}

extern "C" void kernel_launch(int rank, int nranks, ncclComm_t comm, hipStream_t stream,
                              void* const* d_in, const int* in_sizes, int n_in,
                              void* d_out, int out_size, const int* in_dtypes, int out_dtype,
                              void* const* d_out_all, void* const* d_ws_all, size_t ws_bytes) {
    (void)comm; (void)in_dtypes; (void)out_dtype; (void)d_out_all;
    if (nranks != NR || n_in != 4 || in_sizes[0] != GM * KSH || in_sizes[1] != KSH * GN || out_size != MSH * GN ||
        ws_bytes < WS_TOTAL || (unsigned)rank >= (unsigned)NR) {
        fprintf(stderr, "kernel_launch: unexpected shapes\n");
        return;
    }
    Params p;
    p.x = (const uint8_t*)d_in[0]; p.w = (const uint8_t*)d_in[1];
    p.sx = (const float*)d_in[2]; p.sw = (const float*)d_in[3];
    p.out = (uint8_t*)d_out;
    for (int i = 0; i < NR; ++i) p.ws[i] = (uint8_t*)d_ws_all[i];
    p.rank = rank; p.pad = 0; p.myws = p.ws[rank];
    HIP_CHECK(hipFuncSetAttribute((const void*)gemm_xchg, hipFuncAttributeMaxDynamicSharedMemorySize, LDS_BYTES));
    hipLaunchKernelGGL(gemm_xchg, dim3(NWG), dim3(NTHR), LDS_BYTES, stream, p);
    HIP_CHECK(hipGetLastError());
}
```

```cpp
#include <hip/hip_runtime.h>
#include <rccl/rccl.h>
#include <cstdio>
#include <cstdint>

#define HIP_CHECK(expr) do { hipError_t _e = (expr); if (_e != hipSuccess) { \
    fprintf(stderr, "HIP error %s at %s:%d\n", hipGetErrorString(_e), __FILE__, __LINE__); \
    return; } } while (0)

namespace {

typedef float f32x4 __attribute__((ext_vector_type(4)));
typedef float f32x16 __attribute__((ext_vector_type(16)));
typedef int i32x4 __attribute__((ext_vector_type(4)));
typedef int i32x8 __attribute__((ext_vector_type(8)));
typedef unsigned u32x2 __attribute__((ext_vector_type(2)));
typedef unsigned u32x4 __attribute__((ext_vector_type(4)));
#define LAS __attribute__((address_space(3)))

constexpr int NR = 8;
constexpr int GM = 4096, GK = 4096, GN = 16384;
constexpr int KSH = GK / NR;
constexpr int MSH = GM / NR;
constexpr int NKS = GK / 64;
constexpr int NWG = 256;
constexpr int NTHR = 512;
constexpr int NCOMM = 32;
constexpr int PB = 4096;

constexpr unsigned OFF_FX = 0;
constexpr unsigned OFF_FW = 16384;
constexpr unsigned OFF_FO = 32768;
constexpr unsigned OFF_FS = 34816;
constexpr unsigned OFF_XC = 36864;
constexpr unsigned OFF_TMO = 37120;
constexpr unsigned OFF_TK = 38912;
constexpr unsigned FLAG_BYTES = 65536;
constexpr unsigned OFF_FD = 65536;
constexpr unsigned OFF_CNT = OFF_FD + NWG * 128;
constexpr unsigned OFF_DBG = OFF_CNT + NWG * 128;
constexpr unsigned OFF_SX = 1u << 20;
constexpr unsigned OFF_SW = OFF_SX + 128 * 16384;
constexpr unsigned S_COPY = 128 * 16384 + 128 * 65536;
constexpr unsigned OFF_XS = OFF_SX + 2 * S_COPY;
constexpr unsigned OFF_WS = OFF_XS + 80 * 131072;
constexpr unsigned PK_SLOT = 17408;
constexpr unsigned PK_DATA_END = 512 + 16384;
constexpr unsigned OFF_PK = OFF_WS + 40 * 524288;
constexpr unsigned WS_TOTAL = OFF_PK + NWG * 8 * PK_SLOT;
constexpr unsigned OUT_BYTES = (unsigned)MSH * GN * 4;
constexpr unsigned OOB = 0x40000000u;
static_assert(WS_TOTAL < OOB, "");

constexpr int NS = 6;
constexpr int SLOT = 24576;
constexpr int LDS_BYTES = NS * SLOT + 64;
constexpr unsigned long long DEADLINE_TICKS = 50000000ull;
constexpr int AUX_SYS = 17;

#define POLL_SLEEP 8
#define POLL_SLOW 32
#define RX_SLEEP 8
#define XMON 100
#define ROT 3
#define STAGE_DELAY 32
#define STAGE_REP 1
#define LEAD 8
#define DEADL 0
#define DL_HALF 36
#define DL_RND 4
#define JC 8
#define START2 8
#define RPOLL 0
#define PH0_DELAY 127
#define MON_SLEEP 4
#define UNIT_TICKS 13
#define EG2 44
#define EG3 26
#define EG4 18
#define ACQ 3
#define GR0 40
#define CHAIN_EARLY 2
#define GRANT_EARLY 16
#define GATE_SLEEP 16
#define LAT_LO 85
#define LAT_HI 110
#define LAT_BACK 170
#define BACKOFF 16
#define WMAX 8

struct Params {
    const uint8_t* x;
    const uint8_t* w;
    uint8_t* myws;
    int rank, pad;
    const float* sx;
    const float* sw;
    uint8_t* out;
    uint8_t* ws[NR];
};

#define WAIT_VM(n) asm volatile("s_waitcnt vmcnt(%0)" :: "n"(n) : "memory")
#define WAIT_LGKM0() asm volatile("s_waitcnt lgkmcnt(0)" ::: "memory")
#define BAR() do { asm volatile("" ::: "memory"); __builtin_amdgcn_s_barrier(); asm volatile("" ::: "memory"); } while (0)

#define STAMP(k) do {} while (0)
#define STAMP_FLUSH() do {} while (0)

__device__ __forceinline__ __amdgpu_buffer_rsrc_t make_rsrc(const void* p, unsigned bytes) {
    return __builtin_amdgcn_make_buffer_rsrc((void*)p, 0, (int)bytes, 0x00020000);
}
__device__ __forceinline__ void flag_store(uint8_t* p, unsigned v) {
    __hip_atomic_store((unsigned*)p, v, __ATOMIC_RELAXED, __HIP_MEMORY_SCOPE_SYSTEM);
}
__device__ __forceinline__ bool gave_up(unsigned long long t0, unsigned& polls, unsigned call, uint8_t* tmo) {
    if ((++polls & 63u) != 0u) return false;
    const unsigned t = __builtin_amdgcn_readfirstlane(__hip_atomic_load((const unsigned*)tmo, __ATOMIC_RELAXED, __HIP_MEMORY_SCOPE_AGENT));
    if (t == call || __builtin_amdgcn_s_memrealtime() - t0 > DEADLINE_TICKS) {
        __hip_atomic_store((unsigned*)tmo, call, __ATOMIC_RELAXED, __HIP_MEMORY_SCOPE_AGENT);
        return true;
    }
    return false;
}
__device__ __forceinline__ bool all_ge(u32x4 v, unsigned c) { return v.x >= c && v.y >= c && v.z >= c && v.w >= c; }

__host__ __device__ __forceinline__ int partner(int p, int m) { return p < 4 ? 4 + ((p + m) & 3) : ((p - m) & 3); }
__host__ __device__ __forceinline__ int gcb_of(int p, int J) {
    const int q = partner(p, J & 3), g = p < 4 ? p * 4 + (q & 3) : q * 4 + (p & 3);
    return g * 8 + (J >> 2);
}
__host__ __device__ __forceinline__ int grg_of(int p, int i, int e) {
    if (e < 2) return p * 16 + 2 * i + e;
    if (e < 6) return partner(p, e - 2) * 16 + i;
    if (i < 7) return ((p & 4) | ((p + 1 + e - 6) & 3)) * 16 + 9 + i;
    return partner(p, e - 6) * 16 + 8;
}
__host__ __device__ __forceinline__ int entry_of(int role, int i, int J) {
    if (role < 2) return role;
    if (role < 5) return 2 + ((J + role - 1) & 3);
    return i < 7 ? role + 1 : 6 + ((J + role - 4) & 3);
}
__host__ __device__ __forceinline__ int mirror(int role, int i) { return role >= 5 && i < 7 ? 12 - role : role; }

__device__ __forceinline__ unsigned ws_block(int J, unsigned call) { return OFF_WS + (unsigned)(J < 8 && (call & 1u) ? J + 32 : J) * 524288u; }

__device__ __forceinline__ bool wait_operands(uint8_t* ws, int i, int J, int lane, unsigned call, int ph, int rank) {
    const unsigned long long t0 = __builtin_amdgcn_s_memrealtime();
    const __amdgpu_buffer_rsrc_t rf = make_rsrc(ws, FLAG_BYTES);
    unsigned polls = 0;
    const unsigned wbase = OFF_FW + (J * 2 + ph - 1) * 256;
    if (ph != 0) {
        for (;;) {
            unsigned v = call;
            if (lane < 8 && lane != rank) v = __builtin_amdgcn_raw_buffer_load_b32(rf, wbase + lane * 32 + i * 4, 0, AUX_SYS);
            if (__builtin_popcountll(__ballot(v >= call)) >= 57 + 2) break;
            if (gave_up(t0, polls, call, ws + OFF_TMO)) return false;
            __builtin_amdgcn_s_sleep(POLL_SLOW);
            asm volatile("" ::: "memory");
        }
    }
    const int d0 = (lane & 7) * 4, nv = i < 7 ? 18 : 20;
    const bool xneed = ph == 0 || ((lane >> 3) != rank && d0 < nv);
    const bool xhalf = ph != 0 && d0 + 2 >= nv;
    const unsigned xoff = (ph == 0 ? OFF_FO : OFF_FX + (i * 2 + ph - 1) * 1024) + lane * 16;
    const bool wneed = ph != 0 && lane < 16 && (lane >> 1) != rank;
    bool xok = false, wok = false;
    for (;;) {
        u32x4 v = {call, call, call, call}, w = v;
        if (xneed && !xok) v = __builtin_amdgcn_raw_buffer_load_b128(rf, xoff, 0, AUX_SYS);
        if (wneed && !wok) w = __builtin_amdgcn_raw_buffer_load_b128(rf, wbase + lane * 16, 0, AUX_SYS);
        if (xhalf) { v.z = call; v.w = call; }
        xok = __all(all_ge(v, call)); wok = __all(all_ge(w, call));
        if (wok && xok) return true;
        if (gave_up(t0, polls, call, ws + OFF_TMO)) return false;
        if (xok) __builtin_amdgcn_s_sleep(POLL_SLEEP); else __builtin_amdgcn_s_sleep(POLL_SLOW);
        asm volatile("" ::: "memory");
    }
}

__device__ __forceinline__ void grant(uint8_t* ws, int rank, int peer, unsigned call, unsigned first, int ng, int lane) {
    const bool same = ((rank ^ peer) & 4) == 0;
    const unsigned nu = same ? 224u : 216u;
    unsigned n = first - (call - 1u) * (GR0 + nu) + (unsigned)lane;
    const unsigned self = same ? 28u : 27u;
    if (n >= self * (JC / 4)) n += self;
    if (lane < ng && n < nu) {
        unsigned tile, role;
        if (same) { tile = (n / 7u) * 8u + n % 7u; role = 5u + ((unsigned)(peer - rank - 1) & 3u); }
        else {
            const unsigned m = (rank < 4 ? (unsigned)(peer - rank) : (unsigned)(rank - peer)) & 3u;
            const unsigned c = n / 9u, u = n % 9u, tt = c % 3u, t = tt < m ? tt : tt + 1u;
            tile = ((c / 3u) * 4u + t) * 8u + (u < 8u ? u : 7u);
            role = (u < 8u ? 2u : 5u) + ((m - t - 1u) & 3u);
        }
        flag_store(ws + OFF_FD + tile * 128 + 64 + role * 8 + 4, call);
    }
}

struct Frags { i32x4 a[2]; i32x4 b[4][2]; };
__device__ __forceinline__ void read_frags(Frags& f, LAS const uint8_t* pa, LAS const uint8_t* pb) {
    f.a[0] = *(LAS const i32x4*)(pa); f.a[1] = *(LAS const i32x4*)(pa + 1024);
#pragma unroll
    for (int c = 0; c < 4; ++c) { f.b[c][0] = *(LAS const i32x4*)(pb + c * 2048); f.b[c][1] = *(LAS const i32x4*)(pb + c * 2048 + 1024); }
}
__device__ __forceinline__ i32x8 cat(i32x4 a, i32x4 b) { return __builtin_shufflevector(a, b, 0, 1, 2, 3, 4, 5, 6, 7); }
template <int C0, int C1> __device__ __forceinline__ void mma(f32x16 (&acc)[4], const Frags& f) {
    const i32x8 a = cat(f.a[0], f.a[1]);
#pragma unroll
    for (int c = C0; c < C1; ++c)
        acc[c] = __builtin_amdgcn_mfma_scale_f32_32x32x64_f8f6f4(a, cat(f.b[c][0], f.b[c][1]), acc[c], 0, 0, 0, 127, 0, 127);
}

constexpr int XPC = 146;
constexpr int NPC = 2 * XPC + (32 + 24) * 8;
struct Piece { unsigned so, dso, fo; int staged; };
__device__ __forceinline__ Piece piece_of(int pc, int rank, int peer, unsigned call) {
    Piece r;
    const unsigned cp = (call & 1u) * S_COPY;
    constexpr int E = (LEAD - 8) * 8, NP2 = 2 * (32 - LEAD);
    int n = -1, half = 0, V = 0;
    if (pc < XPC) n = pc;
    else if (pc < XPC + E) V = pc - XPC;
    else if (pc < 2 * XPC + E) { n = pc - XPC - E; half = 1; }
    else V = pc - 2 * XPC;
    if (n >= 0) {
        int i = n / 18; i = i < 7 ? i : 7;
        const int rem = n - 18 * i, e = rem >> 1;
        const unsigned o = (unsigned)(half * 2 + (rem & 1)) * PB;
        r.so = OFF_SX + cp + (unsigned)grg_of(rank, i, e) * 16384u + o;
        r.dso = OFF_XS + (unsigned)(i * 10 + e) * 131072u + (unsigned)peer * 16384u + o;
        r.fo = OFF_FX + (unsigned)((i * 2 + half) * 8 + peer) * 128u + (unsigned)rem * 4u;
        r.staged = half ^ 1;
    } else {
        const int k = V & 7, U = (V >> 3) - (LEAD - 8);
        int J;
        if (U < 0) { half = 0; J = LEAD + U; }
        else if (U < NP2) { half = (U & 1) ^ 1; J = (U >> 1) + (U & 1) * LEAD; }
        else { half = 1; J = U - NP2 + 32 - LEAD; }
        const unsigned o = (unsigned)(half * 8 + k) * PB;
        r.so = OFF_SW + cp + (unsigned)gcb_of(rank, J) * 65536u + o;
        r.dso = ws_block(J, call) + (unsigned)peer * 65536u + o;
        r.fo = OFF_FW + (unsigned)((J * 2 + half) * 8 + peer) * 32u + (unsigned)k * 4u;
        r.staged = 0;
    }
    return r;
}

__global__ void __launch_bounds__(NTHR, 2) gemm_xchg(const Params p) {
    extern __shared__ __attribute__((aligned(16))) uint8_t smem_raw[];
    LAS uint8_t* lds = (LAS uint8_t*)smem_raw;
    const int tid = threadIdx.x, lane = tid & 63, wv = __builtin_amdgcn_readfirstlane(tid >> 6);
    const int rank = p.rank;
    uint8_t* myws = p.myws;
    const int b = blockIdx.x, xcd = b & 7, slot = b >> 3;
    const bool comm = slot >= 28;
    const int rot = (wv + ROT * slot) & 7;
    STAMP(0);
    unsigned* cntp = (unsigned*)(myws + OFF_CNT + b * 128);
    const unsigned cnt_v = __hip_atomic_load(cntp, __ATOMIC_RELAXED, __HIP_MEMORY_SCOPE_AGENT);
    LAS unsigned* ldsflag = (LAS unsigned*)(lds + NS * SLOT);
    if (tid < 2 * NR) *(LAS unsigned*)(lds + NR * PK_SLOT + tid * 4) = 0u;
    if (tid < 2) ldsflag[tid] = 0u;
    const __amdgpu_buffer_rsrc_t rm = make_rsrc(myws, WS_TOTAL);
    unsigned call = 0;
    uint8_t* const lanews = p.ws[lane & 7];

    {
        const int l = lane & 31, kg = lane >> 5;
        if (slot >= 20 && slot < 28) {
            const int y = b - 160, half = y >> 5, grg = 4 * (y & 31) + (wv >> 1), ks = half * 4 + 2 * (wv & 1);
            if (half) for (int z = 0; z < STAGE_REP; ++z) __builtin_amdgcn_s_sleep(STAGE_DELAY);
            const uint8_t* src = p.x + (size_t)(grg * 32 + l) * KSH + ks * 64 + kg * 16;
            u32x4 v[4];
#pragma unroll
            for (int u = 0; u < 4; ++u) v[u] = *(const u32x4*)(src + u * 32);
            call = __builtin_amdgcn_readfirstlane(cnt_v) + 1u;
#pragma unroll
            for (int u = 0; u < 4; ++u) __builtin_amdgcn_raw_buffer_store_b128(v[u], rm, OFF_SX + (call & 1u) * S_COPY + grg * 16384 + ks * 2048 + u * 1024 + lane * 16, 0, AUX_SYS);
            if (half == 0) {
                WAIT_VM(0);
                BAR();
                if (wv == 0) {
                    unsigned t = 0;
                    if (lane == 0) t = __hip_atomic_fetch_add((unsigned*)(myws + OFF_XC), 1u, __ATOMIC_RELAXED, __HIP_MEMORY_SCOPE_AGENT);
                    if ((unsigned)__builtin_amdgcn_readfirstlane(t) == call * 32u - 1u && lane < NR) flag_store(lanews + OFF_FS + rank * 256 + 128, call);
                }
            }
        }
        const bool push = slot >= 16 && slot < 20;
        int gcb, half;
        if (push) { const int n = b - 128; gcb = (n & 15) * 8 + (n >> 4); half = 0; }
        else { const int n = b < 128 ? b : b - 32, g = n / 14, r = n - g * 14; gcb = g * 8 + (r < 8 ? r : r - 6); half = r < 8 ? 1 : 0; }
        if (slot < 16 || (slot >= 20 && slot < 24)) for (int z = 0; z < STAGE_REP; ++z) __builtin_amdgcn_s_sleep(STAGE_DELAY);
        const int sl = 4 * half + (wv >> 1), h = wv & 1;
        const uint8_t* wsrc = p.w + (size_t)(sl * 64 + h * 32 + (lane >> 3)) * GN + gcb * 128 + (lane & 7) * 16;
#pragma unroll
        for (int u = 0; u < 4; ++u)
            __builtin_amdgcn_global_load_lds((const void*)(wsrc + (size_t)u * 8 * GN), (LAS void*)(lds + wv * 4096 + u * 1024), 16, 0, 0);
        const int ha = gcb >> 5, hb = 4 + ((gcb >> 3) & 3), Jh = (gcb & 7) * 4 + ((hb - ha) & 3);
        uint8_t* wa = p.ws[ha]; uint8_t* wb = p.ws[hb];
        WAIT_VM(0);
        unsigned d[16];
#pragma unroll
        for (int kk = 0; kk < 16; ++kk) d[kk] = *(LAS const unsigned*)(lds + wv * 4096 + (kg * 16 + kk) * 128 + l * 4);
        u32x4 o[4];
#pragma unroll
        for (int m = 0; m < 4; ++m) {
            const unsigned d0 = d[4 * m], d1 = d[4 * m + 1], d2 = d[4 * m + 2], d3 = d[4 * m + 3];
            const unsigned t0 = __builtin_amdgcn_perm(d1, d0, 0x05010400u), t1 = __builtin_amdgcn_perm(d1, d0, 0x07030602u);
            const unsigned t2 = __builtin_amdgcn_perm(d3, d2, 0x05010400u), t3 = __builtin_amdgcn_perm(d3, d2, 0x07030602u);
            o[0][m] = __builtin_amdgcn_perm(t2, t0, 0x05040100u);
            o[1][m] = __builtin_amdgcn_perm(t2, t0, 0x07060302u);
            o[2][m] = __builtin_amdgcn_perm(t3, t1, 0x05040100u);
            o[3][m] = __builtin_amdgcn_perm(t3, t1, 0x07060302u);
        }
        call = __builtin_amdgcn_readfirstlane(cnt_v) + 1u;
        const unsigned loff = OFF_SW + (call & 1u) * S_COPY + (unsigned)(gcb * 8 + sl) * 8192u, roff = ws_block(Jh, call) + (unsigned)(rank * 8 + sl) * 8192u;
        const unsigned lo = h * 1024 + lane * 16;
        if (push) {
            const __amdgpu_buffer_rsrc_t ra = make_rsrc(wa, WS_TOTAL), rb = make_rsrc(wb, WS_TOTAL);
            const unsigned oa = (ha == rank ? loff : roff) + lo, ob = (hb == rank ? loff : roff) + lo;
#pragma unroll
            for (int c = 0; c < 4; ++c) {
                if (c) WAIT_VM(1);
                __builtin_amdgcn_raw_buffer_store_b128(o[c], ra, oa + c * 2048, 0, AUX_SYS);
                if (c) WAIT_VM(1);
                __builtin_amdgcn_raw_buffer_store_b128(o[c], rb, ob + c * 2048, 0, AUX_SYS);
            }
        } else {
#pragma unroll
            for (int c = 0; c < 4; ++c) __builtin_amdgcn_raw_buffer_store_b128(o[c], rm, loff + lo + c * 2048, 0, AUX_SYS);
        }
        WAIT_VM(0);
        WAIT_LGKM0();
        BAR();
        if (rot == rank) {
            if (lane == 0) flag_store(myws + OFF_FO + b * 4, call);
            if (push && lane < 16) flag_store((lane < 8 ? wa : wb) + OFF_FW + (Jh * 2 * 8 + rank) * 32 + (lane & 7) * 4, call);
            if (b == XMON) {
                const unsigned long long t0 = __builtin_amdgcn_s_memrealtime();
                unsigned polls = 0;
                for (;;) {
                    const u32x4 v = __builtin_amdgcn_raw_buffer_load_b128(rm, OFF_FO + lane * 16, 0, AUX_SYS);
                    if (__all(all_ge(v, call))) {
                        if (lane < NR) flag_store(lanews + OFF_FS + rank * 256, call);
                        break;
                    }
                    if (gave_up(t0, polls, call, myws + OFF_TMO)) break;
                    __builtin_amdgcn_s_sleep(16);
                    asm volatile("" ::: "memory");
                }
            }
        }
    }

    if (comm) {
        const int pk_ = b - 224;
        volatile LAS unsigned* fs = (volatile LAS unsigned*)(lds + NR * PK_SLOT);
        const unsigned long long t0 = __builtin_amdgcn_s_memrealtime();
        unsigned polls = 0;
        if (rot == rank) {
            for (;;) {
                unsigned v = call;
                if (lane < 2 * NR) v = __hip_atomic_load((const unsigned*)(myws + OFF_FS + lane * 128), __ATOMIC_RELAXED, __HIP_MEMORY_SCOPE_SYSTEM);
                if ((__ballot(v < call) & 0xAAAAull) == 0 && lane < 2 * NR) fs[lane] = v;
                if (__all(v >= call)) break;
                if (gave_up(t0, polls, call, myws + OFF_TMO)) { if (lane == 0) *(volatile LAS unsigned*)(ldsflag + 1) = 1u; break; }
                __builtin_amdgcn_s_sleep(MON_SLEEP);
            }
        } else {
            const int peer = rot;
            const __amdgpu_buffer_rsrc_t rs = make_rsrc(p.ws[peer], WS_TOTAL);
            unsigned dead = 0;
            auto wait_staged = [&](int idx) {
                while ((unsigned)__builtin_amdgcn_readfirstlane(fs[peer * 2 + idx]) < call) {
                    if (__builtin_amdgcn_readfirstlane(*(volatile LAS unsigned*)(ldsflag + 1)) != 0u) { dead = OOB; break; }
                    __builtin_amdgcn_s_sleep(1);
                }
                asm volatile("" ::: "memory");
            };
            const unsigned lo = lane * 16, one = lane == 0 ? 0u : OOB;
            const unsigned tkbase = (call - 1u) * NPC - NCOMM;
            Piece cur = piece_of(pk_, rank, peer, call);
            unsigned pfo = OOB;
            u32x4 v[2];
            wait_staged(cur.staged);
#pragma unroll
            for (int u = 0; u < 2; ++u) {
                __builtin_amdgcn_raw_buffer_store_b128((u32x4){0u, 0u, 0u, 0u}, rm, OOB + lo + u * 1024, 0, AUX_SYS);
                v[u] = __builtin_amdgcn_raw_buffer_load_b128(rs, lo + u * 1024, cur.so, AUX_SYS);
                asm volatile("" ::: "memory");
            }
            int tk = __builtin_amdgcn_raw_ptr_buffer_atomic_add_i32(1, rm, one + OFF_TK + peer * 256, 0, 0);
            bool granted = false;
            for (;;) {
                Piece nxt = cur;
                bool more = false;
                unsigned lo2 = OOB + lo, tko = OOB;
#pragma unroll
                for (int u = 0; u < 4; ++u) {
                    __builtin_amdgcn_raw_buffer_store_b128(v[u & 1], rm, lo + u * 1024, cur.dso, AUX_SYS);
                    if (u < 2) v[u & 1] = __builtin_amdgcn_raw_buffer_load_b128(rs, lo + (u + 2) * 1024, cur.so, AUX_SYS);
                    else v[u & 1] = __builtin_amdgcn_raw_buffer_load_b128(rs, lo2 + (u - 2) * 1024, nxt.so, AUX_SYS);
                    if (u == 0) {
                        WAIT_VM(4);
                        __builtin_amdgcn_raw_buffer_store_b32(call, rm, one | pfo | dead, 0, AUX_SYS);
                    }
                    if (u == 1) {
                        const unsigned pc = (unsigned)__builtin_amdgcn_readfirstlane(tk) - tkbase;
                        more = pc < (unsigned)NPC;
                        if (more) { nxt = piece_of((int)pc, rank, peer, call); wait_staged(nxt.staged); lo2 = lo; tko = one; }
                        if (pc + GRANT_EARLY >= (unsigned)NPC && !granted) {
                            granted = true;
                            const int ng = GR0 / NCOMM + (pk_ < GR0 % NCOMM ? 1 : 0);
                            const int g = __builtin_amdgcn_raw_ptr_buffer_atomic_add_i32(ng, rm, one + OFF_TK + peer * 256 + 128, 0, 0);
                            grant(myws, rank, peer, call, (unsigned)__builtin_amdgcn_readfirstlane(g), ng, lane);
                        }
                    }
                    if (u == 3) tk = __builtin_amdgcn_raw_ptr_buffer_atomic_add_i32(1, rm, tko + OFF_TK + peer * 256, 0, 0);
                    asm volatile("" ::: "memory");
                }
                pfo = cur.fo;
                cur = nxt;
                if (!more) break;
            }
            WAIT_VM(0);
            __builtin_amdgcn_raw_buffer_store_b32(call, rm, one | pfo | dead, 0, AUX_SYS);
        }
    }
    STAMP(1);
    const float qnan = __builtin_nanf("");
    float sxw = p.sx[0] * p.sw[0];
    asm volatile("" : "+v"(sxw));
    const __amdgpu_buffer_rsrc_t ro = make_rsrc(p.out, OUT_BYTES);
    LAS uint8_t* pk = lds + wv * PK_SLOT;
    const int ti = 2 * (xcd & 3) + (slot & 1);
    int tJ = 2 * (slot >> 1) + (xcd >> 2);
    for (int job = comm ? 2 : (slot < 4 ? 0 : 1); job < 2; ++job) {
    f32x16 acc[4];
#pragma unroll
    for (int c = 0; c < 4; ++c)
#pragma unroll
        for (int r = 0; r < 16; ++r) acc[c][r] = 0.f;
    const int ent = entry_of(wv, ti, tJ), grg = grg_of(rank, ti, ent), gcb = gcb_of(rank, tJ);
    for (int ph = 0; ph < 3; ++ph) {
        if (wv == 0) {
            if (ph == 0 && tJ < 28) __builtin_amdgcn_s_sleep(PH0_DELAY);
            const bool ok = wait_operands(myws, ti, tJ, lane, call, ph, rank);
            if (ph == 1 && tJ < 8) __builtin_amdgcn_fence(__ATOMIC_ACQUIRE, "");
            else __builtin_amdgcn_fence(__ATOMIC_ACQUIRE, "agent");
            if (lane == 0 && !ok) *ldsflag = 1u;
            asm volatile("s_waitcnt vmcnt(0)" ::: "memory");
        }
        WAIT_LGKM0();
        BAR();
        STAMP(2);
        const int nstep = ph == 0 ? 8 : 28;
        const int sh = ph == 0 ? 3 : 2, sbase = ph == 0 ? 0 : rank + 1, sadd = ph == 2 ? 4 : 0;
        const uint8_t* ga = myws + (ph == 0 ? OFF_SX + (call & 1u) * S_COPY + (unsigned)grg * 16384u : OFF_XS + (unsigned)(ti * 10 + ent) * 131072u) + lane * 16;
        const uint8_t* gb = myws + (ph == 0 ? OFF_SW + (call & 1u) * S_COPY + (unsigned)gcb * 65536u : ws_block(tJ, call)) + wv * 1024 + lane * 16;
        auto dma = [&](int n, int sl) {
            const unsigned s = (unsigned)((((sbase + (n >> sh)) & 7) << 3) + (n & ((1 << sh) - 1)) + sadd);
            LAS uint8_t* d = lds + sl * SLOT;
            __builtin_amdgcn_global_load_lds((const void*)(ga + s * 2048u), (LAS void*)(d + wv * 2048), 16, 0, 0);
            __builtin_amdgcn_global_load_lds((const void*)(ga + s * 2048u + 1024), (LAS void*)(d + wv * 2048 + 1024), 16, 0, 0);
            __builtin_amdgcn_global_load_lds((const void*)(gb + s * 8192u), (LAS void*)(d + 16384 + wv * 1024), 16, 0, 0);
        };
        LAS const uint8_t* pa = lds + wv * 2048 + lane * 16;
        LAS const uint8_t* pb = lds + 16384 + lane * 16;
#pragma unroll
        for (int s = 0; s < NS; ++s) dma(s, s);
        WAIT_VM(3 * (NS - 1));
        BAR();
        Frags f0, f1;
        read_frags(f0, pa, pb);
        int sl = 0;
#define SGB(mask, n) __builtin_amdgcn_sched_group_barrier(mask, n, 0)
#define KSTEP(FC, FN, S, VMN, DO_DMA) do { \
            __builtin_amdgcn_sched_barrier(0); \
            WAIT_VM(VMN); WAIT_LGKM0(); BAR(); \
            __builtin_amdgcn_sched_barrier(0); \
            mma<0, 1>(acc, FC); \
            __builtin_amdgcn_sched_barrier(0); \
            const int nsl = sl + 1 == NS ? 0 : sl + 1; \
            read_frags(FN, pa + nsl * SLOT, pb + nsl * SLOT); \
            if (DO_DMA) dma((S) + NS, sl); \
            mma<1, 4>(acc, FC); \
            SGB(0x100, 6); SGB(0x8, 1); SGB(0x100, 4); SGB(0x10, 1); \
            SGB(0x8, 1); SGB(0x10, 2); \
            SGB(0x8, 1); \
            sl = nsl; } while (0)
        for (int s = 0; s < nstep - NS; s += 2) {
            KSTEP(f0, f1, s, 3 * (NS - 2), true);
            KSTEP(f1, f0, s + 1, 3 * (NS - 2), true);
        }
        static_assert(NS == 6, "tail below is written for NS == 6");
        KSTEP(f0, f1, 0, 12, false);
        KSTEP(f1, f0, 0, 9, false);
        KSTEP(f0, f1, 0, 6, false);
        KSTEP(f1, f0, 0, 3, false);
        KSTEP(f0, f1, 0, 0, false);
        mma<0, 4>(acc, f1);
#undef KSTEP
    }
    STAMP(3);
    WAIT_LGKM0();
    BAR();
    const float scale = *ldsflag != 0u ? qnan : sxw;
    if (wv < 2) {
        const unsigned off0 = (unsigned)((grg & 15) * 32 + 4 * (lane >> 5)) * (GN * 4) + (unsigned)(gcb * 128 + 4 * (lane & 31)) * 4;
#pragma unroll
        for (int r = 0; r < 16; ++r) {
            f32x4 v;
#pragma unroll
            for (int c = 0; c < 4; ++c) { const float y = acc[c][r] * scale; v[c] = y <= 0.f ? 0.f : y; }
            __builtin_amdgcn_raw_buffer_store_b128(__builtin_bit_cast(u32x4, v), ro, off0 + (unsigned)((r & 3) + 8 * (r >> 2)) * (GN * 4), 0, AUX_SYS);
        }
    } else {
        const int tile = tJ * 8 + ti;
        unsigned mk[2] = {0u, 0u}, base = 512 + (unsigned)(size_t)pk;
        const unsigned dump = (unsigned)(size_t)pk + PK_DATA_END + lane * 4;
#pragma unroll
        for (int r = 0; r < 16; ++r)
#pragma unroll
            for (int c = 0; c < 4; ++c) {
                const float y = acc[c][r] * scale;
                const bool nz = !(y <= 0.f);
                const unsigned long long m = __ballot(nz);
                const unsigned pos = __builtin_amdgcn_mbcnt_hi((unsigned)(m >> 32), __builtin_amdgcn_mbcnt_lo((unsigned)m, 0u)) * 4 + base;
                *(LAS float*)(size_t)(nz ? pos : dump) = y;
                base += (unsigned)__builtin_popcountll(m) * 4;
                asm("" : "+s"(base));
                unsigned long long co;
                asm("v_addc_co_u32 %0, %1, %0, %0, %2" : "+v"(mk[r >> 3]), "=s"(co) : "s"(m));
            }
        base -= (unsigned)(size_t)pk;
        *(LAS u32x2*)(pk + lane * 8) = (u32x2){mk[0], mk[1]};
        const unsigned bytes = base;
        const __amdgpu_buffer_rsrc_t rp = make_rsrc(myws + OFF_PK + (unsigned)(tile * 8 + wv) * PK_SLOT, (bytes + 127u) & ~127u);
        for (unsigned o = 0; o < bytes; o += 5120) {
            u32x4 v[5];
#pragma unroll
            for (int u = 0; u < 5; ++u) v[u] = *(LAS const u32x4*)(pk + (o + u * 1024 < PK_SLOT ? o + u * 1024 : 0u) + lane * 16);
#pragma unroll
            for (int u = 0; u < 5; ++u) __builtin_amdgcn_raw_buffer_store_b128(v[u], rp, o + u * 1024 + lane * 16, 0, AUX_SYS);
        }
        WAIT_VM(0);
        if (lane == 0) __hip_atomic_store((unsigned long long*)(p.ws[(grg >> 4) & 7] + OFF_FD + tile * 128 + mirror(wv, ti) * 8),
                                          ((unsigned long long)call << 32) | bytes, __ATOMIC_RELAXED, __HIP_MEMORY_SCOPE_SYSTEM);
    }
    STAMP(4);
    if (job == 0) tJ += 28;
    }
    if (comm) tJ += JC - 28; else if (tJ < JC + 4) tJ -= 4;
    if (rot < 2) {
        if (rot == 0 && lane == 0) __hip_atomic_store(cntp, call, __ATOMIC_RELAXED, __HIP_MEMORY_SCOPE_AGENT);
        STAMP(5); STAMP(6);
        STAMP_FLUSH();
        return;
    }
    {
        const int tile = tJ * 8 + ti, g2 = grg_of(rank, ti, entry_of(rot, ti, tJ)), peer = (g2 >> 4) & 7;
        const unsigned off0 = (unsigned)((g2 & 15) * 32 + 4 * (lane >> 5)) * (GN * 4) + (unsigned)(gcb_of(peer, tJ) * 128 + 4 * (lane & 31)) * 4;
        bool ok = true, permit = comm;
        unsigned bytes = 0;
        const unsigned long long t0 = __builtin_amdgcn_s_memrealtime();
        unsigned polls = 0;
        const unsigned long long* fp = (const unsigned long long*)(myws + OFF_FD + tile * 128 + (lane & 1) * 64 + rot * 8);
        for (;;) {
            unsigned long long f = (unsigned long long)call << 32;
            if (lane < 2) f = __hip_atomic_load(fp, __ATOMIC_RELAXED, __HIP_MEMORY_SCOPE_SYSTEM);
            bytes = __builtin_amdgcn_readfirstlane((unsigned)f);
            const unsigned long long miss = __ballot((unsigned)(f >> 32) < call);
            permit = comm || (miss & 2) == 0;
            if ((miss & 1) == 0) break;
            if (gave_up(t0, polls, call, myws + OFF_TMO)) { ok = false; break; }
            __builtin_amdgcn_s_sleep(RX_SLEEP);
        }
        asm volatile("" ::: "memory");
        if (!ok) {
            for (int r = 0; r < 16; ++r)
                __builtin_amdgcn_raw_buffer_store_b128(__builtin_bit_cast(u32x4, (f32x4){qnan, qnan, qnan, qnan}), ro, off0 + (unsigned)((r & 3) + 8 * (r >> 2)) * (GN * 4), 0, AUX_SYS);
            return;
        }
        if (bytes > PK_DATA_END) bytes = PK_DATA_END;
        const int nld = (int)((bytes + 1023u) >> 10);
        const uint8_t* src = p.ws[peer] + OFF_PK + (unsigned)(tile * 8 + mirror(rot, ti)) * PK_SLOT + lane * 16;
        int win = 1;
        bool passed = false, pass = false;
        unsigned ta = 0, tstart = 0;
        int g = 0, i = 0;
        const int behind = (NWG - 1 - tile) * 7 / 8;
        auto issue = [&]() {
            const int n = win < nld - i ? win : nld - i;
            pass = i + n + CHAIN_EARLY >= nld && !passed;
            ta = (unsigned)__builtin_amdgcn_s_memrealtime();
            if (pass) g = __builtin_amdgcn_raw_ptr_buffer_atomic_add_i32(1, rm, (lane == 0 ? 0u : OOB) + OFF_TK + peer * 256 + 128, 0, 0);
            for (int u = 0; u < n; ++u, ++i)
                if ((unsigned)(i * 1024 + (lane & ~7) * 16) < bytes)
                    __builtin_amdgcn_global_load_lds((const void*)(src + i * 1024), (LAS void*)(pk + i * 1024), 16, 0, AUX_SYS);
        };
        auto land = [&](bool steer) {
            WAIT_VM(0);
            asm volatile("" : "+v"(g));
            const unsigned now = (unsigned)__builtin_amdgcn_s_memrealtime(), lat = now - ta;
            if (pass) { passed = true; grant(myws, rank, peer, call, (unsigned)__builtin_amdgcn_readfirstlane(g), 1, lane); }
            if (!steer) return;
            if (lat < LAT_LO) win = win * 2 < WMAX ? win * 2 : WMAX;
            else if (lat > LAT_HI) { if (win > 1) win >>= 1; else if (lat > LAT_BACK) __builtin_amdgcn_s_sleep(BACKOFF); }
            const int left = GR0 + 27 + behind - (int)((now - tstart) / UNIT_TICKS);
            const int wmin = left > EG2 ? 1 : (left > EG3 ? 2 : (left > EG4 ? 3 : (left > EG4 / 2 ? 4 : 8)));
            win = win < wmin ? wmin : win;
        };
        *(LAS unsigned*)(pk + PK_DATA_END + 256 + lane * 4) = 0u;
        issue(); land(false);
        const u32x2 mm = *(LAS const u32x2*)(pk + lane * 8);
        WAIT_LGKM0();
        tstart = (unsigned)__builtin_amdgcn_s_memrealtime();
        STAMP(5);
        unsigned ad[64];
        unsigned base = 512;
        const unsigned zero = PK_DATA_END + 256 + lane * 4;
#pragma unroll
        for (int q = 0; q < 2; ++q) {
            const bool fly = permit && i < nld;
            if (fly) issue();
#pragma unroll
            for (int k = q * 32; k < (q + 1) * 32; ++k) {
                const bool nz = (int)((k < 32 ? mm.x : mm.y) << (k & 31)) < 0;
                const unsigned long long m = __ballot(nz);
                const unsigned pos = __builtin_amdgcn_mbcnt_hi((unsigned)(m >> 32), __builtin_amdgcn_mbcnt_lo((unsigned)m, 0u)) * 4 + base;
                base += (unsigned)__builtin_popcountll(m) * 4;
                ad[k] = (unsigned)(size_t)pk + (nz ? pos : zero);
                asm volatile("" : "+v"(ad[k]));
            }
            if (fly) land(true);
        }
        if (!permit && i < nld) {
            for (;;) {
                unsigned f = 0;
                if (lane == 0) f = __hip_atomic_load((const unsigned*)(myws + OFF_FD + tile * 128 + 64 + rot * 8 + 4), __ATOMIC_RELAXED, __HIP_MEMORY_SCOPE_SYSTEM);
                if ((unsigned)__builtin_amdgcn_readfirstlane(f) >= call || gave_up(t0, polls, call, myws + OFF_TMO)) break;
                __builtin_amdgcn_s_sleep(GATE_SLEEP);
            }
            tstart = (unsigned)__builtin_amdgcn_s_memrealtime();
            STAMP(5);
            if (tJ < START2) win = 2;
        }
        while (i < nld) { issue(); land(true); }
        __builtin_amdgcn_s_waitcnt(0x0F70);
        STAMP(6);
#pragma unroll
        for (int r = 0; r < 16; ++r) {
            f32x4 v;
#pragma unroll
            for (int c = 0; c < 4; ++c) v[c] = *(LAS const float*)(size_t)ad[r * 4 + c];
            __builtin_amdgcn_raw_buffer_store_b128(__builtin_bit_cast(u32x4, v), ro, off0 + (unsigned)((r & 3) + 8 * (r >> 2)) * (GN * 4), 0, AUX_SYS);
        }
        STAMP_FLUSH();
    }
}

}

extern "C" void kernel_launch(int rank, int nranks, ncclComm_t comm, hipStream_t stream,
                              void* const* d_in, const int* in_sizes, int n_in,
                              void* d_out, int out_size, const int* in_dtypes, int out_dtype,
                              void* const* d_out_all, void* const* d_ws_all, size_t ws_bytes) {
    (void)comm; (void)in_dtypes; (void)out_dtype; (void)d_out_all;
    if (nranks != NR || n_in != 4 || in_sizes[0] != GM * KSH || in_sizes[1] != KSH * GN || out_size != MSH * GN ||
        ws_bytes < WS_TOTAL || (unsigned)rank >= (unsigned)NR) {
        fprintf(stderr, "kernel_launch: unexpected shapes\n");
        return;
    }
    Params p;
    p.x = (const uint8_t*)d_in[0]; p.w = (const uint8_t*)d_in[1];
    p.sx = (const float*)d_in[2]; p.sw = (const float*)d_in[3];
    p.out = (uint8_t*)d_out;
    for (int i = 0; i < NR; ++i) p.ws[i] = (uint8_t*)d_ws_all[i];
    p.rank = rank; p.pad = 0; p.myws = p.ws[rank];
    HIP_CHECK(hipFuncSetAttribute((const void*)gemm_xchg, hipFuncAttributeMaxDynamicSharedMemorySize, LDS_BYTES));
    hipLaunchKernelGGL(gemm_xchg, dim3(NWG), dim3(NTHR), LDS_BYTES, stream, p);
    HIP_CHECK(hipGetLastError());
}
```

```cpp
#include <hip/hip_runtime.h>
#include <rccl/rccl.h>
#include <cstdio>
#include <cstdint>

#define HIP_CHECK(expr) do { hipError_t _e = (expr); if (_e != hipSuccess) { \
    fprintf(stderr, "HIP error %s at %s:%d\n", hipGetErrorString(_e), __FILE__, __LINE__); \
    return; } } while (0)

namespace {

typedef float f32x4 __attribute__((ext_vector_type(4)));
typedef float f32x16 __attribute__((ext_vector_type(16)));
typedef int i32x4 __attribute__((ext_vector_type(4)));
typedef int i32x8 __attribute__((ext_vector_type(8)));
typedef unsigned u32x2 __attribute__((ext_vector_type(2)));
typedef unsigned u32x4 __attribute__((ext_vector_type(4)));
#define LAS __attribute__((address_space(3)))

constexpr int NR = 8;
constexpr int GM = 4096, GK = 4096, GN = 16384;
constexpr int KSH = GK / NR;
constexpr int MSH = GM / NR;
constexpr int NWG = 256;
constexpr int NTHR = 512;
constexpr int NCOMM = 32;
constexpr int PB = 4096;

constexpr unsigned OFF_FX = 0;
constexpr unsigned OFF_FW = 16384;
constexpr unsigned OFF_FO = 32768;
constexpr unsigned OFF_FS = 40960;
constexpr unsigned OFF_XC = 34816;
constexpr unsigned OFF_TMO = 37120;
constexpr unsigned OFF_TK = 38912;
constexpr unsigned FLAG_BYTES = 65536;
constexpr unsigned OFF_FD = 65536;
constexpr unsigned OFF_CNT = OFF_FD + NWG * 128;
constexpr unsigned OFF_SX = 1u << 20;
constexpr unsigned OFF_SW = OFF_SX + 128 * 16384;
constexpr unsigned S_COPY = 128 * 16384 + 128 * 65536;
constexpr unsigned OFF_XS = OFF_SX + 2 * S_COPY;
constexpr unsigned OFF_WS = OFF_XS + 80 * 131072;
constexpr unsigned PK_SLOT = 17408;
constexpr unsigned PK_DATA_END = 512 + 16384;
constexpr unsigned OFF_PK = OFF_WS + 40 * 524288;
constexpr unsigned WS_TOTAL = OFF_PK + NWG * 8 * PK_SLOT;
constexpr unsigned OUT_BYTES = (unsigned)MSH * GN * 4;
constexpr unsigned OOB = 0x40000000u;
static_assert(WS_TOTAL < OOB, "");

constexpr int NS = 6;
constexpr int SLOT = 24576;
constexpr int LDS_BYTES = NS * SLOT + 64;
constexpr unsigned long long DEADLINE_TICKS = 50000000ull;
constexpr int AUX_SYS = 17;

constexpr int POLL_SLEEP = 8, POLL_SLOW = 32, RX_SLEEP = 8, GATE_SLEEP = 16, MON_SLEEP = 4;
constexpr int STAGE_DELAY = 32, STAGE_DELAY2 = 64, PH0_DELAY = 127;
constexpr int XMON = 100;
constexpr int NXC = 4;
constexpr int ROT = 3;
constexpr int PUSH_WIN = 2;
constexpr int LEAD = 8;
constexpr int JC = 8;
constexpr int GR0 = 40;
constexpr int GRANT_EARLY = 16, CHAIN_EARLY = 2, START2 = 8;
constexpr unsigned LAT_LO = 85, LAT_HI = 110, LAT_BACK = 170;
constexpr int BACKOFF = 16, WMAX = 8;
constexpr int UNIT_TICKS = 13, EG2 = 56, EG3 = 36, EG4 = 24;

struct Params {
    const uint8_t* x;
    const uint8_t* w;
    uint8_t* myws;
    int rank, pad;
    const float* sx;
    const float* sw;
    uint8_t* out;
    uint8_t* ws[NR];
};

#define WAIT_VM(n) asm volatile("s_waitcnt vmcnt(%0)" :: "n"(n) : "memory")
#define WAIT_LGKM0() asm volatile("s_waitcnt lgkmcnt(0)" ::: "memory")
#define BAR() do { asm volatile("" ::: "memory"); __builtin_amdgcn_s_barrier(); asm volatile("" ::: "memory"); } while (0)

__device__ __forceinline__ __amdgpu_buffer_rsrc_t make_rsrc(const void* p, unsigned bytes) {
    return __builtin_amdgcn_make_buffer_rsrc((void*)p, 0, (int)bytes, 0x00020000);
}
__device__ __forceinline__ void flag_store(uint8_t* p, unsigned v) {
    __hip_atomic_store((unsigned*)p, v, __ATOMIC_RELAXED, __HIP_MEMORY_SCOPE_SYSTEM);
}
__device__ __forceinline__ bool gave_up(unsigned long long t0, unsigned& polls, unsigned call, uint8_t* tmo) {
    if ((++polls & 63u) != 0u) return false;
    const unsigned t = __builtin_amdgcn_readfirstlane(__hip_atomic_load((const unsigned*)tmo, __ATOMIC_RELAXED, __HIP_MEMORY_SCOPE_AGENT));
    if (t == call || __builtin_amdgcn_s_memrealtime() - t0 > DEADLINE_TICKS) {
        __hip_atomic_store((unsigned*)tmo, call, __ATOMIC_RELAXED, __HIP_MEMORY_SCOPE_AGENT);
        return true;
    }
    return false;
}
__device__ __forceinline__ bool all_ge(u32x4 v, unsigned c) { return v.x >= c && v.y >= c && v.z >= c && v.w >= c; }

__device__ __forceinline__ int partner(int p, int m) { return p < 4 ? 4 + ((p + m) & 3) : ((p - m) & 3); }
__device__ __forceinline__ int gcb_of(int p, int J) {
    const int q = partner(p, J & 3), g = p < 4 ? p * 4 + (q & 3) : q * 4 + (p & 3);
    return g * 8 + (J >> 2);
}
__device__ __forceinline__ int grg_of(int p, int i, int e) {
    if (e < 2) return p * 16 + 2 * i + e;
    if (e < 6) return partner(p, e - 2) * 16 + i;
    if (i < 7) return ((p & 4) | ((p + 1 + e - 6) & 3)) * 16 + 9 + i;
    return partner(p, e - 6) * 16 + 8;
}
__device__ __forceinline__ int entry_of(int role, int i, int J) {
    if (role < 2) return role;
    if (role < 5) return 2 + ((J + role - 1) & 3);
    return i < 7 ? role + 1 : 6 + ((J + role - 4) & 3);
}
__device__ __forceinline__ int mirror(int role, int i) { return role >= 5 && i < 7 ? 12 - role : role; }

__device__ __forceinline__ unsigned ws_block(int J, unsigned call) { return OFF_WS + (unsigned)(J < 8 && (call & 1u) ? J + 32 : J) * 524288u; }

__device__ __forceinline__ bool wait_operands(uint8_t* ws, int i, int J, int lane, unsigned call, int ph, int rank) {
    const unsigned long long t0 = __builtin_amdgcn_s_memrealtime();
    const __amdgpu_buffer_rsrc_t rf = make_rsrc(ws, FLAG_BYTES);
    unsigned polls = 0;
    const unsigned wbase = OFF_FW + (J * 2 + ph - 1) * 256;
    if (ph != 0) {
        for (;;) {
            unsigned v = call;
            if (lane < 8 && lane != rank) v = __builtin_amdgcn_raw_buffer_load_b32(rf, wbase + lane * 32 + i * 4, 0, AUX_SYS);
            if (__builtin_popcountll(__ballot(v >= call)) >= 57 + 2) break;
            if (gave_up(t0, polls, call, ws + OFF_TMO)) return false;
            __builtin_amdgcn_s_sleep(POLL_SLOW);
            asm volatile("" ::: "memory");
        }
    }
    const int d0 = (lane & 7) * 4, nv = i < 7 ? 18 : 20;
    const bool xneed = ph == 0 || ((lane >> 3) != rank && d0 < nv);
    const bool xhalf = ph != 0 && d0 + 2 >= nv;
    const unsigned xoff = (ph == 0 ? OFF_FO : OFF_FX + (i * 2 + ph - 1) * 1024) + lane * 16;
    const bool wneed = ph != 0 && lane < 16 && (lane >> 1) != rank;
    bool xok = false, wok = false;
    for (;;) {
        u32x4 v = {call, call, call, call}, w = v;
        if (xneed && !xok) v = __builtin_amdgcn_raw_buffer_load_b128(rf, xoff, 0, AUX_SYS);
        if (wneed && !wok) w = __builtin_amdgcn_raw_buffer_load_b128(rf, wbase + lane * 16, 0, AUX_SYS);
        if (xhalf) { v.z = call; v.w = call; }
        xok = __all(all_ge(v, call)); wok = __all(all_ge(w, call));
        if (wok && xok) return true;
        if (gave_up(t0, polls, call, ws + OFF_TMO)) return false;
        if (xok) __builtin_amdgcn_s_sleep(POLL_SLEEP); else __builtin_amdgcn_s_sleep(POLL_SLOW);
        asm volatile("" ::: "memory");
    }
}

__device__ __forceinline__ void grant(uint8_t* ws, int rank, int peer, unsigned call, unsigned first, int ng, int lane) {
    const bool same = ((rank ^ peer) & 4) == 0;
    const unsigned nu = same ? 224u : 216u;
    unsigned n = first - (call - 1u) * (GR0 + nu) + (unsigned)lane;
    const unsigned self = same ? 28u : 27u;
    if (n >= self * (JC / 4)) n += self;
    if (lane < ng && n < nu) {
        unsigned tile, role;
        if (same) { tile = (n / 7u) * 8u + n % 7u; role = 5u + ((unsigned)(peer - rank - 1) & 3u); }
        else {
            const unsigned m = (rank < 4 ? (unsigned)(peer - rank) : (unsigned)(rank - peer)) & 3u;
            const unsigned c = n / 9u, u = n % 9u, tt = c % 3u, t = tt < m ? tt : tt + 1u;
            tile = ((c / 3u) * 4u + t) * 8u + (u < 8u ? u : 7u);
            role = (u < 8u ? 2u : 5u) + ((m - t - 1u) & 3u);
        }
        flag_store(ws + OFF_FD + tile * 128 + 64 + role * 8 + 4, call);
    }
}

struct Frags { i32x4 a[2]; i32x4 b[4][2]; };
__device__ __forceinline__ void read_frags(Frags& f, LAS const uint8_t* pa, LAS const uint8_t* pb) {
    f.a[0] = *(LAS const i32x4*)(pa); f.a[1] = *(LAS const i32x4*)(pa + 1024);
#pragma unroll
    for (int c = 0; c < 4; ++c) { f.b[c][0] = *(LAS const i32x4*)(pb + c * 2048); f.b[c][1] = *(LAS const i32x4*)(pb + c * 2048 + 1024); }
}
__device__ __forceinline__ i32x8 cat(i32x4 a, i32x4 b) { return __builtin_shufflevector(a, b, 0, 1, 2, 3, 4, 5, 6, 7); }
template <int C0, int C1> __device__ __forceinline__ void mma(f32x16 (&acc)[4], const Frags& f) {
    const i32x8 a = cat(f.a[0], f.a[1]);
#pragma unroll
    for (int c = C0; c < C1; ++c)
        acc[c] = __builtin_amdgcn_mfma_scale_f32_32x32x64_f8f6f4(a, cat(f.b[c][0], f.b[c][1]), acc[c], 0, 0, 0, 127, 0, 127);
}

constexpr int XPC = 146;
constexpr int NPC = 2 * XPC + (32 + 28) * 8;
struct Piece { unsigned so, dso, fo; int staged; };
__device__ __forceinline__ Piece piece_of(int pc, int rank, int peer, unsigned call) {
    Piece r;
    const unsigned cp = (call & 1u) * S_COPY;
    constexpr int E = (LEAD - 4) * 8, NP2 = 2 * (32 - LEAD);
    int n = -1, half = 0, V = 0;
    if (pc < XPC) n = pc;
    else if (pc < XPC + E) V = pc - XPC;
    else if (pc < 2 * XPC + E) { n = pc - XPC - E; half = 1; }
    else V = pc - 2 * XPC;
    if (n >= 0) {
        int i = n / 18; i = i < 7 ? i : 7;
        const int rem = n - 18 * i, e = rem >> 1;
        const unsigned o = (unsigned)(half * 2 + (rem & 1)) * PB;
        r.so = OFF_SX + cp + (unsigned)grg_of(rank, i, e) * 16384u + o;
        r.dso = OFF_XS + (unsigned)(i * 10 + e) * 131072u + (unsigned)peer * 16384u + o;
        r.fo = OFF_FX + (unsigned)((i * 2 + half) * 8 + peer) * 128u + (unsigned)rem * 4u;
        r.staged = half ^ 1;
    } else {
        const int k = V & 7, U = (V >> 3) - (LEAD - 4);
        int J;
        if (U < 0) { half = 0; J = LEAD + U; }
        else if (U < NP2) { half = (U & 1) ^ 1; J = (U >> 1) + (U & 1) * LEAD; }
        else { half = 1; J = U - NP2 + 32 - LEAD; }
        const unsigned o = (unsigned)(half * 8 + k) * PB;
        r.so = OFF_SW + cp + (unsigned)gcb_of(rank, J) * 65536u + o;
        r.dso = ws_block(J, call) + (unsigned)peer * 65536u + o;
        r.fo = OFF_FW + (unsigned)((J * 2 + half) * 8 + peer) * 32u + (unsigned)k * 4u;
        r.staged = 0;
    }
    return r;
}

__global__ void __launch_bounds__(NTHR, 2) gemm_xchg(const Params p) {
    extern __shared__ __attribute__((aligned(16))) uint8_t smem_raw[];
    LAS uint8_t* lds = (LAS uint8_t*)smem_raw;
    const int tid = threadIdx.x, lane = tid & 63, wv = __builtin_amdgcn_readfirstlane(tid >> 6);
    const int rank = p.rank;
    uint8_t* myws = p.myws;
    const int b = blockIdx.x, xcd = b & 7, slot = b >> 3;
    const bool comm = slot >= 28;
    const int rot = (wv + ROT * slot) & 7;
    unsigned* cntp = (unsigned*)(myws + OFF_CNT + b * 128);
    const unsigned cnt_v = __hip_atomic_load(cntp, __ATOMIC_RELAXED, __HIP_MEMORY_SCOPE_AGENT);
    LAS unsigned* ldsflag = (LAS unsigned*)(lds + NS * SLOT);
    if (tid < 2 * NR) *(LAS unsigned*)(lds + NR * PK_SLOT + tid * 4) = 0u;
    if (tid < 2) ldsflag[tid] = 0u;
    const __amdgpu_buffer_rsrc_t rm = make_rsrc(myws, WS_TOTAL);
    unsigned call = 0;
    uint8_t* const lanews = p.ws[lane & 7];

    {
        const int l = lane & 31, kg = lane >> 5;
        if (slot < 16) {
            const uint8_t* src = p.x + (size_t)(b * 32 + l) * KSH + wv * 32 + kg * 16;
            const u32x4 v0 = *(const u32x4*)src, v1 = *(const u32x4*)(src + 256);
            call = __builtin_amdgcn_readfirstlane(cnt_v) + 1u;
            const unsigned o = OFF_SX + (call & 1u) * S_COPY + b * 16384 + wv * 1024 + lane * 16;
            __builtin_amdgcn_raw_buffer_store_b128(v0, rm, o, 0, AUX_SYS);
            WAIT_VM(0);
            BAR();
            if (wv == 0) {
                unsigned t = 0;
                if (lane == 0) t = __hip_atomic_fetch_add((unsigned*)(myws + OFF_XC + (b & (NXC - 1)) * 128), 1u, __ATOMIC_RELAXED, __HIP_MEMORY_SCOPE_AGENT);
                if ((unsigned)__builtin_amdgcn_readfirstlane(t) == call * (128u / NXC) - 1u && lane < NR)
                    flag_store(lanews + OFF_FS + rank * 1024 + 128 + (b & (NXC - 1)) * 128, call);
            }
            __builtin_amdgcn_s_sleep(STAGE_DELAY);
            __builtin_amdgcn_raw_buffer_store_b128(v1, rm, o + 8192, 0, AUX_SYS);
        }
        if (slot >= 20 && slot < 28) __builtin_amdgcn_s_sleep(STAGE_DELAY2);
        const bool pusher = slot >= 16 && slot < 20;
        for (int job = pusher ? 0 : 1; job < (pusher && b >= 144 ? 1 : 2); ++job) {
            const bool push = job == 0;
            int gcb, half;
            if (push) { gcb = (b & 15) * 8; half = 0; }
            else { const int n = b < 144 ? b : b - 16, g = n / 15, r = n - g * 15; gcb = g * 8 + (r < 8 ? r : r - 7); half = r < 8 ? 1 : 0; }
            const int sl = 4 * half + (wv >> 1), h = wv & 1;
            const uint8_t* wsrc = p.w + (size_t)(sl * 64 + h * 32 + (lane >> 3)) * GN + gcb * 128 + (lane & 7) * 16;
#pragma unroll
            for (int u = 0; u < 4; ++u)
                __builtin_amdgcn_global_load_lds((const void*)(wsrc + (size_t)u * 8 * GN), (LAS void*)(lds + wv * 4096 + u * 1024), 16, 0, 0);
            const int ha = gcb >> 5, hb = 4 + ((gcb >> 3) & 3), Jh = (hb - ha) & 3;
            const int dst = push ? ((b & 16) ? hb : ha) : rank;
            uint8_t* wd = p.ws[dst];
            WAIT_VM(0);
            unsigned d[16];
#pragma unroll
            for (int kk = 0; kk < 16; ++kk) d[kk] = *(LAS const unsigned*)(lds + wv * 4096 + (kg * 16 + kk) * 128 + l * 4);
            u32x4 o[4];
#pragma unroll
            for (int m = 0; m < 4; ++m) {
                const unsigned d0 = d[4 * m], d1 = d[4 * m + 1], d2 = d[4 * m + 2], d3 = d[4 * m + 3];
                const unsigned t0 = __builtin_amdgcn_perm(d1, d0, 0x05010400u), t1 = __builtin_amdgcn_perm(d1, d0, 0x07030602u);
                const unsigned t2 = __builtin_amdgcn_perm(d3, d2, 0x05010400u), t3 = __builtin_amdgcn_perm(d3, d2, 0x07030602u);
                o[0][m] = __builtin_amdgcn_perm(t2, t0, 0x05040100u);
                o[1][m] = __builtin_amdgcn_perm(t2, t0, 0x07060302u);
                o[2][m] = __builtin_amdgcn_perm(t3, t1, 0x05040100u);
                o[3][m] = __builtin_amdgcn_perm(t3, t1, 0x07060302u);
            }
            call = __builtin_amdgcn_readfirstlane(cnt_v) + 1u;
            const unsigned off = (dst == rank ? OFF_SW + (call & 1u) * S_COPY + (unsigned)(gcb * 8 + sl) * 8192u
                                              : ws_block(Jh, call) + (unsigned)(rank * 8 + sl) * 8192u) + h * 1024 + lane * 16;
            const __amdgpu_buffer_rsrc_t rd = make_rsrc(wd, WS_TOTAL);
#pragma unroll
            for (int c = 0; c < 4; ++c) {
                if (c >= PUSH_WIN && dst != rank) WAIT_VM(PUSH_WIN - 1);
                __builtin_amdgcn_raw_buffer_store_b128(o[c], rd, off + c * 2048, 0, AUX_SYS);
            }
            if (push) {
                WAIT_VM(0);
                BAR();
                if (wv == 0 && lane < 8) flag_store(wd + OFF_FW + (Jh * 2 * 8 + rank) * 32 + lane * 4, call);
            }
        }
        WAIT_VM(0);
        WAIT_LGKM0();
        BAR();
        if (rot == rank) {
            if (lane == 0) flag_store(myws + OFF_FO + b * 4, call);
            if (b == XMON) {
                const unsigned long long t0 = __builtin_amdgcn_s_memrealtime();
                unsigned polls = 0;
                for (;;) {
                    const u32x4 v = __builtin_amdgcn_raw_buffer_load_b128(rm, OFF_FO + lane * 16, 0, AUX_SYS);
                    if (__all(all_ge(v, call))) {
                        if (lane < NR) flag_store(lanews + OFF_FS + rank * 1024, call);
                        break;
                    }
                    if (gave_up(t0, polls, call, myws + OFF_TMO)) break;
                    __builtin_amdgcn_s_sleep(16);
                    asm volatile("" ::: "memory");
                }
            }
        }
    }

    if (comm) {
        const int pk_ = b - 224;
        volatile LAS unsigned* fs = (volatile LAS unsigned*)(lds + NR * PK_SLOT);
        const unsigned long long t0 = __builtin_amdgcn_s_memrealtime();
        unsigned polls = 0;
        if (rot == rank) {
            for (;;) {
                unsigned v = call;
                if ((lane & 7) <= NXC) v = __hip_atomic_load((const unsigned*)(myws + OFF_FS + lane * 128), __ATOMIC_RELAXED, __HIP_MEMORY_SCOPE_SYSTEM);
                if ((__ballot(v < call) & 0xFEFEFEFEFEFEFEFEull) == 0 && (lane & 7) < 2) fs[(lane >> 3) * 2 + (lane & 1)] = v;
                if (__all(v >= call)) break;
                if (gave_up(t0, polls, call, myws + OFF_TMO)) { if (lane == 0) *(volatile LAS unsigned*)(ldsflag + 1) = 1u; break; }
                __builtin_amdgcn_s_sleep(MON_SLEEP);
            }
        } else {
            const int peer = rot;
            const __amdgpu_buffer_rsrc_t rs = make_rsrc(p.ws[peer], WS_TOTAL);
            unsigned dead = 0;
            auto wait_staged = [&](int idx) {
                while ((unsigned)__builtin_amdgcn_readfirstlane(fs[peer * 2 + idx]) < call) {
                    if (__builtin_amdgcn_readfirstlane(*(volatile LAS unsigned*)(ldsflag + 1)) != 0u) { dead = OOB; break; }
                    __builtin_amdgcn_s_sleep(1);
                }
                asm volatile("" ::: "memory");
            };
            const unsigned lo = lane * 16, one = lane == 0 ? 0u : OOB;
            const unsigned tkbase = (call - 1u) * NPC - NCOMM;
            Piece cur = piece_of(pk_, rank, peer, call);
            unsigned pfo = OOB;
            u32x4 v[2];
            wait_staged(cur.staged);
#pragma unroll
            for (int u = 0; u < 2; ++u) {
                __builtin_amdgcn_raw_buffer_store_b128((u32x4){0u, 0u, 0u, 0u}, rm, OOB + lo + u * 1024, 0, AUX_SYS);
                v[u] = __builtin_amdgcn_raw_buffer_load_b128(rs, lo + u * 1024, cur.so, AUX_SYS);
                asm volatile("" ::: "memory");
            }
            int tk = __builtin_amdgcn_raw_ptr_buffer_atomic_add_i32(1, rm, one + OFF_TK + peer * 256, 0, 0);
            bool granted = false;
            for (;;) {
                Piece nxt = cur;
                bool more = false;
                unsigned lo2 = OOB + lo, tko = OOB;
#pragma unroll
                for (int u = 0; u < 4; ++u) {
                    __builtin_amdgcn_raw_buffer_store_b128(v[u & 1], rm, lo + u * 1024, cur.dso, AUX_SYS);
                    if (u < 2) v[u & 1] = __builtin_amdgcn_raw_buffer_load_b128(rs, lo + (u + 2) * 1024, cur.so, AUX_SYS);
                    else v[u & 1] = __builtin_amdgcn_raw_buffer_load_b128(rs, lo2 + (u - 2) * 1024, nxt.so, AUX_SYS);
                    if (u == 0) {
                        WAIT_VM(4);
                        __builtin_amdgcn_raw_buffer_store_b32(call, rm, one | pfo | dead, 0, AUX_SYS);
                    }
                    if (u == 1) {
                        const unsigned pc = (unsigned)__builtin_amdgcn_readfirstlane(tk) - tkbase;
                        more = pc < (unsigned)NPC;
                        if (more) { nxt = piece_of((int)pc, rank, peer, call); wait_staged(nxt.staged); lo2 = lo; tko = one; }
                        if (pc + GRANT_EARLY >= (unsigned)NPC && !granted) {
                            granted = true;
                            const int ng = GR0 / NCOMM + (pk_ < GR0 % NCOMM ? 1 : 0);
                            const int g = __builtin_amdgcn_raw_ptr_buffer_atomic_add_i32(ng, rm, one + OFF_TK + peer * 256 + 128, 0, 0);
                            grant(myws, rank, peer, call, (unsigned)__builtin_amdgcn_readfirstlane(g), ng, lane);
                        }
                    }
                    if (u == 3) tk = __builtin_amdgcn_raw_ptr_buffer_atomic_add_i32(1, rm, tko + OFF_TK + peer * 256, 0, 0);
                    asm volatile("" ::: "memory");
                }
                pfo = cur.fo;
                cur = nxt;
                if (!more) break;
            }
            WAIT_VM(0);
            __builtin_amdgcn_raw_buffer_store_b32(call, rm, one | pfo | dead, 0, AUX_SYS);
        }
    }
    const float qnan = __builtin_nanf("");
    float sxw = p.sx[0] * p.sw[0];
    asm volatile("" : "+v"(sxw));
    const __amdgpu_buffer_rsrc_t ro = make_rsrc(p.out, OUT_BYTES);
    LAS uint8_t* pk = lds + wv * PK_SLOT;
    const int ti = 2 * (xcd & 3) + (slot & 1);
    int tJ = 2 * (slot >> 1) + (xcd >> 2);
    for (int job = comm ? 2 : (slot < 4 ? 0 : 1); job < 2; ++job) {
    f32x16 acc[4];
#pragma unroll
    for (int c = 0; c < 4; ++c)
#pragma unroll
        for (int r = 0; r < 16; ++r) acc[c][r] = 0.f;
    const int ent = entry_of(wv, ti, tJ), grg = grg_of(rank, ti, ent), gcb = gcb_of(rank, tJ);
    for (int ph = 0; ph < 3; ++ph) {
        if (wv == 0) {
            if (ph == 0 && tJ < 28) __builtin_amdgcn_s_sleep(PH0_DELAY);
            const bool ok = wait_operands(myws, ti, tJ, lane, call, ph, rank);
            if (ph == 1 && tJ < 8) __builtin_amdgcn_fence(__ATOMIC_ACQUIRE, "");
            else __builtin_amdgcn_fence(__ATOMIC_ACQUIRE, "agent");
            if (lane == 0 && !ok) *ldsflag = 1u;
            asm volatile("s_waitcnt vmcnt(0)" ::: "memory");
        }
        WAIT_LGKM0();
        BAR();
        const int nstep = ph == 0 ? 8 : 28;
        const int sh = ph == 0 ? 3 : 2, sbase = ph == 0 ? 0 : rank + 1, sadd = ph == 2 ? 4 : 0;
        const uint8_t* ga = myws + (ph == 0 ? OFF_SX + (call & 1u) * S_COPY + (unsigned)grg * 16384u : OFF_XS + (unsigned)(ti * 10 + ent) * 131072u) + lane * 16;
        const uint8_t* gb = myws + (ph == 0 ? OFF_SW + (call & 1u) * S_COPY + (unsigned)gcb * 65536u : ws_block(tJ, call)) + wv * 1024 + lane * 16;
        auto dma = [&](int n, int sl) {
            const unsigned s = (unsigned)((((sbase + (n >> sh)) & 7) << 3) + (n & ((1 << sh) - 1)) + sadd);
            LAS uint8_t* d = lds + sl * SLOT;
            __builtin_amdgcn_global_load_lds((const void*)(ga + s * 2048u), (LAS void*)(d + wv * 2048), 16, 0, 0);
            __builtin_amdgcn_global_load_lds((const void*)(ga + s * 2048u + 1024), (LAS void*)(d + wv * 2048 + 1024), 16, 0, 0);
            __builtin_amdgcn_global_load_lds((const void*)(gb + s * 8192u), (LAS void*)(d + 16384 + wv * 1024), 16, 0, 0);
        };
        LAS const uint8_t* pa = lds + wv * 2048 + lane * 16;
        LAS const uint8_t* pb = lds + 16384 + lane * 16;
#pragma unroll
        for (int s = 0; s < NS; ++s) dma(s, s);
        WAIT_VM(3 * (NS - 1));
        BAR();
        Frags f0, f1;
        read_frags(f0, pa, pb);
        int sl = 0;
#define SGB(mask, n) __builtin_amdgcn_sched_group_barrier(mask, n, 0)
#define KSTEP(FC, FN, S, VMN, DO_DMA) do { \
            __builtin_amdgcn_sched_barrier(0); \
            WAIT_VM(VMN); WAIT_LGKM0(); BAR(); \
            __builtin_amdgcn_sched_barrier(0); \
            mma<0, 1>(acc, FC); \
            __builtin_amdgcn_sched_barrier(0); \
            const int nsl = sl + 1 == NS ? 0 : sl + 1; \
            read_frags(FN, pa + nsl * SLOT, pb + nsl * SLOT); \
            if (DO_DMA) dma((S) + NS, sl); \
            mma<1, 4>(acc, FC); \
            SGB(0x100, 6); SGB(0x8, 1); SGB(0x100, 4); SGB(0x10, 1); \
            SGB(0x8, 1); SGB(0x10, 2); \
            SGB(0x8, 1); \
            sl = nsl; } while (0)
        for (int s = 0; s < nstep - NS; s += 2) {
            KSTEP(f0, f1, s, 3 * (NS - 2), true);
            KSTEP(f1, f0, s + 1, 3 * (NS - 2), true);
        }
        static_assert(NS == 6, "tail below is written for NS == 6");
        KSTEP(f0, f1, 0, 12, false);
        KSTEP(f1, f0, 0, 9, false);
        KSTEP(f0, f1, 0, 6, false);
        KSTEP(f1, f0, 0, 3, false);
        KSTEP(f0, f1, 0, 0, false);
        mma<0, 4>(acc, f1);
#undef KSTEP
    }
    WAIT_LGKM0();
    BAR();
    const float scale = *ldsflag != 0u ? qnan : sxw;
    if (wv < 2) {
        const unsigned off0 = (unsigned)((grg & 15) * 32 + 4 * (lane >> 5)) * (GN * 4) + (unsigned)(gcb * 128 + 4 * (lane & 31)) * 4;
#pragma unroll
        for (int r = 0; r < 16; ++r) {
            f32x4 v;
#pragma unroll
            for (int c = 0; c < 4; ++c) { const float y = acc[c][r] * scale; v[c] = y <= 0.f ? 0.f : y; }
            __builtin_amdgcn_raw_buffer_store_b128(__builtin_bit_cast(u32x4, v), ro, off0 + (unsigned)((r & 3) + 8 * (r >> 2)) * (GN * 4), 0, AUX_SYS);
        }
    } else {
        const int tile = tJ * 8 + ti;
        unsigned mk[2] = {0u, 0u}, base = 512 + (unsigned)(size_t)pk;
        const unsigned dump = (unsigned)(size_t)pk + PK_DATA_END + lane * 4;
#pragma unroll
        for (int r = 0; r < 16; ++r)
#pragma unroll
            for (int c = 0; c < 4; ++c) {
                const float y = acc[c][r] * scale;
                const bool nz = !(y <= 0.f);
                const unsigned long long m = __ballot(nz);
                const unsigned pos = __builtin_amdgcn_mbcnt_hi((unsigned)(m >> 32), __builtin_amdgcn_mbcnt_lo((unsigned)m, 0u)) * 4 + base;
                *(LAS float*)(size_t)(nz ? pos : dump) = y;
                base += (unsigned)__builtin_popcountll(m) * 4;
                asm("" : "+s"(base));
                mk[r >> 3] = mk[r >> 3] * 2 + (nz ? 1u : 0u);
            }
        base -= (unsigned)(size_t)pk;
        *(LAS u32x2*)(pk + lane * 8) = (u32x2){mk[0], mk[1]};
        const unsigned bytes = base;
        const __amdgpu_buffer_rsrc_t rp = make_rsrc(myws + OFF_PK + (unsigned)(tile * 8 + wv) * PK_SLOT, (bytes + 127u) & ~127u);
        for (unsigned o = 0; o < bytes; o += 5120) {
            u32x4 v[5];
#pragma unroll
            for (int u = 0; u < 5; ++u) v[u] = *(LAS const u32x4*)(pk + (o + u * 1024 < PK_SLOT ? o + u * 1024 : 0u) + lane * 16);
#pragma unroll
            for (int u = 0; u < 5; ++u) __builtin_amdgcn_raw_buffer_store_b128(v[u], rp, o + u * 1024 + lane * 16, 0, AUX_SYS);
        }
        WAIT_VM(0);
        if (lane == 0) __hip_atomic_store((unsigned long long*)(p.ws[(grg >> 4) & 7] + OFF_FD + tile * 128 + mirror(wv, ti) * 8),
                                          ((unsigned long long)call << 32) | bytes, __ATOMIC_RELAXED, __HIP_MEMORY_SCOPE_SYSTEM);
    }
    if (job == 0) tJ += 28;
    }
    if (comm) tJ += JC - 28; else if (tJ < JC + 4) tJ -= 4;
    if (rot < 2) {
        if (rot == 0 && lane == 0) __hip_atomic_store(cntp, call, __ATOMIC_RELAXED, __HIP_MEMORY_SCOPE_AGENT);
        return;
    }
    {
        const int tile = tJ * 8 + ti, g2 = grg_of(rank, ti, entry_of(rot, ti, tJ)), peer = (g2 >> 4) & 7;
        const unsigned off0 = (unsigned)((g2 & 15) * 32 + 4 * (lane >> 5)) * (GN * 4) + (unsigned)(gcb_of(peer, tJ) * 128 + 4 * (lane & 31)) * 4;
        bool ok = true, permit = comm;
        unsigned bytes = 0;
        const unsigned long long t0 = __builtin_amdgcn_s_memrealtime();
        unsigned polls = 0;
        const unsigned long long* fp = (const unsigned long long*)(myws + OFF_FD + tile * 128 + (lane & 1) * 64 + rot * 8);
        for (;;) {
            unsigned long long f = (unsigned long long)call << 32;
            if (lane < 2) f = __hip_atomic_load(fp, __ATOMIC_RELAXED, __HIP_MEMORY_SCOPE_SYSTEM);
            bytes = __builtin_amdgcn_readfirstlane((unsigned)f);
            const unsigned long long miss = __ballot((unsigned)(f >> 32) < call);
            permit = comm || (miss & 2) == 0;
            if ((miss & 1) == 0) break;
            if (gave_up(t0, polls, call, myws + OFF_TMO)) { ok = false; break; }
            __builtin_amdgcn_s_sleep(RX_SLEEP);
        }
        asm volatile("" ::: "memory");
        if (!ok) {
            for (int r = 0; r < 16; ++r)
                __builtin_amdgcn_raw_buffer_store_b128(__builtin_bit_cast(u32x4, (f32x4){qnan, qnan, qnan, qnan}), ro, off0 + (unsigned)((r & 3) + 8 * (r >> 2)) * (GN * 4), 0, AUX_SYS);
            return;
        }
        if (bytes > PK_DATA_END) bytes = PK_DATA_END;
        const int nld = (int)((bytes + 1023u) >> 10);
        const uint8_t* src = p.ws[peer] + OFF_PK + (unsigned)(tile * 8 + mirror(rot, ti)) * PK_SLOT + lane * 16;
        int win = 1;
        bool passed = false, pass = false;
        unsigned ta = 0, tstart = 0;
        int g = 0, i = 0;
        const int behind = (NWG - 1 - tile) * 7 / 8;
        auto issue = [&]() {
            const int n = win < nld - i ? win : nld - i;
            pass = i + n + CHAIN_EARLY >= nld && !passed;
            ta = (unsigned)__builtin_amdgcn_s_memrealtime();
            if (pass) g = __builtin_amdgcn_raw_ptr_buffer_atomic_add_i32(1, rm, (lane == 0 ? 0u : OOB) + OFF_TK + peer * 256 + 128, 0, 0);
            for (int u = 0; u < n; ++u, ++i)
                if ((unsigned)(i * 1024 + (lane & ~7) * 16) < bytes)
                    __builtin_amdgcn_global_load_lds((const void*)(src + i * 1024), (LAS void*)(pk + i * 1024), 16, 0, AUX_SYS);
        };
        auto land = [&](bool steer) {
            WAIT_VM(0);
            asm volatile("" : "+v"(g));
            const unsigned now = (unsigned)__builtin_amdgcn_s_memrealtime(), lat = now - ta;
            if (pass) { passed = true; grant(myws, rank, peer, call, (unsigned)__builtin_amdgcn_readfirstlane(g), 1, lane); }
            if (!steer) return;
            if (lat < LAT_LO) win = win * 2 < WMAX ? win * 2 : WMAX;
            else if (lat > LAT_HI) { if (win > 1) win >>= 1; else if (lat > LAT_BACK) __builtin_amdgcn_s_sleep(BACKOFF); }
            const int left = GR0 + 27 + behind - (int)((now - tstart) / UNIT_TICKS);
            const int wmin = left > EG2 ? 1 : (left > EG3 ? 2 : (left > EG4 ? 3 : (left > EG4 / 2 ? 4 : 8)));
            win = win < wmin ? wmin : win;
        };
        *(LAS unsigned*)(pk + PK_DATA_END + 256 + lane * 4) = 0u;
        issue(); land(false);
        const u32x2 mm = *(LAS const u32x2*)(pk + lane * 8);
        WAIT_LGKM0();
        tstart = (unsigned)__builtin_amdgcn_s_memrealtime();
        unsigned ad[64];
        unsigned base = 512;
        const unsigned zero = PK_DATA_END + 256 + lane * 4;
#pragma unroll
        for (int q = 0; q < 2; ++q) {
            const bool fly = permit && i < nld;
            if (fly) issue();
#pragma unroll
            for (int k = q * 32; k < (q + 1) * 32; ++k) {
                const bool nz = (int)((k < 32 ? mm.x : mm.y) << (k & 31)) < 0;
                const unsigned long long m = __ballot(nz);
                const unsigned pos = __builtin_amdgcn_mbcnt_hi((unsigned)(m >> 32), __builtin_amdgcn_mbcnt_lo((unsigned)m, 0u)) * 4 + base;
                base += (unsigned)__builtin_popcountll(m) * 4;
                ad[k] = (unsigned)(size_t)pk + (nz ? pos : zero);
                asm volatile("" : "+v"(ad[k]));
            }
            if (fly) land(true);
        }
        if (!permit && i < nld) {
            for (;;) {
                unsigned f = 0;
                if (lane == 0) f = __hip_atomic_load((const unsigned*)(myws + OFF_FD + tile * 128 + 64 + rot * 8 + 4), __ATOMIC_RELAXED, __HIP_MEMORY_SCOPE_SYSTEM);
                if ((unsigned)__builtin_amdgcn_readfirstlane(f) >= call || gave_up(t0, polls, call, myws + OFF_TMO)) break;
                __builtin_amdgcn_s_sleep(GATE_SLEEP);
            }
            tstart = (unsigned)__builtin_amdgcn_s_memrealtime();
            if (tJ < START2) win = 2;
        }
        while (i < nld) { issue(); land(true); }
        __builtin_amdgcn_s_waitcnt(0x0F70);
#pragma unroll
        for (int r = 0; r < 16; ++r) {
            f32x4 v;
#pragma unroll
            for (int c = 0; c < 4; ++c) v[c] = *(LAS const float*)(size_t)ad[r * 4 + c];
            __builtin_amdgcn_raw_buffer_store_b128(__builtin_bit_cast(u32x4, v), ro, off0 + (unsigned)((r & 3) + 8 * (r >> 2)) * (GN * 4), 0, AUX_SYS);
        }
    }
}

}

extern "C" void kernel_launch(int rank, int nranks, ncclComm_t comm, hipStream_t stream,
                              void* const* d_in, const int* in_sizes, int n_in,
                              void* d_out, int out_size, const int* in_dtypes, int out_dtype,
                              void* const* d_out_all, void* const* d_ws_all, size_t ws_bytes) {
    (void)comm; (void)in_dtypes; (void)out_dtype; (void)d_out_all;
    if (nranks != NR || n_in != 4 || in_sizes[0] != GM * KSH || in_sizes[1] != KSH * GN || out_size != MSH * GN ||
        ws_bytes < WS_TOTAL || (unsigned)rank >= (unsigned)NR) {
        fprintf(stderr, "kernel_launch: unexpected shapes\n");
        return;
    }
    Params p;
    p.x = (const uint8_t*)d_in[0]; p.w = (const uint8_t*)d_in[1];
    p.sx = (const float*)d_in[2]; p.sw = (const float*)d_in[3];
    p.out = (uint8_t*)d_out;
    for (int i = 0; i < NR; ++i) p.ws[i] = (uint8_t*)d_ws_all[i];
    p.rank = rank; p.pad = 0; p.myws = p.ws[rank];
    HIP_CHECK(hipFuncSetAttribute((const void*)gemm_xchg, hipFuncAttributeMaxDynamicSharedMemorySize, LDS_BYTES));
    hipLaunchKernelGGL(gemm_xchg, dim3(NWG), dim3(NTHR), LDS_BYTES, stream, p);
    HIP_CHECK(hipGetLastError());
}
```

```cpp
#include <hip/hip_runtime.h>
#include <rccl/rccl.h>
#include <cstdio>
#include <cstdint>

#define HIP_CHECK(expr) do { hipError_t _e = (expr); if (_e != hipSuccess) { \
    fprintf(stderr, "HIP error %s at %s:%d\n", hipGetErrorString(_e), __FILE__, __LINE__); \
    return; } } while (0)

namespace {

typedef float f32x4 __attribute__((ext_vector_type(4)));
typedef float f32x16 __attribute__((ext_vector_type(16)));
typedef int i32x4 __attribute__((ext_vector_type(4)));
typedef int i32x8 __attribute__((ext_vector_type(8)));
typedef unsigned u32x2 __attribute__((ext_vector_type(2)));
typedef unsigned u32x4 __attribute__((ext_vector_type(4)));
#define LAS __attribute__((address_space(3)))

constexpr int NR = 8;
constexpr int GM = 4096, GK = 4096, GN = 16384;
constexpr int KSH = GK / NR;
constexpr int MSH = GM / NR;
constexpr int NWG = 256;
constexpr int NTHR = 512;
constexpr int NCOMM = 32;
constexpr int PB = 4096;

constexpr unsigned OFF_FX = 0;
constexpr unsigned OFF_FW = 16384;
constexpr unsigned OFF_FO = 32768;
constexpr unsigned OFF_FS = 40960;
constexpr unsigned OFF_XC = 34816;
constexpr unsigned OFF_TMO = 37120;
constexpr unsigned OFF_TK = 38912;
constexpr unsigned FLAG_BYTES = 65536;
constexpr unsigned OFF_FD = 65536;
constexpr unsigned OFF_CNT = OFF_FD + NWG * 128;
constexpr unsigned OFF_SX = 1u << 20;
constexpr unsigned OFF_SW = OFF_SX + 128 * 16384;
constexpr unsigned S_COPY = 128 * 16384 + 128 * 65536;
constexpr unsigned OFF_XS = OFF_SX + 2 * S_COPY;
constexpr unsigned OFF_WS = OFF_XS + 80 * 131072;
constexpr unsigned PK_SLOT = 17408;
constexpr unsigned PK_DATA_END = 512 + 16384;
constexpr unsigned OFF_PK = OFF_WS + 40 * 524288;
constexpr unsigned WS_TOTAL = OFF_PK + NWG * 8 * PK_SLOT;
constexpr unsigned OUT_BYTES = (unsigned)MSH * GN * 4;
constexpr unsigned OOB = 0x40000000u;
static_assert(WS_TOTAL < OOB, "");

constexpr int NS = 6;
constexpr int SLOT = 24576;
constexpr int LDS_BYTES = NS * SLOT + 64;
constexpr unsigned long long DEADLINE_TICKS = 50000000ull;
constexpr int AUX_SYS = 17;

constexpr int POLL_SLEEP = 8, POLL_SLOW = 32, RX_SLEEP = 8, GATE_SLEEP = 16, MON_SLEEP = 4;
constexpr int STAGE_DELAY = 32, STAGE_DELAY2 = 64, PH0_DELAY = 127;
constexpr int XMON = 100;
constexpr int NXC = 4;
constexpr int ROT = 3;
constexpr int PUSH_WIN = 2;
constexpr int LEAD = 8;
constexpr int JC = 8;
constexpr int GR0 = 40;
constexpr int GRANT_EARLY = 16, CHAIN_EARLY = 2, START2 = 8;
constexpr unsigned LAT_LO = 85, LAT_HI = 110, LAT_BACK = 170;
constexpr int BACKOFF = 16, WMAX = 8;
constexpr int UNIT_TICKS = 13, EG2 = 56, EG3 = 36, EG4 = 24;

struct Params {
    const uint8_t* x;
    const uint8_t* w;
    uint8_t* myws;
    int rank, pad;
    const float* sx;
    const float* sw;
    uint8_t* out;
    uint8_t* ws[NR];
};

#define WAIT_VM(n) asm volatile("s_waitcnt vmcnt(%0)" :: "n"(n) : "memory")
#define WAIT_LGKM0() asm volatile("s_waitcnt lgkmcnt(0)" ::: "memory")
#define BAR() do { asm volatile("" ::: "memory"); __builtin_amdgcn_s_barrier(); asm volatile("" ::: "memory"); } while (0)

__device__ __forceinline__ __amdgpu_buffer_rsrc_t make_rsrc(const void* p, unsigned bytes) {
    return __builtin_amdgcn_make_buffer_rsrc((void*)p, 0, (int)bytes, 0x00020000);
}
__device__ __forceinline__ void flag_store(uint8_t* p, unsigned v) {
    __hip_atomic_store((unsigned*)p, v, __ATOMIC_RELAXED, __HIP_MEMORY_SCOPE_SYSTEM);
}
__device__ __forceinline__ bool gave_up(unsigned long long t0, unsigned& polls, unsigned call, uint8_t* tmo) {
    if ((++polls & 63u) != 0u) return false;
    const unsigned t = __builtin_amdgcn_readfirstlane(__hip_atomic_load((const unsigned*)tmo, __ATOMIC_RELAXED, __HIP_MEMORY_SCOPE_AGENT));
    if (t == call || __builtin_amdgcn_s_memrealtime() - t0 > DEADLINE_TICKS) {
        __hip_atomic_store((unsigned*)tmo, call, __ATOMIC_RELAXED, __HIP_MEMORY_SCOPE_AGENT);
        return true;
    }
    return false;
}
__device__ __forceinline__ bool all_ge(u32x4 v, unsigned c) { return v.x >= c && v.y >= c && v.z >= c && v.w >= c; }

__device__ __forceinline__ int partner(int p, int m) { return p < 4 ? 4 + ((p + m) & 3) : ((p - m) & 3); }
__device__ __forceinline__ int gcb_of(int p, int J) {
    const int q = partner(p, J & 3), g = p < 4 ? p * 4 + (q & 3) : q * 4 + (p & 3);
    return g * 8 + (J >> 2);
}
__device__ __forceinline__ int grg_of(int p, int i, int e) {
    if (e < 2) return p * 16 + 2 * i + e;
    if (e < 6) return partner(p, e - 2) * 16 + i;
    if (i < 7) return ((p & 4) | ((p + 1 + e - 6) & 3)) * 16 + 9 + i;
    return partner(p, e - 6) * 16 + 8;
}
__device__ __forceinline__ int entry_of(int role, int i, int J) {
    if (role < 2) return role;
    if (role < 5) return 2 + ((J + role - 1) & 3);
    return i < 7 ? role + 1 : 6 + ((J + role - 4) & 3);
}
__device__ __forceinline__ int mirror(int role, int i) { return role >= 5 && i < 7 ? 12 - role : role; }

__device__ __forceinline__ unsigned ws_block(int J, unsigned call) { return OFF_WS + (unsigned)(J < 8 && (call & 1u) ? J + 32 : J) * 524288u; }

__device__ __forceinline__ bool wait_operands(uint8_t* ws, int i, int J, int lane, unsigned call, int ph, int rank) {
    const unsigned long long t0 = __builtin_amdgcn_s_memrealtime();
    const __amdgpu_buffer_rsrc_t rf = make_rsrc(ws, FLAG_BYTES);
    unsigned polls = 0;
    const unsigned wbase = OFF_FW + (J * 2 + ph - 1) * 256;
    if (ph != 0) {
        for (;;) {
            unsigned v = call;
            if (lane < 8 && lane != rank) v = __builtin_amdgcn_raw_buffer_load_b32(rf, wbase + lane * 32 + i * 4, 0, AUX_SYS);
            if (__builtin_popcountll(__ballot(v >= call)) >= 57 + 2) break;
            if (gave_up(t0, polls, call, ws + OFF_TMO)) return false;
            __builtin_amdgcn_s_sleep(POLL_SLOW);
            asm volatile("" ::: "memory");
        }
    }
    const int d0 = (lane & 7) * 4, nv = i < 7 ? 18 : 20;
    const bool xneed = ph == 0 || ((lane >> 3) != rank && d0 < nv);
    const bool xhalf = ph != 0 && d0 + 2 >= nv;
    const unsigned xoff = (ph == 0 ? OFF_FO : OFF_FX + (i * 2 + ph - 1) * 1024) + lane * 16;
    const bool wneed = ph != 0 && lane < 16 && (lane >> 1) != rank;
    bool xok = false, wok = false;
    for (;;) {
        u32x4 v = {call, call, call, call}, w = v;
        if (xneed && !xok) v = __builtin_amdgcn_raw_buffer_load_b128(rf, xoff, 0, AUX_SYS);
        if (wneed && !wok) w = __builtin_amdgcn_raw_buffer_load_b128(rf, wbase + lane * 16, 0, AUX_SYS);
        if (xhalf) { v.z = call; v.w = call; }
        xok = __all(all_ge(v, call)); wok = __all(all_ge(w, call));
        if (wok && xok) return true;
        if (gave_up(t0, polls, call, ws + OFF_TMO)) return false;
        if (xok) __builtin_amdgcn_s_sleep(POLL_SLEEP); else __builtin_amdgcn_s_sleep(POLL_SLOW);
        asm volatile("" ::: "memory");
    }
}

__device__ __forceinline__ void grant(uint8_t* ws, int rank, int peer, unsigned call, unsigned first, int ng, int lane) {
    const bool same = ((rank ^ peer) & 4) == 0;
    const unsigned nu = same ? 224u : 216u;
    unsigned n = first - (call - 1u) * (GR0 + nu) + (unsigned)lane;
    const unsigned self = same ? 28u : 27u;
    if (n >= self * (JC / 4)) n += self;
    if (lane < ng && n < nu) {
        unsigned tile, role;
        if (same) { tile = (n / 7u) * 8u + n % 7u; role = 5u + ((unsigned)(peer - rank - 1) & 3u); }
        else {
            const unsigned m = (rank < 4 ? (unsigned)(peer - rank) : (unsigned)(rank - peer)) & 3u;
            const unsigned c = n / 9u, u = n % 9u, tt = c % 3u, t = tt < m ? tt : tt + 1u;
            tile = ((c / 3u) * 4u + t) * 8u + (u < 8u ? u : 7u);
            role = (u < 8u ? 2u : 5u) + ((m - t - 1u) & 3u);
        }
        flag_store(ws + OFF_FD + tile * 128 + 64 + role * 8 + 4, call);
    }
}

struct Frags { i32x4 a[2]; i32x4 b[4][2]; };
__device__ __forceinline__ void read_frags(Frags& f, LAS const uint8_t* pa, LAS const uint8_t* pb) {
    f.a[0] = *(LAS const i32x4*)(pa); f.a[1] = *(LAS const i32x4*)(pa + 1024);
#pragma unroll
    for (int c = 0; c < 4; ++c) { f.b[c][0] = *(LAS const i32x4*)(pb + c * 2048); f.b[c][1] = *(LAS const i32x4*)(pb + c * 2048 + 1024); }
}
__device__ __forceinline__ i32x8 cat(i32x4 a, i32x4 b) { return __builtin_shufflevector(a, b, 0, 1, 2, 3, 4, 5, 6, 7); }
template <int C0, int C1> __device__ __forceinline__ void mma(f32x16 (&acc)[4], const Frags& f) {
    const i32x8 a = cat(f.a[0], f.a[1]);
#pragma unroll
    for (int c = C0; c < C1; ++c)
        acc[c] = __builtin_amdgcn_mfma_scale_f32_32x32x64_f8f6f4(a, cat(f.b[c][0], f.b[c][1]), acc[c], 0, 0, 0, 127, 0, 127);
}

constexpr int XPC = 146;
constexpr int NPC = 2 * XPC + (32 + 28) * 8;
struct Piece { unsigned so, dso, fo; int staged; };
__device__ __forceinline__ Piece piece_of(int pc, int rank, int peer, unsigned call) {
    Piece r;
    const unsigned cp = (call & 1u) * S_COPY;
    constexpr int E = (LEAD - 4) * 8, NP2 = 2 * (32 - LEAD);
    int n = -1, half = 0, V = 0;
    if (pc < XPC) n = pc;
    else if (pc < XPC + E) V = pc - XPC;
    else if (pc < 2 * XPC + E) { n = pc - XPC - E; half = 1; }
    else V = pc - 2 * XPC;
    if (n >= 0) {
        int i = n / 18; i = i < 7 ? i : 7;
        const int rem = n - 18 * i, e = rem >> 1;
        const unsigned o = (unsigned)(half * 2 + (rem & 1)) * PB;
        r.so = OFF_SX + cp + (unsigned)grg_of(rank, i, e) * 16384u + o;
        r.dso = OFF_XS + (unsigned)(i * 10 + e) * 131072u + (unsigned)peer * 16384u + o;
        r.fo = OFF_FX + (unsigned)((i * 2 + half) * 8 + peer) * 128u + (unsigned)rem * 4u;
        r.staged = half ^ 1;
    } else {
        const int k = V & 7, U = (V >> 3) - (LEAD - 4);
        int J;
        if (U < 0) { half = 0; J = LEAD + U; }
        else if (U < NP2) { half = (U & 1) ^ 1; J = (U >> 1) + (U & 1) * LEAD; }
        else { half = 1; J = U - NP2 + 32 - LEAD; }
        const unsigned o = (unsigned)(half * 8 + k) * PB;
        r.so = OFF_SW + cp + (unsigned)gcb_of(rank, J) * 65536u + o;
        r.dso = ws_block(J, call) + (unsigned)peer * 65536u + o;
        r.fo = OFF_FW + (unsigned)((J * 2 + half) * 8 + peer) * 32u + (unsigned)k * 4u;
        r.staged = 0;
    }
    return r;
}

__global__ void __launch_bounds__(NTHR, 2) gemm_xchg(const Params p) {
    extern __shared__ __attribute__((aligned(16))) uint8_t smem_raw[];
    LAS uint8_t* lds = (LAS uint8_t*)smem_raw;
    const int tid = threadIdx.x, lane = tid & 63, wv = __builtin_amdgcn_readfirstlane(tid >> 6);
    const int rank = p.rank;
    uint8_t* myws = p.myws;
    const int b = blockIdx.x, xcd = b & 7, slot = b >> 3;
    const bool comm = slot >= 28;
    const int rot = (wv + ROT * slot) & 7;
    unsigned* cntp = (unsigned*)(myws + OFF_CNT + b * 128);
    const unsigned cnt_v = __hip_atomic_load(cntp, __ATOMIC_RELAXED, __HIP_MEMORY_SCOPE_AGENT);
    LAS unsigned* ldsflag = (LAS unsigned*)(lds + NS * SLOT);
    if (tid < 2 * NR) *(LAS unsigned*)(lds + NR * PK_SLOT + tid * 4) = 0u;
    if (tid < 2) ldsflag[tid] = 0u;
    const __amdgpu_buffer_rsrc_t rm = make_rsrc(myws, WS_TOTAL);
    unsigned call = 0;
    uint8_t* const lanews = p.ws[lane & 7];

    {
        const int l = lane & 31, kg = lane >> 5;
        if (slot < 16) {
            const uint8_t* src = p.x + (size_t)(b * 32 + l) * KSH + wv * 32 + kg * 16;
            const u32x4 v0 = *(const u32x4*)src;
            call = __builtin_amdgcn_readfirstlane(cnt_v) + 1u;
            const unsigned o = OFF_SX + (call & 1u) * S_COPY + b * 16384 + wv * 1024 + lane * 16;
            __builtin_amdgcn_raw_buffer_store_b128(v0, rm, o, 0, AUX_SYS);
            WAIT_VM(0);
            BAR();
            if (wv == 0) {
                unsigned t = 0;
                if (lane == 0) t = __hip_atomic_fetch_add((unsigned*)(myws + OFF_XC + (b & (NXC - 1)) * 128), 1u, __ATOMIC_RELAXED, __HIP_MEMORY_SCOPE_AGENT);
                if ((unsigned)__builtin_amdgcn_readfirstlane(t) == call * (128u / NXC) - 1u && lane < NR)
                    flag_store(lanews + OFF_FS + rank * 1024 + 128 + (b & (NXC - 1)) * 128, call);
            }
            __builtin_amdgcn_s_sleep(STAGE_DELAY);
            const u32x4 v1 = *(const u32x4*)(src + 256);
            __builtin_amdgcn_raw_buffer_store_b128(v1, rm, o + 8192, 0, AUX_SYS);
        }
        if (slot >= 20 && slot < 28) __builtin_amdgcn_s_sleep(STAGE_DELAY2);
        const bool pusher = slot >= 16 && slot < 20;
        for (int job = pusher ? 0 : 1; job < (comm || (pusher && b >= 144) ? 1 : (slot >= 24 ? 3 : 2)); ++job) {
            const bool push = job == 0;
            int gcb, half;
            if (push) { gcb = (b & 15) * 8; half = 0; }
            else { const int n = (b < 144 ? b : b - 16) + (job == 2 ? 32 : 0), g = n / 15, r = n - g * 15; gcb = g * 8 + (r < 8 ? r : r - 7); half = r < 8 ? 1 : 0; }
            const int sl = 4 * half + (wv >> 1), h = wv & 1;
            const uint8_t* wsrc = p.w + (size_t)(sl * 64 + h * 32 + (lane >> 3)) * GN + gcb * 128 + (lane & 7) * 16;
#pragma unroll
            for (int u = 0; u < 4; ++u)
                __builtin_amdgcn_global_load_lds((const void*)(wsrc + (size_t)u * 8 * GN), (LAS void*)(lds + wv * 4096 + u * 1024), 16, 0, 0);
            const int ha = gcb >> 5, hb = 4 + ((gcb >> 3) & 3), Jh = (hb - ha) & 3;
            const int dst = push ? ((b & 16) ? hb : ha) : rank;
            uint8_t* wd = p.ws[dst];
            WAIT_VM(0);
            unsigned d[16];
#pragma unroll
            for (int kk = 0; kk < 16; ++kk) d[kk] = *(LAS const unsigned*)(lds + wv * 4096 + (kg * 16 + kk) * 128 + l * 4);
            u32x4 o[4];
#pragma unroll
            for (int m = 0; m < 4; ++m) {
                const unsigned d0 = d[4 * m], d1 = d[4 * m + 1], d2 = d[4 * m + 2], d3 = d[4 * m + 3];
                const unsigned t0 = __builtin_amdgcn_perm(d1, d0, 0x05010400u), t1 = __builtin_amdgcn_perm(d1, d0, 0x07030602u);
                const unsigned t2 = __builtin_amdgcn_perm(d3, d2, 0x05010400u), t3 = __builtin_amdgcn_perm(d3, d2, 0x07030602u);
                o[0][m] = __builtin_amdgcn_perm(t2, t0, 0x05040100u);
                o[1][m] = __builtin_amdgcn_perm(t2, t0, 0x07060302u);
                o[2][m] = __builtin_amdgcn_perm(t3, t1, 0x05040100u);
                o[3][m] = __builtin_amdgcn_perm(t3, t1, 0x07060302u);
            }
            call = __builtin_amdgcn_readfirstlane(cnt_v) + 1u;
            const unsigned off = (dst == rank ? OFF_SW + (call & 1u) * S_COPY + (unsigned)(gcb * 8 + sl) * 8192u
                                              : ws_block(Jh, call) + (unsigned)(rank * 8 + sl) * 8192u) + h * 1024 + lane * 16;
            const __amdgpu_buffer_rsrc_t rd = make_rsrc(wd, WS_TOTAL);
#pragma unroll
            for (int c = 0; c < 4; ++c) {
                if (c >= PUSH_WIN && dst != rank) WAIT_VM(PUSH_WIN - 1);
                __builtin_amdgcn_raw_buffer_store_b128(o[c], rd, off + c * 2048, 0, AUX_SYS);
            }
            if (push) {
                WAIT_VM(0);
                BAR();
                if (wv == 0 && lane < 8) flag_store(wd + OFF_FW + (Jh * 2 * 8 + rank) * 32 + lane * 4, call);
            }
        }
        call = __builtin_amdgcn_readfirstlane(cnt_v) + 1u;
        WAIT_VM(0);
        WAIT_LGKM0();
        BAR();
        if (rot == rank) {
            if (lane == 0) flag_store(myws + OFF_FO + b * 4, call);
            if (b == XMON) {
                const unsigned long long t0 = __builtin_amdgcn_s_memrealtime();
                unsigned polls = 0;
                for (;;) {
                    const u32x4 v = __builtin_amdgcn_raw_buffer_load_b128(rm, OFF_FO + lane * 16, 0, AUX_SYS);
                    if (__all(all_ge(v, call))) {
                        if (lane < NR) flag_store(lanews + OFF_FS + rank * 1024, call);
                        break;
                    }
                    if (gave_up(t0, polls, call, myws + OFF_TMO)) break;
                    __builtin_amdgcn_s_sleep(16);
                    asm volatile("" ::: "memory");
                }
            }
        }
    }

    if (comm) {
        const int pk_ = b - 224;
        volatile LAS unsigned* fs = (volatile LAS unsigned*)(lds + NR * PK_SLOT);
        const unsigned long long t0 = __builtin_amdgcn_s_memrealtime();
        unsigned polls = 0;
        if (rot == rank) {
            for (;;) {
                unsigned v = call;
                if ((lane & 7) <= NXC) v = __hip_atomic_load((const unsigned*)(myws + OFF_FS + lane * 128), __ATOMIC_RELAXED, __HIP_MEMORY_SCOPE_SYSTEM);
                if ((__ballot(v < call) & 0xFEFEFEFEFEFEFEFEull) == 0 && (lane & 7) < 2) fs[(lane >> 3) * 2 + (lane & 1)] = v;
                if (__all(v >= call)) break;
                if (gave_up(t0, polls, call, myws + OFF_TMO)) { if (lane == 0) *(volatile LAS unsigned*)(ldsflag + 1) = 1u; break; }
                __builtin_amdgcn_s_sleep(MON_SLEEP);
            }
        } else {
            const int peer = rot;
            const __amdgpu_buffer_rsrc_t rs = make_rsrc(p.ws[peer], WS_TOTAL);
            unsigned dead = 0;
            auto wait_staged = [&](int idx) {
                while ((unsigned)__builtin_amdgcn_readfirstlane(fs[peer * 2 + idx]) < call) {
                    if (__builtin_amdgcn_readfirstlane(*(volatile LAS unsigned*)(ldsflag + 1)) != 0u) { dead = OOB; break; }
                    __builtin_amdgcn_s_sleep(1);
                }
                asm volatile("" ::: "memory");
            };
            const unsigned lo = lane * 16, one = lane == 0 ? 0u : OOB;
            const unsigned tkbase = (call - 1u) * NPC - NCOMM;
            Piece cur = piece_of(pk_, rank, peer, call);
            unsigned pfo = OOB;
            u32x4 v[2];
            wait_staged(cur.staged);
#pragma unroll
            for (int u = 0; u < 2; ++u) {
                __builtin_amdgcn_raw_buffer_store_b128((u32x4){0u, 0u, 0u, 0u}, rm, OOB + lo + u * 1024, 0, AUX_SYS);
                v[u] = __builtin_amdgcn_raw_buffer_load_b128(rs, lo + u * 1024, cur.so, AUX_SYS);
                asm volatile("" ::: "memory");
            }
            int tk = __builtin_amdgcn_raw_ptr_buffer_atomic_add_i32(1, rm, one + OFF_TK + peer * 256, 0, 0);
            bool granted = false;
            for (;;) {
                Piece nxt = cur;
                bool more = false;
                unsigned lo2 = OOB + lo, tko = OOB;
#pragma unroll
                for (int u = 0; u < 4; ++u) {
                    __builtin_amdgcn_raw_buffer_store_b128(v[u & 1], rm, lo + u * 1024, cur.dso, AUX_SYS);
                    if (u < 2) v[u & 1] = __builtin_amdgcn_raw_buffer_load_b128(rs, lo + (u + 2) * 1024, cur.so, AUX_SYS);
                    else v[u & 1] = __builtin_amdgcn_raw_buffer_load_b128(rs, lo2 + (u - 2) * 1024, nxt.so, AUX_SYS);
                    if (u == 0) {
                        WAIT_VM(4);
                        __builtin_amdgcn_raw_buffer_store_b32(call, rm, one | pfo | dead, 0, AUX_SYS);
                    }
                    if (u == 1) {
                        const unsigned pc = (unsigned)__builtin_amdgcn_readfirstlane(tk) - tkbase;
                        more = pc < (unsigned)NPC;
                        if (more) { nxt = piece_of((int)pc, rank, peer, call); wait_staged(nxt.staged); lo2 = lo; tko = one; }
                        if (pc + GRANT_EARLY >= (unsigned)NPC && !granted) {
                            granted = true;
                            const int ng = GR0 / NCOMM + (pk_ < GR0 % NCOMM ? 1 : 0);
                            const int g = __builtin_amdgcn_raw_ptr_buffer_atomic_add_i32(ng, rm, one + OFF_TK + peer * 256 + 128, 0, 0);
                            grant(myws, rank, peer, call, (unsigned)__builtin_amdgcn_readfirstlane(g), ng, lane);
                        }
                    }
                    if (u == 3) tk = __builtin_amdgcn_raw_ptr_buffer_atomic_add_i32(1, rm, tko + OFF_TK + peer * 256, 0, 0);
                    asm volatile("" ::: "memory");
                }
                pfo = cur.fo;
                cur = nxt;
                if (!more) break;
            }
            WAIT_VM(0);
            __builtin_amdgcn_raw_buffer_store_b32(call, rm, one | pfo | dead, 0, AUX_SYS);
        }
    }
    const float qnan = __builtin_nanf("");
    float sxw = p.sx[0] * p.sw[0];
    asm volatile("" : "+v"(sxw));
    const __amdgpu_buffer_rsrc_t ro = make_rsrc(p.out, OUT_BYTES);
    LAS uint8_t* pk = lds + wv * PK_SLOT;
    const int ti = 2 * (xcd & 3) + (slot & 1);
    int tJ = 2 * (slot >> 1) + (xcd >> 2);
    for (int job = comm ? 2 : (slot < 4 ? 0 : 1); job < 2; ++job) {
    f32x16 acc[4];
#pragma unroll
    for (int c = 0; c < 4; ++c)
#pragma unroll
        for (int r = 0; r < 16; ++r) acc[c][r] = 0.f;
    const int ent = entry_of(wv, ti, tJ), grg = grg_of(rank, ti, ent), gcb = gcb_of(rank, tJ);
    for (int ph = 0; ph < 3; ++ph) {
        if (wv == 0) {
            if (ph == 0 && tJ < 28) __builtin_amdgcn_s_sleep(PH0_DELAY);
            const bool ok = wait_operands(myws, ti, tJ, lane, call, ph, rank);
            if (ph == 1 && tJ < 8) __builtin_amdgcn_fence(__ATOMIC_ACQUIRE, "");
            else __builtin_amdgcn_fence(__ATOMIC_ACQUIRE, "agent");
            if (lane == 0 && !ok) *ldsflag = 1u;
            asm volatile("s_waitcnt vmcnt(0)" ::: "memory");
        }
        WAIT_LGKM0();
        BAR();
        const int nstep = ph == 0 ? 8 : 28;
        const int sh = ph == 0 ? 3 : 2, sbase = ph == 0 ? 0 : rank + 1, sadd = ph == 2 ? 4 : 0;
        const uint8_t* ga = myws + (ph == 0 ? OFF_SX + (call & 1u) * S_COPY + (unsigned)grg * 16384u : OFF_XS + (unsigned)(ti * 10 + ent) * 131072u) + lane * 16;
        const uint8_t* gb = myws + (ph == 0 ? OFF_SW + (call & 1u) * S_COPY + (unsigned)gcb * 65536u : ws_block(tJ, call)) + wv * 1024 + lane * 16;
        auto dma = [&](int n, int sl) {
            const unsigned s = (unsigned)((((sbase + (n >> sh)) & 7) << 3) + (n & ((1 << sh) - 1)) + sadd);
            LAS uint8_t* d = lds + sl * SLOT;
            __builtin_amdgcn_global_load_lds((const void*)(ga + s * 2048u), (LAS void*)(d + wv * 2048), 16, 0, 0);
            __builtin_amdgcn_global_load_lds((const void*)(ga + s * 2048u + 1024), (LAS void*)(d + wv * 2048 + 1024), 16, 0, 0);
            __builtin_amdgcn_global_load_lds((const void*)(gb + s * 8192u), (LAS void*)(d + 16384 + wv * 1024), 16, 0, 0);
        };
        LAS const uint8_t* pa = lds + wv * 2048 + lane * 16;
        LAS const uint8_t* pb = lds + 16384 + lane * 16;
#pragma unroll
        for (int s = 0; s < NS; ++s) dma(s, s);
        WAIT_VM(3 * (NS - 1));
        BAR();
        Frags f0, f1;
        read_frags(f0, pa, pb);
        int sl = 0;
#define SGB(mask, n) __builtin_amdgcn_sched_group_barrier(mask, n, 0)
#define KSTEP(FC, FN, S, VMN, DO_DMA) do { \
            __builtin_amdgcn_sched_barrier(0); \
            WAIT_VM(VMN); WAIT_LGKM0(); BAR(); \
            __builtin_amdgcn_sched_barrier(0); \
            mma<0, 1>(acc, FC); \
            __builtin_amdgcn_sched_barrier(0); \
            const int nsl = sl + 1 == NS ? 0 : sl + 1; \
            read_frags(FN, pa + nsl * SLOT, pb + nsl * SLOT); \
            if (DO_DMA) dma((S) + NS, sl); \
            mma<1, 4>(acc, FC); \
            SGB(0x100, 6); SGB(0x8, 1); SGB(0x100, 4); SGB(0x10, 1); \
            SGB(0x8, 1); SGB(0x10, 2); \
            SGB(0x8, 1); \
            sl = nsl; } while (0)
        for (int s = 0; s < nstep - NS; s += 2) {
            KSTEP(f0, f1, s, 3 * (NS - 2), true);
            KSTEP(f1, f0, s + 1, 3 * (NS - 2), true);
        }
        static_assert(NS == 6, "tail below is written for NS == 6");
        KSTEP(f0, f1, 0, 12, false);
        KSTEP(f1, f0, 0, 9, false);
        KSTEP(f0, f1, 0, 6, false);
        KSTEP(f1, f0, 0, 3, false);
        KSTEP(f0, f1, 0, 0, false);
        mma<0, 4>(acc, f1);
#undef KSTEP
    }
    WAIT_LGKM0();
    BAR();
    const float scale = *ldsflag != 0u ? qnan : sxw;
    if (wv < 2) {
        const unsigned off0 = (unsigned)((grg & 15) * 32 + 4 * (lane >> 5)) * (GN * 4) + (unsigned)(gcb * 128 + 4 * (lane & 31)) * 4;
#pragma unroll
        for (int r = 0; r < 16; ++r) {
            f32x4 v;
#pragma unroll
            for (int c = 0; c < 4; ++c) { const float y = acc[c][r] * scale; v[c] = y <= 0.f ? 0.f : y; }
            __builtin_amdgcn_raw_buffer_store_b128(__builtin_bit_cast(u32x4, v), ro, off0 + (unsigned)((r & 3) + 8 * (r >> 2)) * (GN * 4), 0, AUX_SYS);
        }
    } else {
        const int tile = tJ * 8 + ti;
        unsigned mk[2] = {0u, 0u}, base = 512 + (unsigned)(size_t)pk;
        const unsigned dump = (unsigned)(size_t)pk + PK_DATA_END + lane * 4;
#pragma unroll
        for (int r = 0; r < 16; ++r)
#pragma unroll
            for (int c = 0; c < 4; ++c) {
                const float y = acc[c][r] * scale;
                const bool nz = !(y <= 0.f);
                const unsigned long long m = __ballot(nz);
                const unsigned pos = __builtin_amdgcn_mbcnt_hi((unsigned)(m >> 32), __builtin_amdgcn_mbcnt_lo((unsigned)m, 0u)) * 4 + base;
                *(LAS float*)(size_t)(nz ? pos : dump) = y;
                base += (unsigned)__builtin_popcountll(m) * 4;
                asm("" : "+s"(base));
                mk[r >> 3] = mk[r >> 3] * 2 + (nz ? 1u : 0u);
            }
        base -= (unsigned)(size_t)pk;
        *(LAS u32x2*)(pk + lane * 8) = (u32x2){mk[0], mk[1]};
        const unsigned bytes = base;
        const __amdgpu_buffer_rsrc_t rp = make_rsrc(myws + OFF_PK + (unsigned)(tile * 8 + wv) * PK_SLOT, (bytes + 127u) & ~127u);
        for (unsigned o = 0; o < bytes; o += 5120) {
            u32x4 v[5];
#pragma unroll
            for (int u = 0; u < 5; ++u) v[u] = *(LAS const u32x4*)(pk + (o + u * 1024 < PK_SLOT ? o + u * 1024 : 0u) + lane * 16);
#pragma unroll
            for (int u = 0; u < 5; ++u) __builtin_amdgcn_raw_buffer_store_b128(v[u], rp, o + u * 1024 + lane * 16, 0, AUX_SYS);
        }
        WAIT_VM(0);
        if (lane == 0) __hip_atomic_store((unsigned long long*)(p.ws[(grg >> 4) & 7] + OFF_FD + tile * 128 + mirror(wv, ti) * 8),
                                          ((unsigned long long)call << 32) | bytes, __ATOMIC_RELAXED, __HIP_MEMORY_SCOPE_SYSTEM);
    }
    if (job == 0) tJ += 28;
    }
    if (comm) tJ += JC - 28; else if (tJ < JC + 4) tJ -= 4;
    if (rot < 2) {
        if (rot == 0 && lane == 0) __hip_atomic_store(cntp, call, __ATOMIC_RELAXED, __HIP_MEMORY_SCOPE_AGENT);
        return;
    }
    {
        const int tile = tJ * 8 + ti, g2 = grg_of(rank, ti, entry_of(rot, ti, tJ)), peer = (g2 >> 4) & 7;
        const unsigned off0 = (unsigned)((g2 & 15) * 32 + 4 * (lane >> 5)) * (GN * 4) + (unsigned)(gcb_of(peer, tJ) * 128 + 4 * (lane & 31)) * 4;
        bool ok = true, permit = comm;
        unsigned bytes = 0;
        const unsigned long long t0 = __builtin_amdgcn_s_memrealtime();
        unsigned polls = 0;
        const unsigned long long* fp = (const unsigned long long*)(myws + OFF_FD + tile * 128 + (lane & 1) * 64 + rot * 8);
        for (;;) {
            unsigned long long f = (unsigned long long)call << 32;
            if (lane < 2) f = __hip_atomic_load(fp, __ATOMIC_RELAXED, __HIP_MEMORY_SCOPE_SYSTEM);
            bytes = __builtin_amdgcn_readfirstlane((unsigned)f);
            const unsigned long long miss = __ballot((unsigned)(f >> 32) < call);
            permit = comm || (miss & 2) == 0;
            if ((miss & 1) == 0) break;
            if (gave_up(t0, polls, call, myws + OFF_TMO)) { ok = false; break; }
            __builtin_amdgcn_s_sleep(RX_SLEEP);
        }
        asm volatile("" ::: "memory");
        if (!ok) {
            for (int r = 0; r < 16; ++r)
                __builtin_amdgcn_raw_buffer_store_b128(__builtin_bit_cast(u32x4, (f32x4){qnan, qnan, qnan, qnan}), ro, off0 + (unsigned)((r & 3) + 8 * (r >> 2)) * (GN * 4), 0, AUX_SYS);
            return;
        }
        if (bytes > PK_DATA_END) bytes = PK_DATA_END;
        const int nld = (int)((bytes + 1023u) >> 10);
        const uint8_t* src = p.ws[peer] + OFF_PK + (unsigned)(tile * 8 + mirror(rot, ti)) * PK_SLOT + lane * 16;
        int win = 1;
        bool passed = false, pass = false;
        unsigned ta = 0, tstart = 0;
        int g = 0, i = 0;
        const int behind = (NWG - 1 - tile) * 7 / 8;
        auto issue = [&]() {
            const int n = win < nld - i ? win : nld - i;
            pass = i + n + CHAIN_EARLY >= nld && !passed;
            ta = (unsigned)__builtin_amdgcn_s_memrealtime();
            if (pass) g = __builtin_amdgcn_raw_ptr_buffer_atomic_add_i32(1, rm, (lane == 0 ? 0u : OOB) + OFF_TK + peer * 256 + 128, 0, 0);
            for (int u = 0; u < n; ++u, ++i)
                if ((unsigned)(i * 1024 + (lane & ~7) * 16) < bytes)
                    __builtin_amdgcn_global_load_lds((const void*)(src + i * 1024), (LAS void*)(pk + i * 1024), 16, 0, AUX_SYS);
        };
        auto land = [&](bool steer) {
            WAIT_VM(0);
            asm volatile("" : "+v"(g));
            const unsigned now = (unsigned)__builtin_amdgcn_s_memrealtime(), lat = now - ta;
            if (pass) { passed = true; grant(myws, rank, peer, call, (unsigned)__builtin_amdgcn_readfirstlane(g), 1, lane); }
            if (!steer) return;
            if (lat < LAT_LO) win = win * 2 < WMAX ? win * 2 : WMAX;
            else if (lat > LAT_HI) { if (win > 1) win >>= 1; else if (lat > LAT_BACK) __builtin_amdgcn_s_sleep(BACKOFF); }
            const int left = GR0 + 27 + behind - (int)((now - tstart) / UNIT_TICKS);
            const int wmin = left > EG2 ? 1 : (left > EG3 ? 2 : (left > EG4 ? 3 : (left > EG4 / 2 ? 4 : 8)));
            win = win < wmin ? wmin : win;
        };
        *(LAS unsigned*)(pk + PK_DATA_END + 256 + lane * 4) = 0u;
        issue(); land(false);
        const u32x2 mm = *(LAS const u32x2*)(pk + lane * 8);
        WAIT_LGKM0();
        tstart = (unsigned)__builtin_amdgcn_s_memrealtime();
        unsigned ad[64];
        unsigned base = 512;
        const unsigned zero = PK_DATA_END + 256 + lane * 4;
#pragma unroll
        for (int q = 0; q < 2; ++q) {
            const bool fly = permit && i < nld;
            if (fly) issue();
#pragma unroll
            for (int k = q * 32; k < (q + 1) * 32; ++k) {
                const bool nz = (int)((k < 32 ? mm.x : mm.y) << (k & 31)) < 0;
                const unsigned long long m = __ballot(nz);
                const unsigned pos = __builtin_amdgcn_mbcnt_hi((unsigned)(m >> 32), __builtin_amdgcn_mbcnt_lo((unsigned)m, 0u)) * 4 + base;
                base += (unsigned)__builtin_popcountll(m) * 4;
                ad[k] = (unsigned)(size_t)pk + (nz ? pos : zero);
                asm volatile("" : "+v"(ad[k]));
            }
            if (fly) land(true);
        }
        if (!permit && i < nld) {
            for (;;) {
                unsigned f = 0;
                if (lane == 0) f = __hip_atomic_load((const unsigned*)(myws + OFF_FD + tile * 128 + 64 + rot * 8 + 4), __ATOMIC_RELAXED, __HIP_MEMORY_SCOPE_SYSTEM);
                if ((unsigned)__builtin_amdgcn_readfirstlane(f) >= call || gave_up(t0, polls, call, myws + OFF_TMO)) break;
                __builtin_amdgcn_s_sleep(GATE_SLEEP);
            }
            tstart = (unsigned)__builtin_amdgcn_s_memrealtime();
            if (tJ < START2) win = 2;
        }
        while (i < nld) { issue(); land(true); }
        __builtin_amdgcn_s_waitcnt(0x0F70);
#pragma unroll
        for (int r = 0; r < 16; ++r) {
            f32x4 v;
#pragma unroll
            for (int c = 0; c < 4; ++c) v[c] = *(LAS const float*)(size_t)ad[r * 4 + c];
            __builtin_amdgcn_raw_buffer_store_b128(__builtin_bit_cast(u32x4, v), ro, off0 + (unsigned)((r & 3) + 8 * (r >> 2)) * (GN * 4), 0, AUX_SYS);
        }
    }
}

}

extern "C" void kernel_launch(int rank, int nranks, ncclComm_t comm, hipStream_t stream,
                              void* const* d_in, const int* in_sizes, int n_in,
                              void* d_out, int out_size, const int* in_dtypes, int out_dtype,
                              void* const* d_out_all, void* const* d_ws_all, size_t ws_bytes) {
    (void)comm; (void)in_dtypes; (void)out_dtype; (void)d_out_all;
    if (nranks != NR || n_in != 4 || in_sizes[0] != GM * KSH || in_sizes[1] != KSH * GN || out_size != MSH * GN ||
        ws_bytes < WS_TOTAL || (unsigned)rank >= (unsigned)NR) {
        fprintf(stderr, "kernel_launch: unexpected shapes\n");
        return;
    }
    Params p;
    p.x = (const uint8_t*)d_in[0]; p.w = (const uint8_t*)d_in[1];
    p.sx = (const float*)d_in[2]; p.sw = (const float*)d_in[3];
    p.out = (uint8_t*)d_out;
    for (int i = 0; i < NR; ++i) p.ws[i] = (uint8_t*)d_ws_all[i];
    p.rank = rank; p.pad = 0; p.myws = p.ws[rank];
    HIP_CHECK(hipFuncSetAttribute((const void*)gemm_xchg, hipFuncAttributeMaxDynamicSharedMemorySize, LDS_BYTES));
    hipLaunchKernelGGL(gemm_xchg, dim3(NWG), dim3(NTHR), LDS_BYTES, stream, p);
    HIP_CHECK(hipGetLastError());
}
```

```cpp
#include <hip/hip_runtime.h>
#include <rccl/rccl.h>
#include <cstdio>
#include <cstdint>

#define HIP_CHECK(expr) do { hipError_t _e = (expr); if (_e != hipSuccess) { \
    fprintf(stderr, "HIP error %s at %s:%d\n", hipGetErrorString(_e), __FILE__, __LINE__); \
    return; } } while (0)

namespace {

typedef float f32x4 __attribute__((ext_vector_type(4)));
typedef float f32x16 __attribute__((ext_vector_type(16)));
typedef int i32x4 __attribute__((ext_vector_type(4)));
typedef int i32x8 __attribute__((ext_vector_type(8)));
typedef unsigned u32x2 __attribute__((ext_vector_type(2)));
typedef unsigned u32x4 __attribute__((ext_vector_type(4)));
#define LAS __attribute__((address_space(3)))

constexpr int NR = 8;
constexpr int GM = 4096, GK = 4096, GN = 16384;
constexpr int KSH = GK / NR;
constexpr int MSH = GM / NR;
constexpr int NWG = 256;
constexpr int NTHR = 512;
constexpr int NCOMM = 32;
constexpr int PB = 4096;

constexpr unsigned OFF_FX = 0;
constexpr unsigned OFF_FW = 16384;
constexpr unsigned OFF_FO = 32768;
constexpr unsigned OFF_FS = 40960;
constexpr unsigned OFF_XC = 34816;
constexpr unsigned OFF_TMO = 37120;
constexpr unsigned OFF_TK = 38912;
constexpr unsigned FLAG_BYTES = 65536;
constexpr unsigned OFF_FD = 65536;
constexpr unsigned OFF_CNT = OFF_FD + NWG * 128;
constexpr unsigned OFF_SX = 1u << 20;
constexpr unsigned OFF_SW = OFF_SX + 128 * 16384;
constexpr unsigned S_COPY = 128 * 16384 + 128 * 65536;
constexpr unsigned OFF_XS = OFF_SX + 2 * S_COPY;
constexpr unsigned OFF_WS = OFF_XS + 80 * 131072;
constexpr unsigned PK_SLOT = 17408;
constexpr unsigned PK_DATA_END = 512 + 16384;
constexpr unsigned OFF_PK = OFF_WS + 40 * 524288;
constexpr unsigned WS_TOTAL = OFF_PK + NWG * 8 * PK_SLOT;
constexpr unsigned OUT_BYTES = (unsigned)MSH * GN * 4;
constexpr unsigned OOB = 0x40000000u;
static_assert(WS_TOTAL < OOB, "");

constexpr int NS = 6;
constexpr int SLOT = 24576;
constexpr int LDS_BYTES = NS * SLOT + 64;
constexpr unsigned long long DEADLINE_TICKS = 50000000ull;
constexpr int AUX_SYS = 17;

constexpr int POLL_SLEEP = 8, POLL_SLOW = 32, RX_SLEEP = 8, GATE_SLEEP = 16, MON_SLEEP = 4;
constexpr int STAGE_DELAY = 32, STAGE_DELAY2 = 64, PH0_DELAY = 127;
constexpr int XMON = 100;
constexpr int NXC = 4;
constexpr int ROT = 3;
constexpr int PUSH_WIN = 2;
constexpr int LEAD = 8, P1 = 16;
constexpr int NHELP = 48;
constexpr int GRANT_EARLY = 32;
constexpr unsigned LAT_LO = 85, LAT_HI = 110, LAT_BACK = 170;
constexpr int BACKOFF = 16, WMAX = 8;
constexpr int UNIT_TICKS = 14, TKAHEAD = 12, EG2 = 56, EG3 = 36, EG4 = 24;

struct Params {
    const uint8_t* x;
    const uint8_t* w;
    uint8_t* myws;
    int rank, pad;
    const float* sx;
    const float* sw;
    uint8_t* out;
    uint8_t* ws[NR];
};

#define WAIT_VM(n) asm volatile("s_waitcnt vmcnt(%0)" :: "n"(n) : "memory")
#define WAIT_LGKM0() asm volatile("s_waitcnt lgkmcnt(0)" ::: "memory")
#define BAR() do { asm volatile("" ::: "memory"); __builtin_amdgcn_s_barrier(); asm volatile("" ::: "memory"); } while (0)

__device__ __forceinline__ __amdgpu_buffer_rsrc_t make_rsrc(const void* p, unsigned bytes) {
    return __builtin_amdgcn_make_buffer_rsrc((void*)p, 0, (int)bytes, 0x00020000);
}
__device__ __forceinline__ void flag_store(uint8_t* p, unsigned v) {
    __hip_atomic_store((unsigned*)p, v, __ATOMIC_RELAXED, __HIP_MEMORY_SCOPE_SYSTEM);
}
__device__ __forceinline__ bool gave_up(unsigned long long t0, unsigned& polls, unsigned call, uint8_t* tmo) {
    if ((++polls & 63u) != 0u) return false;
    const unsigned t = __builtin_amdgcn_readfirstlane(__hip_atomic_load((const unsigned*)tmo, __ATOMIC_RELAXED, __HIP_MEMORY_SCOPE_AGENT));
    if (t == call || __builtin_amdgcn_s_memrealtime() - t0 > DEADLINE_TICKS) {
        __hip_atomic_store((unsigned*)tmo, call, __ATOMIC_RELAXED, __HIP_MEMORY_SCOPE_AGENT);
        return true;
    }
    return false;
}
__device__ __forceinline__ bool all_ge(u32x4 v, unsigned c) { return v.x >= c && v.y >= c && v.z >= c && v.w >= c; }

__device__ __forceinline__ int partner(int p, int m) { return p < 4 ? 4 + ((p + m) & 3) : ((p - m) & 3); }
__device__ __forceinline__ int gcb_of(int p, int J) {
    const int q = partner(p, J & 3), g = p < 4 ? p * 4 + (q & 3) : q * 4 + (p & 3);
    return g * 8 + (J >> 2);
}
__device__ __forceinline__ int grg_of(int p, int i, int e) {
    if (e < 2) return p * 16 + 2 * i + e;
    if (e < 6) return partner(p, e - 2) * 16 + i;
    if (i < 7) return ((p & 4) | ((p + 1 + e - 6) & 3)) * 16 + 9 + i;
    return partner(p, e - 6) * 16 + 8;
}
__device__ __forceinline__ int entry_of(int role, int i, int J) {
    if (role < 2) return role;
    if (role < 5) return 2 + ((J + role - 1) & 3);
    return i < 7 ? role + 1 : 6 + ((J + role - 4) & 3);
}
__device__ __forceinline__ int mirror(int role, int i) { return role >= 5 && i < 7 ? 12 - role : role; }

__device__ __forceinline__ unsigned ws_block(int J, unsigned call) { return OFF_WS + (unsigned)(J < 8 && (call & 1u) ? J + 32 : J) * 524288u; }

__device__ __forceinline__ bool wait_operands(uint8_t* ws, int i, int J, int lane, unsigned call, int ph, int rank) {
    const unsigned long long t0 = __builtin_amdgcn_s_memrealtime();
    const __amdgpu_buffer_rsrc_t rf = make_rsrc(ws, FLAG_BYTES);
    unsigned polls = 0;
    const unsigned wbase = OFF_FW + (J * 2 + ph - 1) * 256;
    if (ph != 0) {
        for (;;) {
            unsigned v = call;
            if (lane < 8 && lane != rank) v = __builtin_amdgcn_raw_buffer_load_b32(rf, wbase + lane * 32 + i * 4, 0, AUX_SYS);
            if (__builtin_popcountll(__ballot(v >= call)) >= 57 + 2) break;
            if (gave_up(t0, polls, call, ws + OFF_TMO)) return false;
            __builtin_amdgcn_s_sleep(POLL_SLOW);
            asm volatile("" ::: "memory");
        }
    }
    const int d0 = (lane & 7) * 4, nv = i < 7 ? 18 : 20;
    const bool xneed = ph == 0 || ((lane >> 3) != rank && d0 < nv);
    const bool xhalf = ph != 0 && d0 + 2 >= nv;
    const unsigned xoff = (ph == 0 ? OFF_FO : OFF_FX + (i * 2 + ph - 1) * 1024) + lane * 16;
    const bool wneed = ph != 0 && lane < 16 && (lane >> 1) != rank;
    bool xok = false, wok = false;
    for (;;) {
        u32x4 v = {call, call, call, call}, w = v;
        if (xneed && !xok) v = __builtin_amdgcn_raw_buffer_load_b128(rf, xoff, 0, AUX_SYS);
        if (wneed && !wok) w = __builtin_amdgcn_raw_buffer_load_b128(rf, wbase + lane * 16, 0, AUX_SYS);
        if (xhalf) { v.z = call; v.w = call; }
        xok = __all(all_ge(v, call)); wok = __all(all_ge(w, call));
        if (wok && xok) return true;
        if (gave_up(t0, polls, call, ws + OFF_TMO)) return false;
        if (xok) __builtin_amdgcn_s_sleep(POLL_SLEEP); else __builtin_amdgcn_s_sleep(POLL_SLOW);
        asm volatile("" ::: "memory");
    }
}

__device__ __forceinline__ int unit_of(unsigned n, int rank, int peer) {
    unsigned tile, role;
    if (((rank ^ peer) & 4) == 0) { tile = (n / 7u) * 8u + n % 7u; role = 5u + ((unsigned)(peer - rank - 1) & 3u); }
    else {
        const unsigned m = (rank < 4 ? (unsigned)(peer - rank) : (unsigned)(rank - peer)) & 3u;
        const unsigned c = n / 9u, u = n % 9u, tt = c % 3u, t = tt < m ? tt : tt + 1u;
        tile = ((c / 3u) * 4u + t) * 8u + (u < 8u ? u : 7u);
        role = (u < 8u ? 2u : 5u) + ((m - t - 1u) & 3u);
    }
    return (int)(tile * 8u + role);
}

struct Frags { i32x4 a[2]; i32x4 b[4][2]; };
__device__ __forceinline__ void read_frags(Frags& f, LAS const uint8_t* pa, LAS const uint8_t* pb) {
    f.a[0] = *(LAS const i32x4*)(pa); f.a[1] = *(LAS const i32x4*)(pa + 1024);
#pragma unroll
    for (int c = 0; c < 4; ++c) { f.b[c][0] = *(LAS const i32x4*)(pb + c * 2048); f.b[c][1] = *(LAS const i32x4*)(pb + c * 2048 + 1024); }
}
__device__ __forceinline__ i32x8 cat(i32x4 a, i32x4 b) { return __builtin_shufflevector(a, b, 0, 1, 2, 3, 4, 5, 6, 7); }
template <int C0, int C1> __device__ __forceinline__ void mma(f32x16 (&acc)[4], const Frags& f) {
    const i32x8 a = cat(f.a[0], f.a[1]);
#pragma unroll
    for (int c = C0; c < C1; ++c)
        acc[c] = __builtin_amdgcn_mfma_scale_f32_32x32x64_f8f6f4(a, cat(f.b[c][0], f.b[c][1]), acc[c], 0, 0, 0, 127, 0, 127);
}

constexpr int XPC = 146;
constexpr int NPC = 2 * XPC + (32 + 28) * 8;
struct Piece { unsigned so, dso, fo; int staged; };
__device__ __forceinline__ Piece piece_of(int pc, int rank, int peer, unsigned call) {
    Piece r;
    const unsigned cp = (call & 1u) * S_COPY;
    constexpr int E = (LEAD - 4) * 8, NT = (32 - LEAD - P1) / 2;
    static_assert((32 - LEAD - P1) % 2 == 0 && NT >= 0, "");
    int n = -1, half = 0, V = 0;
    if (pc < XPC) n = pc;
    else if (pc < XPC + E) V = pc - XPC;
    else if (pc < 2 * XPC + E) { n = pc - XPC - E; half = 1; }
    else V = pc - 2 * XPC;
    if (n >= 0) {
        int i = n / 18; i = i < 7 ? i : 7;
        const int rem = n - 18 * i, e = rem >> 1;
        const unsigned o = (unsigned)(half * 2 + (rem & 1)) * PB;
        r.so = OFF_SX + cp + (unsigned)grg_of(rank, i, e) * 16384u + o;
        r.dso = OFF_XS + (unsigned)(i * 10 + e) * 131072u + (unsigned)peer * 16384u + o;
        r.fo = OFF_FX + (unsigned)((i * 2 + half) * 8 + peer) * 128u + (unsigned)rem * 4u;
        r.staged = half ^ 1;
    } else {
        const int k = V & 7, U = (V >> 3) - (LEAD - 4);
        int J;
        if (U < 0) { half = 0; J = LEAD + U; }
        else if (U < 2 * P1) { half = (U & 1) ^ 1; J = (U >> 1) + (U & 1) * LEAD; }
        else if (U < 2 * P1 + 3 * NT) { const int z = U - 2 * P1, q = z / 3, m = z - 3 * q; half = m == 0 ? 1 : 0; J = m == 0 ? P1 + q : P1 + LEAD + 2 * q + m - 1; }
        else { half = 1; J = U - 2 * P1 - 3 * NT + P1 + NT; }
        const unsigned o = (unsigned)(half * 8 + k) * PB;
        r.so = OFF_SW + cp + (unsigned)gcb_of(rank, J) * 65536u + o;
        r.dso = ws_block(J, call) + (unsigned)peer * 65536u + o;
        r.fo = OFF_FW + (unsigned)((J * 2 + half) * 8 + peer) * 32u + (unsigned)k * 4u;
        r.staged = 0;
    }
    return r;
}

__global__ void __launch_bounds__(NTHR, 2) gemm_xchg(const Params p) {
    extern __shared__ __attribute__((aligned(16))) uint8_t smem_raw[];
    LAS uint8_t* lds = (LAS uint8_t*)smem_raw;
    const int tid = threadIdx.x, lane = tid & 63, wv = __builtin_amdgcn_readfirstlane(tid >> 6);
    const int rank = p.rank;
    uint8_t* myws = p.myws;
    const int b = blockIdx.x, xcd = b & 7, slot = b >> 3;
    const bool comm = slot >= 28;
    const int rot = (wv + ROT * slot) & 7;
    unsigned* cntp = (unsigned*)(myws + OFF_CNT + b * 128);
    const unsigned cnt_v = __hip_atomic_load(cntp, __ATOMIC_RELAXED, __HIP_MEMORY_SCOPE_AGENT);
    LAS unsigned* ldsflag = (LAS unsigned*)(lds + NS * SLOT);
    if (tid < 2 * NR) *(LAS unsigned*)(lds + NR * PK_SLOT + tid * 4) = 0u;
    if (tid < 2) ldsflag[tid] = 0u;
    const __amdgpu_buffer_rsrc_t rm = make_rsrc(myws, WS_TOTAL);
    unsigned call = 0;
    uint8_t* const lanews = p.ws[lane & 7];

    {
        const int l = lane & 31, kg = lane >> 5;
        if (slot < 16) {
            const uint8_t* src = p.x + (size_t)(b * 32 + l) * KSH + wv * 32 + kg * 16;
            const u32x4 v0 = *(const u32x4*)src;
            call = __builtin_amdgcn_readfirstlane(cnt_v) + 1u;
            const unsigned o = OFF_SX + (call & 1u) * S_COPY + b * 16384 + wv * 1024 + lane * 16;
            __builtin_amdgcn_raw_buffer_store_b128(v0, rm, o, 0, AUX_SYS);
            WAIT_VM(0);
            BAR();
            if (wv == 0) {
                unsigned t = 0;
                if (lane == 0) t = __hip_atomic_fetch_add((unsigned*)(myws + OFF_XC + (b & (NXC - 1)) * 128), 1u, __ATOMIC_RELAXED, __HIP_MEMORY_SCOPE_AGENT);
                if ((unsigned)__builtin_amdgcn_readfirstlane(t) == call * (128u / NXC) - 1u && lane < NR)
                    flag_store(lanews + OFF_FS + rank * 1024 + 128 + (b & (NXC - 1)) * 128, call);
            }
            __builtin_amdgcn_s_sleep(STAGE_DELAY);
            const u32x4 v1 = *(const u32x4*)(src + 256);
            __builtin_amdgcn_raw_buffer_store_b128(v1, rm, o + 8192, 0, AUX_SYS);
        }
        if (slot >= 20 && slot < 28) __builtin_amdgcn_s_sleep(STAGE_DELAY2);
        const bool pusher = slot >= 16 && slot < 20;
        for (int job = pusher ? 0 : 1; job < (comm || (pusher && b >= 144) ? 1 : (slot >= 24 ? 3 : 2)); ++job) {
            const bool push = job == 0;
            int gcb, half;
            if (push) { gcb = (b & 15) * 8; half = 0; }
            else { const int n = (b < 144 ? b : b - 16) + (job == 2 ? 32 : 0), g = n / 15, r = n - g * 15; gcb = g * 8 + (r < 8 ? r : r - 7); half = r < 8 ? 1 : 0; }
            const int sl = 4 * half + (wv >> 1), h = wv & 1;
            const uint8_t* wsrc = p.w + (size_t)(sl * 64 + h * 32 + (lane >> 3)) * GN + gcb * 128 + (lane & 7) * 16;
#pragma unroll
            for (int u = 0; u < 4; ++u)
                __builtin_amdgcn_global_load_lds((const void*)(wsrc + (size_t)u * 8 * GN), (LAS void*)(lds + wv * 4096 + u * 1024), 16, 0, 0);
            const int ha = gcb >> 5, hb = 4 + ((gcb >> 3) & 3), Jh = (hb - ha) & 3;
            const int dst = push ? ((b & 16) ? hb : ha) : rank;
            uint8_t* wd = p.ws[dst];
            WAIT_VM(0);
            unsigned d[16];
#pragma unroll
            for (int kk = 0; kk < 16; ++kk) d[kk] = *(LAS const unsigned*)(lds + wv * 4096 + (kg * 16 + kk) * 128 + l * 4);
            u32x4 o[4];
#pragma unroll
            for (int m = 0; m < 4; ++m) {
                const unsigned d0 = d[4 * m], d1 = d[4 * m + 1], d2 = d[4 * m + 2], d3 = d[4 * m + 3];
                const unsigned t0 = __builtin_amdgcn_perm(d1, d0, 0x05010400u), t1 = __builtin_amdgcn_perm(d1, d0, 0x07030602u);
                const unsigned t2 = __builtin_amdgcn_perm(d3, d2, 0x05010400u), t3 = __builtin_amdgcn_perm(d3, d2, 0x07030602u);
                o[0][m] = __builtin_amdgcn_perm(t2, t0, 0x05040100u);
                o[1][m] = __builtin_amdgcn_perm(t2, t0, 0x07060302u);
                o[2][m] = __builtin_amdgcn_perm(t3, t1, 0x05040100u);
                o[3][m] = __builtin_amdgcn_perm(t3, t1, 0x07060302u);
            }
            call = __builtin_amdgcn_readfirstlane(cnt_v) + 1u;
            const unsigned off = (dst == rank ? OFF_SW + (call & 1u) * S_COPY + (unsigned)(gcb * 8 + sl) * 8192u
                                              : ws_block(Jh, call) + (unsigned)(rank * 8 + sl) * 8192u) + h * 1024 + lane * 16;
            const __amdgpu_buffer_rsrc_t rd = make_rsrc(wd, WS_TOTAL);
#pragma unroll
            for (int c = 0; c < 4; ++c) {
                if (c >= PUSH_WIN && dst != rank) WAIT_VM(PUSH_WIN - 1);
                __builtin_amdgcn_raw_buffer_store_b128(o[c], rd, off + c * 2048, 0, AUX_SYS);
            }
            if (push) {
                WAIT_VM(0);
                BAR();
                if (wv == 0 && lane < 8) flag_store(wd + OFF_FW + (Jh * 2 * 8 + rank) * 32 + lane * 4, call);
            }
        }
        call = __builtin_amdgcn_readfirstlane(cnt_v) + 1u;
        WAIT_VM(0);
        WAIT_LGKM0();
        BAR();
        if (rot == rank) {
            if (lane == 0) flag_store(myws + OFF_FO + b * 4, call);
            if (b == XMON) {
                const unsigned long long t0 = __builtin_amdgcn_s_memrealtime();
                unsigned polls = 0;
                for (;;) {
                    const u32x4 v = __builtin_amdgcn_raw_buffer_load_b128(rm, OFF_FO + lane * 16, 0, AUX_SYS);
                    if (__all(all_ge(v, call))) {
                        if (lane < NR) flag_store(lanews + OFF_FS + rank * 1024, call);
                        break;
                    }
                    if (gave_up(t0, polls, call, myws + OFF_TMO)) break;
                    __builtin_amdgcn_s_sleep(16);
                    asm volatile("" ::: "memory");
                }
            }
        }
    }

    if (comm) {
        const int pk_ = b - 224;
        volatile LAS unsigned* fs = (volatile LAS unsigned*)(lds + NR * PK_SLOT);
        const unsigned long long t0 = __builtin_amdgcn_s_memrealtime();
        unsigned polls = 0;
        if (rot == rank) {
            for (;;) {
                unsigned v = call;
                if ((lane & 7) <= NXC) v = __hip_atomic_load((const unsigned*)(myws + OFF_FS + lane * 128), __ATOMIC_RELAXED, __HIP_MEMORY_SCOPE_SYSTEM);
                if ((__ballot(v < call) & 0xFEFEFEFEFEFEFEFEull) == 0 && (lane & 7) < 2) fs[(lane >> 3) * 2 + (lane & 1)] = v;
                if (__all(v >= call)) break;
                if (gave_up(t0, polls, call, myws + OFF_TMO)) { if (lane == 0) *(volatile LAS unsigned*)(ldsflag + 1) = 1u; break; }
                __builtin_amdgcn_s_sleep(MON_SLEEP);
            }
        } else {
            const int peer = rot;
            const __amdgpu_buffer_rsrc_t rs = make_rsrc(p.ws[peer], WS_TOTAL);
            unsigned dead = 0;
            auto wait_staged = [&](int idx) {
                while ((unsigned)__builtin_amdgcn_readfirstlane(fs[peer * 2 + idx]) < call) {
                    if (__builtin_amdgcn_readfirstlane(*(volatile LAS unsigned*)(ldsflag + 1)) != 0u) { dead = OOB; break; }
                    __builtin_amdgcn_s_sleep(1);
                }
                asm volatile("" ::: "memory");
            };
            const unsigned lo = lane * 16, one = lane == 0 ? 0u : OOB;
            const unsigned tkbase = (call - 1u) * NPC - NCOMM;
            Piece cur = piece_of(pk_, rank, peer, call);
            unsigned pfo = OOB;
            u32x4 v[2];
            wait_staged(cur.staged);
#pragma unroll
            for (int u = 0; u < 2; ++u) {
                __builtin_amdgcn_raw_buffer_store_b128((u32x4){0u, 0u, 0u, 0u}, rm, OOB + lo + u * 1024, 0, AUX_SYS);
                v[u] = __builtin_amdgcn_raw_buffer_load_b128(rs, lo + u * 1024, cur.so, AUX_SYS);
                asm volatile("" ::: "memory");
            }
            int tk = __builtin_amdgcn_raw_ptr_buffer_atomic_add_i32(1, rm, one + OFF_TK + peer * 256, 0, 0);
            bool granted = false;
            for (;;) {
                Piece nxt = cur;
                bool more = false;
                unsigned lo2 = OOB + lo, tko = OOB;
#pragma unroll
                for (int u = 0; u < 4; ++u) {
                    __builtin_amdgcn_raw_buffer_store_b128(v[u & 1], rm, lo + u * 1024, cur.dso, AUX_SYS);
                    if (u < 2) v[u & 1] = __builtin_amdgcn_raw_buffer_load_b128(rs, lo + (u + 2) * 1024, cur.so, AUX_SYS);
                    else v[u & 1] = __builtin_amdgcn_raw_buffer_load_b128(rs, lo2 + (u - 2) * 1024, nxt.so, AUX_SYS);
                    if (u == 0) {
                        WAIT_VM(4);
                        __builtin_amdgcn_raw_buffer_store_b32(call, rm, one | pfo | dead, 0, AUX_SYS);
                    }
                    if (u == 1) {
                        const unsigned pc = (unsigned)__builtin_amdgcn_readfirstlane(tk) - tkbase;
                        more = pc < (unsigned)NPC;
                        if (more) { nxt = piece_of((int)pc, rank, peer, call); wait_staged(nxt.staged); lo2 = lo; tko = one; }
                        if (pc + GRANT_EARLY >= (unsigned)NPC && !granted) {
                            granted = true;
                            if (lane < 2 && pk_ + lane * 32 < NHELP) flag_store(myws + OFF_FD + (32 + pk_ + lane * 32) * 128 + 64 + peer * 8, call);
                        }
                    }
                    if (u == 3) tk = __builtin_amdgcn_raw_ptr_buffer_atomic_add_i32(1, rm, tko + OFF_TK + peer * 256, 0, 0);
                    asm volatile("" ::: "memory");
                }
                pfo = cur.fo;
                cur = nxt;
                if (!more) break;
            }
            WAIT_VM(0);
            __builtin_amdgcn_raw_buffer_store_b32(call, rm, one | pfo | dead, 0, AUX_SYS);
        }
    }
    const float qnan = __builtin_nanf("");
    float sxw = p.sx[0] * p.sw[0];
    asm volatile("" : "+v"(sxw));
    const __amdgpu_buffer_rsrc_t ro = make_rsrc(p.out, OUT_BYTES);
    LAS uint8_t* pk = lds + wv * PK_SLOT;
    const int ti = 2 * (xcd & 3) + (slot & 1);
    int tJ = 2 * (slot >> 1) + (xcd >> 2);
    for (int job = comm ? 2 : (slot < 4 ? 0 : 1); job < 2; ++job) {
    f32x16 acc[4];
#pragma unroll
    for (int c = 0; c < 4; ++c)
#pragma unroll
        for (int r = 0; r < 16; ++r) acc[c][r] = 0.f;
    const int ent = entry_of(wv, ti, tJ), grg = grg_of(rank, ti, ent), gcb = gcb_of(rank, tJ);
    for (int ph = 0; ph < 3; ++ph) {
        if (wv == 0) {
            if (ph == 0 && tJ < 28) __builtin_amdgcn_s_sleep(PH0_DELAY);
            const bool ok = wait_operands(myws, ti, tJ, lane, call, ph, rank);
            if (ph == 1 && tJ < 8) __builtin_amdgcn_fence(__ATOMIC_ACQUIRE, "");
            else __builtin_amdgcn_fence(__ATOMIC_ACQUIRE, "agent");
            if (lane == 0 && !ok) *ldsflag = 1u;
            asm volatile("s_waitcnt vmcnt(0)" ::: "memory");
        }
        WAIT_LGKM0();
        BAR();
        const int nstep = ph == 0 ? 8 : 28;
        const int sh = ph == 0 ? 3 : 2, sbase = ph == 0 ? 0 : rank + 1, sadd = ph == 2 ? 4 : 0;
        const uint8_t* ga = myws + (ph == 0 ? OFF_SX + (call & 1u) * S_COPY + (unsigned)grg * 16384u : OFF_XS + (unsigned)(ti * 10 + ent) * 131072u) + lane * 16;
        const uint8_t* gb = myws + (ph == 0 ? OFF_SW + (call & 1u) * S_COPY + (unsigned)gcb * 65536u : ws_block(tJ, call)) + wv * 1024 + lane * 16;
        auto dma = [&](int n, int sl) {
            const unsigned s = (unsigned)((((sbase + (n >> sh)) & 7) << 3) + (n & ((1 << sh) - 1)) + sadd);
            LAS uint8_t* d = lds + sl * SLOT;
            __builtin_amdgcn_global_load_lds((const void*)(ga + s * 2048u), (LAS void*)(d + wv * 2048), 16, 0, 0);
            __builtin_amdgcn_global_load_lds((const void*)(ga + s * 2048u + 1024), (LAS void*)(d + wv * 2048 + 1024), 16, 0, 0);
            __builtin_amdgcn_global_load_lds((const void*)(gb + s * 8192u), (LAS void*)(d + 16384 + wv * 1024), 16, 0, 0);
        };
        LAS const uint8_t* pa = lds + wv * 2048 + lane * 16;
        LAS const uint8_t* pb = lds + 16384 + lane * 16;
#pragma unroll
        for (int s = 0; s < NS; ++s) dma(s, s);
        WAIT_VM(3 * (NS - 1));
        BAR();
        Frags f0, f1;
        read_frags(f0, pa, pb);
        int sl = 0;
#define SGB(mask, n) __builtin_amdgcn_sched_group_barrier(mask, n, 0)
#define KSTEP(FC, FN, S, VMN, DO_DMA) do { \
            __builtin_amdgcn_sched_barrier(0); \
            WAIT_VM(VMN); WAIT_LGKM0(); BAR(); \
            __builtin_amdgcn_sched_barrier(0); \
            mma<0, 1>(acc, FC); \
            __builtin_amdgcn_sched_barrier(0); \
            const int nsl = sl + 1 == NS ? 0 : sl + 1; \
            read_frags(FN, pa + nsl * SLOT, pb + nsl * SLOT); \
            if (DO_DMA) dma((S) + NS, sl); \
            mma<1, 4>(acc, FC); \
            SGB(0x100, 6); SGB(0x8, 1); SGB(0x100, 4); SGB(0x10, 1); \
            SGB(0x8, 1); SGB(0x10, 2); \
            SGB(0x8, 1); \
            sl = nsl; } while (0)
        for (int s = 0; s < nstep - NS; s += 2) {
            KSTEP(f0, f1, s, 3 * (NS - 2), true);
            KSTEP(f1, f0, s + 1, 3 * (NS - 2), true);
        }
        static_assert(NS == 6, "tail below is written for NS == 6");
        KSTEP(f0, f1, 0, 12, false);
        KSTEP(f1, f0, 0, 9, false);
        KSTEP(f0, f1, 0, 6, false);
        KSTEP(f1, f0, 0, 3, false);
        KSTEP(f0, f1, 0, 0, false);
        mma<0, 4>(acc, f1);
#undef KSTEP
    }
    WAIT_LGKM0();
    BAR();
    const float scale = *ldsflag != 0u ? qnan : sxw;
    if (wv < 2) {
        const unsigned off0 = (unsigned)((grg & 15) * 32 + 4 * (lane >> 5)) * (GN * 4) + (unsigned)(gcb * 128 + 4 * (lane & 31)) * 4;
#pragma unroll
        for (int r = 0; r < 16; ++r) {
            f32x4 v;
#pragma unroll
            for (int c = 0; c < 4; ++c) { const float y = acc[c][r] * scale; v[c] = y <= 0.f ? 0.f : y; }
            __builtin_amdgcn_raw_buffer_store_b128(__builtin_bit_cast(u32x4, v), ro, off0 + (unsigned)((r & 3) + 8 * (r >> 2)) * (GN * 4), 0, AUX_SYS);
        }
    } else {
        const int tile = tJ * 8 + ti;
        unsigned mk[2] = {0u, 0u}, base = 512 + (unsigned)(size_t)pk;
        const unsigned dump = (unsigned)(size_t)pk + PK_DATA_END + lane * 4;
#pragma unroll
        for (int r = 0; r < 16; ++r)
#pragma unroll
            for (int c = 0; c < 4; ++c) {
                const float y = acc[c][r] * scale;
                const bool nz = !(y <= 0.f);
                const unsigned long long m = __ballot(nz);
                const unsigned pos = __builtin_amdgcn_mbcnt_hi((unsigned)(m >> 32), __builtin_amdgcn_mbcnt_lo((unsigned)m, 0u)) * 4 + base;
                *(LAS float*)(size_t)(nz ? pos : dump) = y;
                base += (unsigned)__builtin_popcountll(m) * 4;
                asm("" : "+s"(base));
                mk[r >> 3] = mk[r >> 3] * 2 + (nz ? 1u : 0u);
            }
        base -= (unsigned)(size_t)pk;
        *(LAS u32x2*)(pk + lane * 8) = (u32x2){mk[0], mk[1]};
        const unsigned bytes = base;
        const __amdgpu_buffer_rsrc_t rp = make_rsrc(myws + OFF_PK + (unsigned)(tile * 8 + wv) * PK_SLOT, (bytes + 127u) & ~127u);
        for (unsigned o = 0; o < bytes; o += 5120) {
            u32x4 v[5];
#pragma unroll
            for (int u = 0; u < 5; ++u) v[u] = *(LAS const u32x4*)(pk + (o + u * 1024 < PK_SLOT ? o + u * 1024 : 0u) + lane * 16);
#pragma unroll
            for (int u = 0; u < 5; ++u) __builtin_amdgcn_raw_buffer_store_b128(v[u], rp, o + u * 1024 + lane * 16, 0, AUX_SYS);
        }
        WAIT_VM(0);
        if (lane == 0) __hip_atomic_store((unsigned long long*)(p.ws[(grg >> 4) & 7] + OFF_FD + tile * 128 + mirror(wv, ti) * 8),
                                          ((unsigned long long)call << 32) | bytes, __ATOMIC_RELAXED, __HIP_MEMORY_SCOPE_SYSTEM);
    }
    if (job == 0) tJ += 28;
    }
    if (rot == rank && lane == 0) __hip_atomic_store(cntp, call, __ATOMIC_RELAXED, __HIP_MEMORY_SCOPE_AGENT);
    const bool helper = b >= 32 && b < 32 + NHELP;
    if (rot == rank || !(comm || helper)) return;
    {
        const int peer = rot;
        const unsigned nu = ((rank ^ peer) & 4) == 0 ? 224u : 216u;
        const unsigned one = lane == 0 ? 0u : OOB;
        const unsigned tkoff = one + OFF_TK + peer * 256 + 128, tkbase = (call - 1u) * (nu + NCOMM + NHELP);
        const unsigned long long t0 = __builtin_amdgcn_s_memrealtime();
        unsigned polls = 0;
        if (helper) {
            for (;;) {
                unsigned f = 0;
                if (lane == 0) f = __hip_atomic_load((const unsigned*)(myws + OFF_FD + b * 128 + 64 + peer * 8), __ATOMIC_RELAXED, __HIP_MEMORY_SCOPE_SYSTEM);
                if ((unsigned)__builtin_amdgcn_readfirstlane(f) >= call || gave_up(t0, polls, call, myws + OFF_TMO)) break;
                __builtin_amdgcn_s_sleep(GATE_SLEEP);
            }
        }
        *(LAS unsigned*)(pk + PK_DATA_END + 256 + lane * 4) = 0u;
        int g = __builtin_amdgcn_raw_ptr_buffer_atomic_add_i32(1, rm, tkoff, 0, 0);
        unsigned n = (unsigned)__builtin_amdgcn_readfirstlane(g) - tkbase;
        u32x2 fnext = {0u, 0u};
        int win = 2;
        while (n < nu) {
            const int un = unit_of(n, rank, peer), tile = un >> 3, role = un & 7, ui = tile & 7, uJ = tile >> 3;
            const int g2 = grg_of(rank, ui, entry_of(role, ui, uJ));
            const unsigned off0 = (unsigned)((g2 & 15) * 32 + 4 * (lane >> 5)) * (GN * 4) + (unsigned)(gcb_of(peer, uJ) * 128 + 4 * (lane & 31)) * 4;
            bool ok = true;
            unsigned bytes = __builtin_amdgcn_readfirstlane(fnext.x);
            if ((unsigned)__builtin_amdgcn_readfirstlane(fnext.y) < call) {
                for (;;) {
                    unsigned long long f = 0;
                    if (lane == 0) f = __hip_atomic_load((const unsigned long long*)(myws + OFF_FD + tile * 128 + role * 8), __ATOMIC_RELAXED, __HIP_MEMORY_SCOPE_SYSTEM);
                    bytes = __builtin_amdgcn_readfirstlane((unsigned)f);
                    if ((unsigned)__builtin_amdgcn_readfirstlane((unsigned)(f >> 32)) >= call) break;
                    if (gave_up(t0, polls, call, myws + OFF_TMO)) { ok = false; break; }
                    __builtin_amdgcn_s_sleep(RX_SLEEP);
                }
            }
            asm volatile("" ::: "memory");
            if (bytes > PK_DATA_END) bytes = PK_DATA_END;
            if (!ok) bytes = 0;
            const int nld = (int)((bytes + 1023u) >> 10);
            const uint8_t* src = p.ws[peer] + OFF_PK + (unsigned)(tile * 8 + mirror(role, ui)) * PK_SLOT + lane * 16;
            unsigned ta = 0;
            const unsigned tstart = (unsigned)__builtin_amdgcn_s_memrealtime();
            int i = 0;
            const int behind = (int)(nu - 1u - n);
            auto issue = [&]() {
                const int c = win < nld - i ? win : nld - i;
                ta = (unsigned)__builtin_amdgcn_s_memrealtime();
                for (int u = 0; u < c; ++u, ++i)
                    if ((unsigned)(i * 1024 + (lane & ~7) * 16) < bytes)
                        __builtin_amdgcn_global_load_lds((const void*)(src + i * 1024), (LAS void*)(pk + i * 1024), 16, 0, AUX_SYS);
            };
            auto land = [&]() {
                WAIT_VM(0);
                const unsigned now = (unsigned)__builtin_amdgcn_s_memrealtime(), lat = now - ta;
                if (lat < LAT_LO) win = win * 2 < WMAX ? win * 2 : WMAX;
                else if (lat > LAT_HI) { if (win > 1) win >>= 1; else if (lat > LAT_BACK) __builtin_amdgcn_s_sleep(BACKOFF); }
                const int left = NCOMM + NHELP + behind - TKAHEAD - (int)((now - tstart) / UNIT_TICKS);
                const int wmin = left > EG2 ? 1 : (left > EG3 ? 2 : (left > EG4 ? 3 : (left > EG4 / 2 ? 4 : 8)));
                win = win < wmin ? wmin : win;
            };
            issue(); WAIT_VM(0);
            u32x2 mm = *(LAS const u32x2*)(pk + lane * 8);
            if (!ok) { mm = (u32x2){0u, 0u}; *(LAS float*)(pk + PK_DATA_END + 256 + lane * 4) = qnan; }
            WAIT_LGKM0();
            unsigned ad[64];
            unsigned base = 512, mid = 0;
            const unsigned zero = PK_DATA_END + 256 + lane * 4;
#pragma unroll
            for (int q = 0; q < 2; ++q) {
                const bool fly = i < nld;
                if (fly) issue();
#pragma unroll
                for (int k = q * 32; k < (q + 1) * 32; ++k) {
                    const bool nz = (int)((k < 32 ? mm.x : mm.y) << (k & 31)) < 0;
                    const unsigned long long m = __ballot(nz);
                    const unsigned pos = __builtin_amdgcn_mbcnt_hi((unsigned)(m >> 32), __builtin_amdgcn_mbcnt_lo((unsigned)m, 0u)) * 4 + base;
                    base += (unsigned)__builtin_popcountll(m) * 4;
                    ad[k] = (unsigned)(size_t)pk + (nz ? pos : zero);
                    asm volatile("" : "+v"(ad[k]));
                }
                if (q == 0) mid = base;
                if (fly) land();
            }
            auto expand = [&](int r0) {
#pragma unroll
                for (int r = r0; r < r0 + 8; ++r) {
                    f32x4 v;
#pragma unroll
                    for (int c = 0; c < 4; ++c) v[c] = *(LAS const float*)(size_t)ad[r * 4 + c];
                    __builtin_amdgcn_raw_buffer_store_b128(__builtin_bit_cast(u32x4, v), ro, off0 + (unsigned)((r & 3) + 8 * (r >> 2)) * (GN * 4), 0, AUX_SYS);
                }
            };
            while ((unsigned)i * 1024u < mid && i < nld) { issue(); land(); }
            __builtin_amdgcn_s_waitcnt(0x0F70);
            {
                f32x4 hv[8];
#pragma unroll
                for (int r = 0; r < 8; ++r)
#pragma unroll
                    for (int c = 0; c < 4; ++c) hv[r][c] = *(LAS const float*)(size_t)ad[r * 4 + c];
                asm volatile("" ::: "memory");
                if (i < nld) issue();
#pragma unroll
                for (int r = 0; r < 8; ++r)
                    __builtin_amdgcn_raw_buffer_store_b128(__builtin_bit_cast(u32x4, hv[r]), ro, off0 + (unsigned)((r & 3) + 8 * (r >> 2)) * (GN * 4), 0, AUX_SYS);
                WAIT_VM(8);
            }
            while (i + 2 * win < nld) { issue(); land(); }
            g = __builtin_amdgcn_raw_ptr_buffer_atomic_add_i32(1, rm, tkoff, 0, 0);
            if (i < nld) { issue(); land(); }
            n = (unsigned)__builtin_amdgcn_readfirstlane(g) - tkbase;
            { const int u2 = unit_of(n < nu ? n : 0u, rank, peer);
              fnext = __builtin_amdgcn_raw_buffer_load_b64(rm, one + OFF_FD + (u2 >> 3) * 128 + (u2 & 7) * 8, 0, AUX_SYS); }
            while (i < nld) { issue(); land(); }
            __builtin_amdgcn_s_waitcnt(0x0F70);
            expand(8);
        }
    }
}

}

extern "C" void kernel_launch(int rank, int nranks, ncclComm_t comm, hipStream_t stream,
                              void* const* d_in, const int* in_sizes, int n_in,
                              void* d_out, int out_size, const int* in_dtypes, int out_dtype,
                              void* const* d_out_all, void* const* d_ws_all, size_t ws_bytes) {
    (void)comm; (void)in_dtypes; (void)out_dtype; (void)d_out_all;
    if (nranks != NR || n_in != 4 || in_sizes[0] != GM * KSH || in_sizes[1] != KSH * GN || out_size != MSH * GN ||
        ws_bytes < WS_TOTAL || (unsigned)rank >= (unsigned)NR) {
        fprintf(stderr, "kernel_launch: unexpected shapes\n");
        return;
    }
    Params p;
    p.x = (const uint8_t*)d_in[0]; p.w = (const uint8_t*)d_in[1];
    p.sx = (const float*)d_in[2]; p.sw = (const float*)d_in[3];
    p.out = (uint8_t*)d_out;
    for (int i = 0; i < NR; ++i) p.ws[i] = (uint8_t*)d_ws_all[i];
    p.rank = rank; p.pad = 0; p.myws = p.ws[rank];
    HIP_CHECK(hipFuncSetAttribute((const void*)gemm_xchg, hipFuncAttributeMaxDynamicSharedMemorySize, LDS_BYTES));
    hipLaunchKernelGGL(gemm_xchg, dim3(NWG), dim3(NTHR), LDS_BYTES, stream, p);
    HIP_CHECK(hipGetLastError());
}
```

```cpp
#include <hip/hip_runtime.h>
#include <rccl/rccl.h>
#include <cstdio>
#include <cstdint>

#define HIP_CHECK(expr) do { hipError_t _e = (expr); if (_e != hipSuccess) { \
    fprintf(stderr, "HIP error %s at %s:%d\n", hipGetErrorString(_e), __FILE__, __LINE__); \
    return; } } while (0)

namespace {

typedef float f32x4 __attribute__((ext_vector_type(4)));
typedef float f32x16 __attribute__((ext_vector_type(16)));
typedef int i32x4 __attribute__((ext_vector_type(4)));
typedef int i32x8 __attribute__((ext_vector_type(8)));
typedef unsigned u32x2 __attribute__((ext_vector_type(2)));
typedef unsigned u32x4 __attribute__((ext_vector_type(4)));
#define LAS __attribute__((address_space(3)))

constexpr int NR = 8;
constexpr int GM = 4096, GK = 4096, GN = 16384;
constexpr int KSH = GK / NR;
constexpr int MSH = GM / NR;
constexpr int NWG = 256;
constexpr int NTHR = 512;
constexpr int NCOMM = 32;
constexpr int PB = 4096;

constexpr unsigned OFF_FX = 0;
constexpr unsigned OFF_FW = 16384;
constexpr unsigned OFF_FO = 32768;
constexpr unsigned OFF_FS = 40960;
constexpr unsigned OFF_XC = 34816;
constexpr unsigned OFF_TMO = 37120;
constexpr unsigned OFF_TK = 38912;
constexpr unsigned FLAG_BYTES = 65536;
constexpr unsigned OFF_FD = 65536;
constexpr unsigned OFF_CNT = OFF_FD + NWG * 128;
constexpr unsigned OFF_SX = 1u << 20;
constexpr unsigned OFF_SW = OFF_SX + 128 * 16384;
constexpr unsigned S_COPY = 128 * 16384 + 128 * 65536;
constexpr unsigned OFF_XS = OFF_SX + 2 * S_COPY;
constexpr unsigned OFF_WS = OFF_XS + 80 * 131072;
constexpr unsigned PK_SLOT = 17408;
constexpr unsigned PK_DATA_END = 512 + 16384;
constexpr unsigned OFF_PK = OFF_WS + 40 * 524288;
constexpr unsigned WS_TOTAL = OFF_PK + NWG * 8 * PK_SLOT;
constexpr unsigned OUT_BYTES = (unsigned)MSH * GN * 4;
constexpr unsigned OOB = 0x40000000u;
static_assert(WS_TOTAL < OOB, "");

constexpr int NS = 6;
constexpr int SLOT = 24576;
constexpr int LDS_BYTES = NS * SLOT + 64;
constexpr unsigned long long DEADLINE_TICKS = 50000000ull;
constexpr int AUX_SYS = 17;

constexpr int POLL_SLEEP = 8, POLL_SLOW = 32, RX_SLEEP = 8, GATE_SLEEP = 16, MON_SLEEP = 4;
constexpr int STAGE_DELAY = 32, STAGE_DELAY2 = 64, PH0_DELAY = 127;
constexpr int XMON = 100;
constexpr int NXC = 4;
constexpr int ROT = 3;
constexpr int PUSH_WIN = 1;
constexpr int LEAD = 8, P1 = 16;
constexpr int NHELP = 48;
constexpr int GRANT_EARLY = 32;
constexpr unsigned LAT_LO = 85, LAT_HI = 110, LAT_BACK = 170;
constexpr int BACKOFF = 16, WMAX = 8;
constexpr int UNIT_TICKS = 14, TKAHEAD = 12, EG2 = 56, EG3 = 36, EG4 = 24;

struct Params {
    const uint8_t* x;
    const uint8_t* w;
    uint8_t* myws;
    int rank, pad;
    const float* sx;
    const float* sw;
    uint8_t* out;
    uint8_t* ws[NR];
};

#define WAIT_VM(n) asm volatile("s_waitcnt vmcnt(%0)" :: "n"(n) : "memory")
#define WAIT_LGKM0() asm volatile("s_waitcnt lgkmcnt(0)" ::: "memory")
#define BAR() do { asm volatile("" ::: "memory"); __builtin_amdgcn_s_barrier(); asm volatile("" ::: "memory"); } while (0)

__device__ __forceinline__ __amdgpu_buffer_rsrc_t make_rsrc(const void* p, unsigned bytes) {
    return __builtin_amdgcn_make_buffer_rsrc((void*)p, 0, (int)bytes, 0x00020000);
}
__device__ __forceinline__ void flag_store(uint8_t* p, unsigned v) {
    __hip_atomic_store((unsigned*)p, v, __ATOMIC_RELAXED, __HIP_MEMORY_SCOPE_SYSTEM);
}
__device__ __forceinline__ bool gave_up(unsigned long long t0, unsigned& polls, unsigned call, uint8_t* tmo) {
    if ((++polls & 63u) != 0u) return false;
    const unsigned t = __builtin_amdgcn_readfirstlane(__hip_atomic_load((const unsigned*)tmo, __ATOMIC_RELAXED, __HIP_MEMORY_SCOPE_AGENT));
    if (t == call || __builtin_amdgcn_s_memrealtime() - t0 > DEADLINE_TICKS) {
        __hip_atomic_store((unsigned*)tmo, call, __ATOMIC_RELAXED, __HIP_MEMORY_SCOPE_AGENT);
        return true;
    }
    return false;
}
__device__ __forceinline__ bool all_ge(u32x4 v, unsigned c) { return v.x >= c && v.y >= c && v.z >= c && v.w >= c; }

__device__ __forceinline__ int partner(int p, int m) { return p < 4 ? 4 + ((p + m) & 3) : ((p - m) & 3); }
__device__ __forceinline__ int gcb_of(int p, int J) {
    const int q = partner(p, J & 3), g = p < 4 ? p * 4 + (q & 3) : q * 4 + (p & 3);
    return g * 8 + (J >> 2);
}
__device__ __forceinline__ int grg_of(int p, int i, int e) {
    if (e < 2) return p * 16 + 2 * i + e;
    if (e < 6) return partner(p, e - 2) * 16 + i;
    if (i < 7) return ((p & 4) | ((p + 1 + e - 6) & 3)) * 16 + 9 + i;
    return partner(p, e - 6) * 16 + 8;
}
__device__ __forceinline__ int entry_of(int role, int i, int J) {
    if (role < 2) return role;
    if (role < 5) return 2 + ((J + role - 1) & 3);
    return i < 7 ? role + 1 : 6 + ((J + role - 4) & 3);
}
__device__ __forceinline__ int mirror(int role, int i) { return role >= 5 && i < 7 ? 12 - role : role; }

__device__ __forceinline__ unsigned ws_block(int J, unsigned call) { return OFF_WS + (unsigned)(J < 8 && (call & 1u) ? J + 32 : J) * 524288u; }

__device__ __forceinline__ bool wait_operands(uint8_t* ws, int i, int J, int lane, unsigned call, int ph, int rank) {
    const unsigned long long t0 = __builtin_amdgcn_s_memrealtime();
    const __amdgpu_buffer_rsrc_t rf = make_rsrc(ws, FLAG_BYTES);
    unsigned polls = 0;
    const unsigned wbase = OFF_FW + (J * 2 + ph - 1) * 256;
    if (ph != 0) {
        for (;;) {
            unsigned v = call;
            if (lane < 8 && lane != rank) v = __builtin_amdgcn_raw_buffer_load_b32(rf, wbase + lane * 32 + i * 4, 0, AUX_SYS);
            if (__builtin_popcountll(__ballot(v >= call)) >= 57 + 2) break;
            if (gave_up(t0, polls, call, ws + OFF_TMO)) return false;
            __builtin_amdgcn_s_sleep(POLL_SLOW);
            asm volatile("" ::: "memory");
        }
    }
    const int d0 = (lane & 7) * 4, nv = i < 7 ? 18 : 20;
    const bool xneed = ph == 0 || ((lane >> 3) != rank && d0 < nv);
    const bool xhalf = ph != 0 && d0 + 2 >= nv;
    const unsigned xoff = (ph == 0 ? OFF_FO : OFF_FX + (i * 2 + ph - 1) * 1024) + lane * 16;
    const bool wneed = ph != 0 && lane < 16 && (lane >> 1) != rank;
    bool xok = false, wok = false;
    for (;;) {
        u32x4 v = {call, call, call, call}, w = v;
        if (xneed && !xok) v = __builtin_amdgcn_raw_buffer_load_b128(rf, xoff, 0, AUX_SYS);
        if (wneed && !wok) w = __builtin_amdgcn_raw_buffer_load_b128(rf, wbase + lane * 16, 0, AUX_SYS);
        if (xhalf) { v.z = call; v.w = call; }
        xok = __all(all_ge(v, call)); wok = __all(all_ge(w, call));
        if (wok && xok) return true;
        if (gave_up(t0, polls, call, ws + OFF_TMO)) return false;
        if (xok) __builtin_amdgcn_s_sleep(POLL_SLEEP); else __builtin_amdgcn_s_sleep(POLL_SLOW);
        asm volatile("" ::: "memory");
    }
}

__device__ __forceinline__ int unit_of(unsigned n, int rank, int peer) {
    unsigned tile, role;
    if (((rank ^ peer) & 4) == 0) { tile = (n / 7u) * 8u + n % 7u; role = 5u + ((unsigned)(peer - rank - 1) & 3u); }
    else {
        const unsigned m = (rank < 4 ? (unsigned)(peer - rank) : (unsigned)(rank - peer)) & 3u;
        const unsigned c = n / 9u, u = n % 9u, tt = c % 3u, t = tt < m ? tt : tt + 1u;
        tile = ((c / 3u) * 4u + t) * 8u + (u < 8u ? u : 7u);
        role = (u < 8u ? 2u : 5u) + ((m - t - 1u) & 3u);
    }
    return (int)(tile * 8u + role);
}

struct Frags { i32x4 a[2]; i32x4 b[4][2]; };
__device__ __forceinline__ void read_frags(Frags& f, LAS const uint8_t* pa, LAS const uint8_t* pb) {
    f.a[0] = *(LAS const i32x4*)(pa); f.a[1] = *(LAS const i32x4*)(pa + 1024);
#pragma unroll
    for (int c = 0; c < 4; ++c) { f.b[c][0] = *(LAS const i32x4*)(pb + c * 2048); f.b[c][1] = *(LAS const i32x4*)(pb + c * 2048 + 1024); }
}
__device__ __forceinline__ i32x8 cat(i32x4 a, i32x4 b) { return __builtin_shufflevector(a, b, 0, 1, 2, 3, 4, 5, 6, 7); }
template <int C0, int C1> __device__ __forceinline__ void mma(f32x16 (&acc)[4], const Frags& f) {
    const i32x8 a = cat(f.a[0], f.a[1]);
#pragma unroll
    for (int c = C0; c < C1; ++c)
        acc[c] = __builtin_amdgcn_mfma_scale_f32_32x32x64_f8f6f4(a, cat(f.b[c][0], f.b[c][1]), acc[c], 0, 0, 0, 127, 0, 127);
}

constexpr int XPC = 146;
constexpr int NPC = 2 * XPC + (32 + 28) * 8;
struct Piece { unsigned so, dso, fo; int staged; };
__device__ __forceinline__ Piece piece_of(int pc, int rank, int peer, unsigned call) {
    Piece r;
    const unsigned cp = (call & 1u) * S_COPY;
    constexpr int E = (LEAD - 4) * 8, NT = (32 - LEAD - P1) / 2;
    static_assert((32 - LEAD - P1) % 2 == 0 && NT >= 0, "");
    int n = -1, half = 0, V = 0;
    if (pc < XPC) n = pc;
    else if (pc < XPC + E) V = pc - XPC;
    else if (pc < 2 * XPC + E) { n = pc - XPC - E; half = 1; }
    else V = pc - 2 * XPC;
    if (n >= 0) {
        int i = n / 18; i = i < 7 ? i : 7;
        const int rem = n - 18 * i, e = rem >> 1;
        const unsigned o = (unsigned)(half * 2 + (rem & 1)) * PB;
        r.so = OFF_SX + cp + (unsigned)grg_of(rank, i, e) * 16384u + o;
        r.dso = OFF_XS + (unsigned)(i * 10 + e) * 131072u + (unsigned)peer * 16384u + o;
        r.fo = OFF_FX + (unsigned)((i * 2 + half) * 8 + peer) * 128u + (unsigned)rem * 4u;
        r.staged = half ^ 1;
    } else {
        const int k = V & 7, U = (V >> 3) - (LEAD - 4);
        int J;
        if (U < 0) { half = 0; J = LEAD + U; }
        else if (U < 2 * P1) { half = (U & 1) ^ 1; J = (U >> 1) + (U & 1) * LEAD; }
        else if (U < 2 * P1 + 3 * NT) { const int z = U - 2 * P1, q = z / 3, m = z - 3 * q; half = m == 0 ? 1 : 0; J = m == 0 ? P1 + q : P1 + LEAD + 2 * q + m - 1; }
        else { half = 1; J = U - 2 * P1 - 3 * NT + P1 + NT; }
        const unsigned o = (unsigned)(half * 8 + k) * PB;
        r.so = OFF_SW + cp + (unsigned)gcb_of(rank, J) * 65536u + o;
        r.dso = ws_block(J, call) + (unsigned)peer * 65536u + o;
        r.fo = OFF_FW + (unsigned)((J * 2 + half) * 8 + peer) * 32u + (unsigned)k * 4u;
        r.staged = 0;
    }
    return r;
}

__global__ void __launch_bounds__(NTHR, 2) gemm_xchg(const Params p) {
    extern __shared__ __attribute__((aligned(16))) uint8_t smem_raw[];
    LAS uint8_t* lds = (LAS uint8_t*)smem_raw;
    const int tid = threadIdx.x, lane = tid & 63, wv = __builtin_amdgcn_readfirstlane(tid >> 6);
    const int rank = p.rank;
    uint8_t* myws = p.myws;
    const uint8_t *px = p.x, *pw = p.w;
    const int b = blockIdx.x, xcd = b & 7, slot = b >> 3;
    const bool comm = slot >= 28;
    const int rot = (wv + ROT * slot) & 7;
    asm volatile("" :: "s"(px), "s"(pw), "s"(myws), "s"(rank));
    uint8_t* const pws = p.ws[(unsigned)(slot - 16) < 4u ? ((b >> 2) & 4) + ((b >> (2 - ((b >> 3) & 2))) & 3) : rot];
    u32x4 xv0 = {0u, 0u, 0u, 0u};
    if (comm) {
        const unsigned o = lane < 4 && rot != rank ? (lane & 1) * (OFF_WS - OFF_SX) + (lane >> 1) * ((lane & 1) ? 32u * 524288u : S_COPY) + OFF_SX : OOB;
        xv0.x = __builtin_amdgcn_raw_buffer_load_b32(make_rsrc(pws, WS_TOTAL), o + (unsigned)(rank * 32 + (b & 31)) * 128u, 0, AUX_SYS);
    }
    else if (slot < 16) xv0 = *(const u32x4*)(px + (size_t)(b * 32 + (lane & 31)) * KSH + wv * 32 + (lane >> 5) * 16);
    else if (slot < 20) {
        const uint8_t* wsrc = pw + (size_t)(wv * 32 + (lane >> 3)) * GN + (b & 15) * 1024 + (lane & 7) * 16;
#pragma unroll
        for (int u = 0; u < 4; ++u)
            __builtin_amdgcn_global_load_lds((const void*)(wsrc + (size_t)u * 8 * GN), (LAS void*)(lds + wv * 4096 + u * 1024), 16, 0, 0);
    }
    asm volatile("" ::: "memory");
    __builtin_amdgcn_sched_barrier(0);
    unsigned* cntp = (unsigned*)(myws + OFF_CNT + b * 128);
    const unsigned cnt_v = __hip_atomic_load(cntp, __ATOMIC_RELAXED, __HIP_MEMORY_SCOPE_AGENT);
    asm volatile("" :: "s"(pws));
    LAS unsigned* ldsflag = (LAS unsigned*)(lds + NS * SLOT);
    if (tid < 2 * NR) *(LAS unsigned*)(lds + NR * PK_SLOT + tid * 4) = 0u;
    if (tid < 2) ldsflag[tid] = 0u;
    const __amdgpu_buffer_rsrc_t rm = make_rsrc(myws, WS_TOTAL);
    unsigned call = 0;
    uint8_t* const lanews = p.ws[lane & 7];

    {
        const int l = lane & 31, kg = lane >> 5;
        if (slot < 16) {
            const uint8_t* src = px + (size_t)(b * 32 + l) * KSH + wv * 32 + kg * 16;
            const u32x4 v0 = xv0;
            call = __builtin_amdgcn_readfirstlane(cnt_v) + 1u;
            const unsigned o = OFF_SX + (call & 1u) * S_COPY + b * 16384 + wv * 1024 + lane * 16;
            __builtin_amdgcn_raw_buffer_store_b128(v0, rm, o, 0, AUX_SYS);
            WAIT_VM(0);
            BAR();
            if (wv == 0) {
                unsigned t = 0;
                if (lane == 0) t = __hip_atomic_fetch_add((unsigned*)(myws + OFF_XC + (b & (NXC - 1)) * 128), 1u, __ATOMIC_RELAXED, __HIP_MEMORY_SCOPE_AGENT);
                if ((unsigned)__builtin_amdgcn_readfirstlane(t) == call * (128u / NXC) - 1u && lane < NR)
                    flag_store(lanews + OFF_FS + rank * 1024 + 128 + (b & (NXC - 1)) * 128, call);
            }
            __builtin_amdgcn_s_sleep(STAGE_DELAY);
            const u32x4 v1 = *(const u32x4*)(src + 256);
            __builtin_amdgcn_raw_buffer_store_b128(v1, rm, o + 8192, 0, AUX_SYS);
        }
        auto xpose = [&](u32x4 (&o)[4]) {
            unsigned d[16];
#pragma unroll
            for (int kk = 0; kk < 16; ++kk) d[kk] = *(LAS const unsigned*)(lds + wv * 4096 + (kg * 16 + kk) * 128 + l * 4);
#pragma unroll
            for (int m = 0; m < 4; ++m) {
                const unsigned d0 = d[4 * m], d1 = d[4 * m + 1], d2 = d[4 * m + 2], d3 = d[4 * m + 3];
                const unsigned t0 = __builtin_amdgcn_perm(d1, d0, 0x05010400u), t1 = __builtin_amdgcn_perm(d1, d0, 0x07030602u);
                const unsigned t2 = __builtin_amdgcn_perm(d3, d2, 0x05010400u), t3 = __builtin_amdgcn_perm(d3, d2, 0x07030602u);
                o[0][m] = __builtin_amdgcn_perm(t2, t0, 0x05040100u);
                o[1][m] = __builtin_amdgcn_perm(t2, t0, 0x07060302u);
                o[2][m] = __builtin_amdgcn_perm(t3, t1, 0x05040100u);
                o[3][m] = __builtin_amdgcn_perm(t3, t1, 0x07060302u);
            }
        };
        const unsigned wlo = (wv >> 1) * 8192 + (wv & 1) * 1024 + lane * 16;
        if (slot >= 16 && slot < 20) {
            const int ha = (b >> 2) & 3, hb = 4 + (b & 3), Jh = (hb - ha) & 3, dst = (b & 16) ? hb : ha;
            uint8_t* wd = pws;
            const __amdgpu_buffer_rsrc_t rd = make_rsrc(wd, WS_TOTAL);
            const unsigned o0 = (dst == rank ? OFF_SW + (unsigned)(b & 15) * 524288u : OFF_WS + (unsigned)Jh * 524288u + (unsigned)rank * 65536u) + wlo;
            const unsigned ostep = dst == rank ? S_COPY : 32u * 524288u;
            asm volatile("" :: "s"(wd), "v"(o0), "s"(ostep));
            WAIT_VM(0);
            u32x4 o[4];
            xpose(o);
            call = __builtin_amdgcn_readfirstlane(cnt_v) + 1u;
#pragma unroll
            for (int c = 0; c < 4; ++c) {
                if (c >= PUSH_WIN) WAIT_VM(PUSH_WIN - 1);
                __builtin_amdgcn_raw_buffer_store_b128(o[c], rd, o0 + c * 2048, (call & 1u) * ostep, AUX_SYS);
            }
            WAIT_VM(0);
            BAR();
            if (wv == 0 && lane < 8) flag_store(wd + OFF_FW + (Jh * 2 * 8 + rank) * 32 + lane * 4, call);
        }
        if (slot >= 20 && slot < 28) __builtin_amdgcn_s_sleep(STAGE_DELAY2);
        for (int job = 0; job < (comm || (b >= 144 && b < 160) ? 0 : (slot >= 24 ? 2 : 1)); ++job) {
            const int n = (b < 144 ? b : b - 16) + job * 32, g = n / 15, r = n - g * 15, gcb = g * 8 + (r < 8 ? r : r - 7), half = r < 8 ? 1 : 0;
            const uint8_t* wsrc = pw + (size_t)(half * 256 + wv * 32 + (lane >> 3)) * GN + gcb * 128 + (lane & 7) * 16;
#pragma unroll
            for (int u = 0; u < 4; ++u)
                __builtin_amdgcn_global_load_lds((const void*)(wsrc + (size_t)u * 8 * GN), (LAS void*)(lds + wv * 4096 + u * 1024), 16, 0, 0);
            WAIT_VM(0);
            u32x4 o[4];
            xpose(o);
            call = __builtin_amdgcn_readfirstlane(cnt_v) + 1u;
#pragma unroll
            for (int c = 0; c < 4; ++c)
                __builtin_amdgcn_raw_buffer_store_b128(o[c], rm, OFF_SW + (unsigned)gcb * 65536u + (unsigned)half * 32768u + wlo + c * 2048, (call & 1u) * S_COPY, AUX_SYS);
        }
        call = __builtin_amdgcn_readfirstlane(cnt_v) + 1u;
        WAIT_VM(0);
        asm volatile("" :: "v"(xv0.x));
        WAIT_LGKM0();
        BAR();
        if (rot == rank) {
            if (lane == 0) flag_store(myws + OFF_FO + b * 4, call);
            if (b == XMON) {
                const unsigned long long t0 = __builtin_amdgcn_s_memrealtime();
                unsigned polls = 0;
                for (;;) {
                    const u32x4 v = __builtin_amdgcn_raw_buffer_load_b128(rm, OFF_FO + lane * 16, 0, AUX_SYS);
                    if (__all(all_ge(v, call))) {
                        if (lane < NR) flag_store(lanews + OFF_FS + rank * 1024, call);
                        break;
                    }
                    if (gave_up(t0, polls, call, myws + OFF_TMO)) break;
                    __builtin_amdgcn_s_sleep(16);
                    asm volatile("" ::: "memory");
                }
            }
        }
    }

    if (comm) {
        const int pk_ = b - 224;
        volatile LAS unsigned* fs = (volatile LAS unsigned*)(lds + NR * PK_SLOT);
        const unsigned long long t0 = __builtin_amdgcn_s_memrealtime();
        unsigned polls = 0;
        if (rot == rank) {
            for (;;) {
                unsigned v = call;
                if ((lane & 7) <= NXC) v = __hip_atomic_load((const unsigned*)(myws + OFF_FS + lane * 128), __ATOMIC_RELAXED, __HIP_MEMORY_SCOPE_SYSTEM);
                if ((__ballot(v < call) & 0xFEFEFEFEFEFEFEFEull) == 0 && (lane & 7) < 2) fs[(lane >> 3) * 2 + (lane & 1)] = v;
                if (__all(v >= call)) break;
                if (gave_up(t0, polls, call, myws + OFF_TMO)) { if (lane == 0) *(volatile LAS unsigned*)(ldsflag + 1) = 1u; break; }
                __builtin_amdgcn_s_sleep(MON_SLEEP);
            }
        } else {
            const int peer = rot;
            const __amdgpu_buffer_rsrc_t rs = make_rsrc(pws, WS_TOTAL);
            unsigned dead = 0;
            auto wait_staged = [&](int idx) {
                while ((unsigned)__builtin_amdgcn_readfirstlane(fs[peer * 2 + idx]) < call) {
                    if (__builtin_amdgcn_readfirstlane(*(volatile LAS unsigned*)(ldsflag + 1)) != 0u) { dead = OOB; break; }
                    __builtin_amdgcn_s_sleep(1);
                }
                asm volatile("" ::: "memory");
            };
            const unsigned lo = lane * 16, one = lane == 0 ? 0u : OOB;
            const unsigned tkbase = (call - 1u) * NPC - NCOMM;
            Piece cur = piece_of(pk_, rank, peer, call);
            unsigned pfo = OOB;
            u32x4 v[2];
            wait_staged(cur.staged);
#pragma unroll
            for (int u = 0; u < 2; ++u) {
                __builtin_amdgcn_raw_buffer_store_b128((u32x4){0u, 0u, 0u, 0u}, rm, OOB + lo + u * 1024, 0, AUX_SYS);
                v[u] = __builtin_amdgcn_raw_buffer_load_b128(rs, lo + u * 1024, cur.so, AUX_SYS);
                asm volatile("" ::: "memory");
            }
            int tk = __builtin_amdgcn_raw_ptr_buffer_atomic_add_i32(1, rm, one + OFF_TK + peer * 256, 0, 0);
            bool granted = false;
            for (;;) {
                Piece nxt = cur;
                bool more = false;
                unsigned lo2 = OOB + lo, tko = OOB;
#pragma unroll
                for (int u = 0; u < 4; ++u) {
                    __builtin_amdgcn_raw_buffer_store_b128(v[u & 1], rm, lo + u * 1024, cur.dso, AUX_SYS);
                    if (u < 2) v[u & 1] = __builtin_amdgcn_raw_buffer_load_b128(rs, lo + (u + 2) * 1024, cur.so, AUX_SYS);
                    else v[u & 1] = __builtin_amdgcn_raw_buffer_load_b128(rs, lo2 + (u - 2) * 1024, nxt.so, AUX_SYS);
                    if (u == 0) {
                        WAIT_VM(4);
                        __builtin_amdgcn_raw_buffer_store_b32(call, rm, one | pfo | dead, 0, AUX_SYS);
                    }
                    if (u == 1) {
                        const unsigned pc = (unsigned)__builtin_amdgcn_readfirstlane(tk) - tkbase;
                        more = pc < (unsigned)NPC;
                        if (more) { nxt = piece_of((int)pc, rank, peer, call); wait_staged(nxt.staged); lo2 = lo; tko = one; }
                        if (pc + GRANT_EARLY >= (unsigned)NPC && !granted) {
                            granted = true;
                            if (lane < 2 && pk_ + lane * 32 < NHELP) flag_store(myws + OFF_FD + (32 + pk_ + lane * 32) * 128 + 64 + peer * 8, call);
                        }
                    }
                    if (u == 3) tk = __builtin_amdgcn_raw_ptr_buffer_atomic_add_i32(1, rm, tko + OFF_TK + peer * 256, 0, 0);
                    asm volatile("" ::: "memory");
                }
                pfo = cur.fo;
                cur = nxt;
                if (!more) break;
            }
            WAIT_VM(0);
            __builtin_amdgcn_raw_buffer_store_b32(call, rm, one | pfo | dead, 0, AUX_SYS);
        }
    }
    const float qnan = __builtin_nanf("");
    float sxw = p.sx[0] * p.sw[0];
    asm volatile("" : "+v"(sxw));
    const __amdgpu_buffer_rsrc_t ro = make_rsrc(p.out, OUT_BYTES);
    LAS uint8_t* pk = lds + wv * PK_SLOT;
    const int ti = 2 * (xcd & 3) + (slot & 1);
    int tJ = 2 * (slot >> 1) + (xcd >> 2);
    for (int job = comm ? 2 : (slot < 4 ? 0 : 1); job < 2; ++job) {
    f32x16 acc[4];
#pragma unroll
    for (int c = 0; c < 4; ++c)
#pragma unroll
        for (int r = 0; r < 16; ++r) acc[c][r] = 0.f;
    const int ent = entry_of(wv, ti, tJ), grg = grg_of(rank, ti, ent), gcb = gcb_of(rank, tJ);
    for (int ph = 0; ph < 3; ++ph) {
        if (wv == 0) {
            if (ph == 0 && tJ < 28) __builtin_amdgcn_s_sleep(PH0_DELAY);
            const bool ok = wait_operands(myws, ti, tJ, lane, call, ph, rank);
            if (ph == 1 && tJ < 8) __builtin_amdgcn_fence(__ATOMIC_ACQUIRE, "");
            else __builtin_amdgcn_fence(__ATOMIC_ACQUIRE, "agent");
            if (lane == 0 && !ok) *ldsflag = 1u;
            asm volatile("s_waitcnt vmcnt(0)" ::: "memory");
        }
        WAIT_LGKM0();
        BAR();
        const int nstep = ph == 0 ? 8 : 28;
        const int sh = ph == 0 ? 3 : 2, sbase = ph == 0 ? 0 : rank + 1, sadd = ph == 2 ? 4 : 0;
        const uint8_t* ga = myws + (ph == 0 ? OFF_SX + (call & 1u) * S_COPY + (unsigned)grg * 16384u : OFF_XS + (unsigned)(ti * 10 + ent) * 131072u) + lane * 16;
        const uint8_t* gb = myws + (ph == 0 ? OFF_SW + (call & 1u) * S_COPY + (unsigned)gcb * 65536u : ws_block(tJ, call)) + wv * 1024 + lane * 16;
        auto dma = [&](int n, int sl) {
            const unsigned s = (unsigned)((((sbase + (n >> sh)) & 7) << 3) + (n & ((1 << sh) - 1)) + sadd);
            LAS uint8_t* d = lds + sl * SLOT;
            __builtin_amdgcn_global_load_lds((const void*)(ga + s * 2048u), (LAS void*)(d + wv * 2048), 16, 0, 0);
            __builtin_amdgcn_global_load_lds((const void*)(ga + s * 2048u + 1024), (LAS void*)(d + wv * 2048 + 1024), 16, 0, 0);
            __builtin_amdgcn_global_load_lds((const void*)(gb + s * 8192u), (LAS void*)(d + 16384 + wv * 1024), 16, 0, 0);
        };
        LAS const uint8_t* pa = lds + wv * 2048 + lane * 16;
        LAS const uint8_t* pb = lds + 16384 + lane * 16;
#pragma unroll
        for (int s = 0; s < NS; ++s) dma(s, s);
        WAIT_VM(3 * (NS - 1));
        BAR();
        Frags f0, f1;
        read_frags(f0, pa, pb);
        int sl = 0;
#define SGB(mask, n) __builtin_amdgcn_sched_group_barrier(mask, n, 0)
#define KSTEP(FC, FN, S, VMN, DO_DMA) do { \
            __builtin_amdgcn_sched_barrier(0); \
            WAIT_VM(VMN); WAIT_LGKM0(); BAR(); \
            __builtin_amdgcn_sched_barrier(0); \
            mma<0, 1>(acc, FC); \
            __builtin_amdgcn_sched_barrier(0); \
            const int nsl = sl + 1 == NS ? 0 : sl + 1; \
            read_frags(FN, pa + nsl * SLOT, pb + nsl * SLOT); \
            if (DO_DMA) dma((S) + NS, sl); \
            mma<1, 4>(acc, FC); \
            SGB(0x100, 6); SGB(0x8, 1); SGB(0x100, 4); SGB(0x10, 1); \
            SGB(0x8, 1); SGB(0x10, 2); \
            SGB(0x8, 1); \
            sl = nsl; } while (0)
        for (int s = 0; s < nstep - NS; s += 2) {
            KSTEP(f0, f1, s, 3 * (NS - 2), true);
            KSTEP(f1, f0, s + 1, 3 * (NS - 2), true);
        }
        static_assert(NS == 6, "tail below is written for NS == 6");
        KSTEP(f0, f1, 0, 12, false);
        KSTEP(f1, f0, 0, 9, false);
        KSTEP(f0, f1, 0, 6, false);
        KSTEP(f1, f0, 0, 3, false);
        KSTEP(f0, f1, 0, 0, false);
        mma<0, 4>(acc, f1);
#undef KSTEP
    }
    WAIT_LGKM0();
    BAR();
    const float scale = *ldsflag != 0u ? qnan : sxw;
    if (wv < 2) {
        const unsigned off0 = (unsigned)((grg & 15) * 32 + 4 * (lane >> 5)) * (GN * 4) + (unsigned)(gcb * 128 + 4 * (lane & 31)) * 4;
#pragma unroll
        for (int r = 0; r < 16; ++r) {
            f32x4 v;
#pragma unroll
            for (int c = 0; c < 4; ++c) { const float y = acc[c][r] * scale; v[c] = y <= 0.f ? 0.f : y; }
            __builtin_amdgcn_raw_buffer_store_b128(__builtin_bit_cast(u32x4, v), ro, off0 + (unsigned)((r & 3) + 8 * (r >> 2)) * (GN * 4), 0, AUX_SYS);
        }
    } else {
        const int tile = tJ * 8 + ti;
        unsigned mk[2] = {0u, 0u}, base = 512 + (unsigned)(size_t)pk;
        const unsigned dump = (unsigned)(size_t)pk + PK_DATA_END + lane * 4;
#pragma unroll
        for (int r = 0; r < 16; ++r)
#pragma unroll
            for (int c = 0; c < 4; ++c) {
                const float y = acc[c][r] * scale;
                const bool nz = !(y <= 0.f);
                const unsigned long long m = __ballot(nz);
                const unsigned pos = __builtin_amdgcn_mbcnt_hi((unsigned)(m >> 32), __builtin_amdgcn_mbcnt_lo((unsigned)m, 0u)) * 4 + base;
                *(LAS float*)(size_t)(nz ? pos : dump) = y;
                base += (unsigned)__builtin_popcountll(m) * 4;
                asm("" : "+s"(base));
                mk[r >> 3] = mk[r >> 3] * 2 + (nz ? 1u : 0u);
            }
        base -= (unsigned)(size_t)pk;
        *(LAS u32x2*)(pk + lane * 8) = (u32x2){mk[0], mk[1]};
        const unsigned bytes = base;
        const __amdgpu_buffer_rsrc_t rp = make_rsrc(myws + OFF_PK + (unsigned)(tile * 8 + wv) * PK_SLOT, (bytes + 127u) & ~127u);
        for (unsigned o = 0; o < bytes; o += 5120) {
            u32x4 v[5];
#pragma unroll
            for (int u = 0; u < 5; ++u) v[u] = *(LAS const u32x4*)(pk + (o + u * 1024 < PK_SLOT ? o + u * 1024 : 0u) + lane * 16);
#pragma unroll
            for (int u = 0; u < 5; ++u) __builtin_amdgcn_raw_buffer_store_b128(v[u], rp, o + u * 1024 + lane * 16, 0, AUX_SYS);
        }
        WAIT_VM(0);
        if (lane == 0) __hip_atomic_store((unsigned long long*)(p.ws[(grg >> 4) & 7] + OFF_FD + tile * 128 + mirror(wv, ti) * 8),
                                          ((unsigned long long)call << 32) | bytes, __ATOMIC_RELAXED, __HIP_MEMORY_SCOPE_SYSTEM);
    }
    if (job == 0) tJ += 28;
    }
    if (rot == rank && lane == 0) __hip_atomic_store(cntp, call, __ATOMIC_RELAXED, __HIP_MEMORY_SCOPE_AGENT);
    const bool helper = b >= 32 && b < 32 + NHELP;
    if (rot == rank || !(comm || helper)) return;
    {
        const int peer = rot;
        const unsigned nu = ((rank ^ peer) & 4) == 0 ? 224u : 216u;
        const unsigned one = lane == 0 ? 0u : OOB;
        const unsigned tkoff = one + OFF_TK + peer * 256 + 128, tkbase = (call - 1u) * (nu + NCOMM + NHELP);
        const unsigned long long t0 = __builtin_amdgcn_s_memrealtime();
        unsigned polls = 0;
        if (helper) {
            for (;;) {
                unsigned f = 0;
                if (lane == 0) f = __hip_atomic_load((const unsigned*)(myws + OFF_FD + b * 128 + 64 + peer * 8), __ATOMIC_RELAXED, __HIP_MEMORY_SCOPE_SYSTEM);
                if ((unsigned)__builtin_amdgcn_readfirstlane(f) >= call || gave_up(t0, polls, call, myws + OFF_TMO)) break;
                __builtin_amdgcn_s_sleep(GATE_SLEEP);
            }
        }
        *(LAS unsigned*)(pk + PK_DATA_END + 256 + lane * 4) = 0u;
        int g = __builtin_amdgcn_raw_ptr_buffer_atomic_add_i32(1, rm, tkoff, 0, 0);
        unsigned n = (unsigned)__builtin_amdgcn_readfirstlane(g) - tkbase;
        u32x2 fnext = {0u, 0u};
        int win = 2;
        while (n < nu) {
            const int un = unit_of(n, rank, peer), tile = un >> 3, role = un & 7, ui = tile & 7, uJ = tile >> 3;
            const int g2 = grg_of(rank, ui, entry_of(role, ui, uJ));
            const unsigned off0 = (unsigned)((g2 & 15) * 32 + 4 * (lane >> 5)) * (GN * 4) + (unsigned)(gcb_of(peer, uJ) * 128 + 4 * (lane & 31)) * 4;
            bool ok = true;
            unsigned bytes = __builtin_amdgcn_readfirstlane(fnext.x);
            if ((unsigned)__builtin_amdgcn_readfirstlane(fnext.y) < call) {
                for (;;) {
                    unsigned long long f = 0;
                    if (lane == 0) f = __hip_atomic_load((const unsigned long long*)(myws + OFF_FD + tile * 128 + role * 8), __ATOMIC_RELAXED, __HIP_MEMORY_SCOPE_SYSTEM);
                    bytes = __builtin_amdgcn_readfirstlane((unsigned)f);
                    if ((unsigned)__builtin_amdgcn_readfirstlane((unsigned)(f >> 32)) >= call) break;
                    if (gave_up(t0, polls, call, myws + OFF_TMO)) { ok = false; break; }
                    __builtin_amdgcn_s_sleep(RX_SLEEP);
                }
            }
            asm volatile("" ::: "memory");
            if (bytes > PK_DATA_END) bytes = PK_DATA_END;
            if (!ok) bytes = 0;
            const int nld = (int)((bytes + 1023u) >> 10);
            const uint8_t* src = p.ws[peer] + OFF_PK + (unsigned)(tile * 8 + mirror(role, ui)) * PK_SLOT + lane * 16;
            unsigned ta = 0;
            const unsigned tstart = (unsigned)__builtin_amdgcn_s_memrealtime();
            int i = 0;
            const int behind = (int)(nu - 1u - n);
            auto issue = [&]() {
                const int c = win < nld - i ? win : nld - i;
                ta = (unsigned)__builtin_amdgcn_s_memrealtime();
                for (int u = 0; u < c; ++u, ++i)
                    if ((unsigned)(i * 1024 + (lane & ~7) * 16) < bytes)
                        __builtin_amdgcn_global_load_lds((const void*)(src + i * 1024), (LAS void*)(pk + i * 1024), 16, 0, AUX_SYS);
            };
            auto land = [&]() {
                WAIT_VM(0);
                const unsigned now = (unsigned)__builtin_amdgcn_s_memrealtime(), lat = now - ta;
                if (lat < LAT_LO) win = win * 2 < WMAX ? win * 2 : WMAX;
                else if (lat > LAT_HI) { if (win > 1) win >>= 1; else if (lat > LAT_BACK) __builtin_amdgcn_s_sleep(BACKOFF); }
                const int left = NCOMM + NHELP + behind - TKAHEAD - (int)((now - tstart) / UNIT_TICKS);
                const int wmin = left > EG2 ? 1 : (left > EG3 ? 2 : (left > EG4 ? 3 : (left > EG4 / 2 ? 4 : 8)));
                win = win < wmin ? wmin : win;
            };
            issue(); WAIT_VM(0);
            u32x2 mm = *(LAS const u32x2*)(pk + lane * 8);
            if (!ok) { mm = (u32x2){0u, 0u}; *(LAS float*)(pk + PK_DATA_END + 256 + lane * 4) = qnan; }
            WAIT_LGKM0();
            unsigned ad[64];
            unsigned base = 512, mid = 0;
            const unsigned zero = PK_DATA_END + 256 + lane * 4;
#pragma unroll
            for (int q = 0; q < 2; ++q) {
                const bool fly = i < nld;
                if (fly) issue();
#pragma unroll
                for (int k = q * 32; k < (q + 1) * 32; ++k) {
                    const bool nz = (int)((k < 32 ? mm.x : mm.y) << (k & 31)) < 0;
                    const unsigned long long m = __ballot(nz);
                    const unsigned pos = __builtin_amdgcn_mbcnt_hi((unsigned)(m >> 32), __builtin_amdgcn_mbcnt_lo((unsigned)m, 0u)) * 4 + base;
                    base += (unsigned)__builtin_popcountll(m) * 4;
                    ad[k] = (unsigned)(size_t)pk + (nz ? pos : zero);
                    asm volatile("" : "+v"(ad[k]));
                }
                if (q == 0) mid = base;
                if (fly) land();
            }
            auto expand = [&](int r0) {
#pragma unroll
                for (int r = r0; r < r0 + 8; ++r) {
                    f32x4 v;
#pragma unroll
                    for (int c = 0; c < 4; ++c) v[c] = *(LAS const float*)(size_t)ad[r * 4 + c];
                    __builtin_amdgcn_raw_buffer_store_b128(__builtin_bit_cast(u32x4, v), ro, off0 + (unsigned)((r & 3) + 8 * (r >> 2)) * (GN * 4), 0, AUX_SYS);
                }
            };
            while ((unsigned)i * 1024u < mid && i < nld) { issue(); land(); }
            __builtin_amdgcn_s_waitcnt(0x0F70);
            {
                f32x4 hv[8];
#pragma unroll
                for (int r = 0; r < 8; ++r)
#pragma unroll
                    for (int c = 0; c < 4; ++c) hv[r][c] = *(LAS const float*)(size_t)ad[r * 4 + c];
                asm volatile("" ::: "memory");
                if (i < nld) issue();
#pragma unroll
                for (int r = 0; r < 8; ++r)
                    __builtin_amdgcn_raw_buffer_store_b128(__builtin_bit_cast(u32x4, hv[r]), ro, off0 + (unsigned)((r & 3) + 8 * (r >> 2)) * (GN * 4), 0, AUX_SYS);
                WAIT_VM(8);
            }
            while (i + 2 * win < nld) { issue(); land(); }
            g = __builtin_amdgcn_raw_ptr_buffer_atomic_add_i32(1, rm, tkoff, 0, 0);
            if (i < nld) { issue(); land(); }
            n = (unsigned)__builtin_amdgcn_readfirstlane(g) - tkbase;
            { const int u2 = unit_of(n < nu ? n : 0u, rank, peer);
              fnext = __builtin_amdgcn_raw_buffer_load_b64(rm, one + OFF_FD + (u2 >> 3) * 128 + (u2 & 7) * 8, 0, AUX_SYS); }
            while (i < nld) { issue(); land(); }
            __builtin_amdgcn_s_waitcnt(0x0F70);
            expand(8);
        }
    }
}

}

extern "C" void kernel_launch(int rank, int nranks, ncclComm_t comm, hipStream_t stream,
                              void* const* d_in, const int* in_sizes, int n_in,
                              void* d_out, int out_size, const int* in_dtypes, int out_dtype,
                              void* const* d_out_all, void* const* d_ws_all, size_t ws_bytes) {
    (void)comm; (void)in_dtypes; (void)out_dtype; (void)d_out_all;
    if (nranks != NR || n_in != 4 || in_sizes[0] != GM * KSH || in_sizes[1] != KSH * GN || out_size != MSH * GN ||
        ws_bytes < WS_TOTAL || (unsigned)rank >= (unsigned)NR) {
        fprintf(stderr, "kernel_launch: unexpected shapes\n");
        return;
    }
    Params p;
    p.x = (const uint8_t*)d_in[0]; p.w = (const uint8_t*)d_in[1];
    p.sx = (const float*)d_in[2]; p.sw = (const float*)d_in[3];
    p.out = (uint8_t*)d_out;
    for (int i = 0; i < NR; ++i) p.ws[i] = (uint8_t*)d_ws_all[i];
    p.rank = rank; p.pad = 0; p.myws = p.ws[rank];
    HIP_CHECK(hipFuncSetAttribute((const void*)gemm_xchg, hipFuncAttributeMaxDynamicSharedMemorySize, LDS_BYTES));
    hipLaunchKernelGGL(gemm_xchg, dim3(NWG), dim3(NTHR), LDS_BYTES, stream, p);
    HIP_CHECK(hipGetLastError());
}
```
